```python
import numpy as np

kernel_name = "gpu_ag_gemm_m4096_k4096_n32768_bf16_silu"


ABSMAX_RTOL = 0.008

COMPUTE_DTYPE = "bfloat16"
EPILOGUE = "silu"
IN_QUANT = None
OUT_QUANT = None
IN_DTYPES = {"x": "bfloat16", "w_mat": "bfloat16"}
WORLD_SIZE = 8

def _to_bf16(x):
    u = np.ascontiguousarray(x, dtype=np.float32).view(np.uint32)
    return ((u + 0x7FFF + ((u >> 16) & 1)) & 0xFFFF0000).view(np.float32)

def setup_inputs():

    x = _to_bf16(np.random.randn(4096, 4096).astype(np.float32))
    w_mat = _to_bf16((np.random.randn(4096, 32768) / np.sqrt(4096)).astype(np.float32))
    return {"x": x, "w_mat": w_mat}

def _epilogue(y):
    return (y / (1.0 + np.exp(-np.clip(y, -60.0, 60.0)))).astype(np.float32)

def reference(x, w_mat):
    y = (x @ w_mat).astype(np.float32)
    return _epilogue(y)
```

```cpp
#include <hip/hip_runtime.h>
#include <rccl/rccl.h>
#include <cstdio>
#include <cstdint>

#define HIP_CHECK(expr) do { hipError_t _e = (expr); if (_e != hipSuccess) { \
    fprintf(stderr, "HIP error %s at %s:%d\n", hipGetErrorString(_e), __FILE__, __LINE__); \
    return; } } while (0)

#define LAS __attribute__((address_space(3)))
typedef short bf16x8 __attribute__((ext_vector_type(8)));
typedef short s16x4 __attribute__((ext_vector_type(4)));
typedef float f32x4 __attribute__((ext_vector_type(4)));
typedef unsigned u32x4 __attribute__((ext_vector_type(4)));
typedef unsigned u32x2 __attribute__((ext_vector_type(2)));

constexpr int NR = 8;
constexpr int GM = 4096, GK = 4096, GN = 4096;
constexpr int BK = 64, NT = GK / BK;
constexpr int HTB = 128 * BK * 2;
constexpr int LDS_SHARE = 8 * HTB;
constexpr int LDS_BYTES = 9 * HTB;
constexpr int NWG = 256;

constexpr size_t WS_CTR = 0;
constexpr size_t WS_TMO = 256;
constexpr size_t WS_FLAGS = 4096;
constexpr size_t WS_STAMPS = 512 * 1024;
constexpr int LDS_PAD = 0;
constexpr int NEV = 14;
constexpr size_t WS_GATHER = (size_t)1 << 20;
constexpr size_t GATHER_BYTES = (size_t)GM * GK * 2;
constexpr size_t WS_NEED = WS_GATHER + 2 * GATHER_BYTES;
constexpr unsigned long long DEADLINE_TICKS = 20000000ull;

struct Params {
    const char* x;
    const char* w;
    char* out;
    char* ws[NR];
    int rank;
    int mode;
    int dbg;
    int slot;
};

__device__ __forceinline__ int lds_byte(int r, int c) { const int st = (r >> 4) * 2 + (c >> 5), rr = r & 15, cc = c & 31, ob = rr * 64 + cc * 2; return st * 1024 + (ob ^ (((ob >> 9) & 1) << 5)); }
__device__ __forceinline__ void stage_rc(int b, int& R, int& C) { const int st = b / 1024, sb = b % 1024, swz = sb ^ (((sb >> 9) & 1) << 5); R = (st >> 1) * 16 + swz / 64; C = (st & 1) * 32 + (swz % 64) / 2; }

__device__ __forceinline__ unsigned cvt_pk_bf16(float lo, float hi) { unsigned r; asm volatile("v_cvt_pk_bf16_f32 %0, %1, %2" : "=v"(r) : "v"(lo), "v"(hi)); return r; }
__device__ __forceinline__ float silu1(float y) {
    const float c = __builtin_amdgcn_fmed3f(y, -60.0f, 60.0f);
    const float e = __builtin_amdgcn_exp2f(c * -1.4426950408889634f);
    return y * __builtin_amdgcn_rcpf(1.0f + e);
}

__global__ __launch_bounds__(512, 2) void ag_gemm_silu(const Params p) {
    extern __shared__ __attribute__((aligned(16))) unsigned char smem_raw[];
    LAS unsigned char* lds = (LAS unsigned char*)smem_raw;
    const int tid = threadIdx.x, wid = __builtin_amdgcn_readfirstlane(tid >> 6), lane = tid & 63, wr = wid >> 2, wc = wid & 3, fr = lane & 15, fq = lane >> 4;
    const unsigned long long t_start = __builtin_amdgcn_s_memrealtime();
    const int bid = blockIdx.x, xcd = bid & 7, idx = bid >> 3;
    const int pm = (xcd >> 1) * 4 + (idx & 3), pn = (xcd & 1) * 8 + (idx >> 2);
    const int rank = p.rank;
    const bool local = (pm >> 1) == rank;
    const bool nocomm = local || p.mode >= 1;
    const int unit = (((pm - 2 * rank - 2) & 15) << 4) + pn;
    char* const wsl = p.ws[rank];

#define WAIT_V(n) asm volatile("s_waitcnt vmcnt(" #n ")" ::: "memory")
#define WAIT_L(n) asm volatile("s_waitcnt lgkmcnt(" #n ")" ::: "memory")
#define BAR __builtin_amdgcn_s_barrier()
#define SCHED __builtin_amdgcn_sched_barrier(0)
#define GLDS(g, l) __builtin_amdgcn_global_load_lds((const unsigned*)(g), (LAS unsigned*)(l), 16, 0, 0)
#define STAGE(bufoff, gbase, voff, voffx) do { \
        if (wid != 3) { GLDS((const char*)(gbase) + (voff)[0], lds + (bufoff) + ldsw); GLDS((const char*)(gbase) + (voff)[1], lds + (bufoff) + ldsw + 8192); } \
        if (wid < 2) GLDS((const char*)(gbase) + (voffx), lds + (bufoff) + 3072 + wid * 8192); } while (0)
#define WAIT_STAGE() do { if (wid < 2) WAIT_V(9); else if (wid != 3) WAIT_V(6); } while (0)
    const unsigned ldsw = (unsigned)wid * 1024u;
    auto src_of = [&](int e, int s) {
        const int g = unit + 256 * e, tau = g / 56, sp = (g - tau * 56) / 7;
        return p.x + (size_t)(sp * 64 + s * 8 + (lane >> 3)) * (GK * 2) + tau * 128 + (lane & 7) * 16;
    };
    GLDS(src_of(0, wid), lds + LDS_SHARE + ldsw);
    const unsigned seq = (unsigned)__builtin_amdgcn_readfirstlane(__hip_atomic_load((const unsigned*)(wsl + WS_CTR), __ATOMIC_RELAXED, __HIP_MEMORY_SCOPE_AGENT)) / NWG + 1u;
    const unsigned par = (p.dbg & 128) ? (seq & 7u) : (seq & 1u);

    const unsigned rsA = local ? (unsigned)(GK * 2) : 128u;
    unsigned voffA[2], voffB[2], voffAx, voffBx;
#pragma unroll
    for (int i = 0; i < 3; ++i) {
        const int tt = i < 2 ? tid : 192 + lane, ii = i < 2 ? i : (wid & 1);
        int R, C; stage_rc(tt * 16 + ii * 8192, R, C);
        const unsigned va = (unsigned)R * rsA + (unsigned)C * 2u;
        const int k = (tt >> 4) + 32 * ii, c = tt & 15, f = (k & 3) | (((k >> 3) & 1) << 2);
        const unsigned vb = (unsigned)(k * (GN * 2) + ((c ^ (f << 1)) * 16));
        if (i < 2) { voffA[i] = va; voffB[i] = vb; } else { voffAx = va; voffBx = vb; }
    }
    const size_t kstepA = local ? (size_t)128 : (size_t)32768, hstepA = (size_t)128 * rsA;
    const size_t kstepB = (size_t)BK * GN * 2, hstepB = 256;
    const char* cA = local ? p.x + (size_t)(pm & 1) * 256 * (GK * 2) : wsl + WS_GATHER + par * GATHER_BYTES + (size_t)pm * (NT * 32768);
    const char* cB = p.w + pn * 512;
    const unsigned* flagp = (const unsigned*)(wsl + WS_FLAGS) + pm * (NT * 32) + lane;
    unsigned* tmo = (unsigned*)(wsl + WS_TMO);
    const int aoff = lds_byte(wr * 64 + fr, fq * 8);
    int boff[2];
    { const int q = (lane >> 2) & 3, pp = lane & 3, g = fq, f = q | ((g & 1) << 2);
#pragma unroll
      for (int n = 0; n < 2; ++n) boff[n] = (int)(unsigned)(uintptr_t)lds + 4 * HTB + (8 * g + q) * 256 + (((wc * 2 + n) ^ f) * 32) + 8 * pp; }

#define SA(b, h) (((b) * 2 + (h)) * HTB)
#define SB(b, h) ((4 + (b) * 2 + (h)) * HTB)
#define LDA(dst, b, h) do { _Pragma("unroll") for (int m = 0; m < 4; ++m) _Pragma("unroll") for (int k = 0; k < 2; ++k) dst[m][k] = *(const LAS bf16x8*)(lds + SA(b, h) + aoff + m * 2048 + k * 1024); } while (0)
#define TRRD(dst, addr, off) asm volatile("ds_read_b64_tr_b16 %0, %1 offset:%2" : "=v"(dst) : "v"(addr), "i"(off) : "memory")
#define LDB(dst, b, h) do { _Pragma("unroll") for (int n = 0; n < 2; ++n) _Pragma("unroll") for (int k = 0; k < 2; ++k) { \
        TRRD(dst[n][k][0], boff[n], (SB(b, h) - 4 * HTB) + k * 8192); TRRD(dst[n][k][1], boff[n], (SB(b, h) - 4 * HTB) + k * 8192 + 1024); } } while (0)
#define WAIT_LB(Bt) asm volatile("s_waitcnt lgkmcnt(0)" : "+v"(Bt[0][0][0]), "+v"(Bt[0][0][1]), "+v"(Bt[0][1][0]), "+v"(Bt[0][1][1]), "+v"(Bt[1][0][0]), "+v"(Bt[1][0][1]), "+v"(Bt[1][1][0]), "+v"(Bt[1][1][1]) :: "memory")
#define BFR(Bt, n, k) __builtin_shufflevector(Bt[n][k][0], Bt[n][k][1], 0, 1, 2, 3, 4, 5, 6, 7)
#define MMA(ai, bj, At, Bt) do { __builtin_amdgcn_s_setprio(1); _Pragma("unroll") for (int m = 0; m < 4; ++m) _Pragma("unroll") for (int n = 0; n < 2; ++n) _Pragma("unroll") for (int k = 0; k < 2; ++k) \
        acc[ai][bj][m][n] = __builtin_amdgcn_mfma_f32_16x16x32_bf16(BFR(Bt, n, k), At[m][k], acc[ai][bj][m][n], 0, 0, 0); __builtin_amdgcn_s_setprio(0); } while (0)

#define STAMP(i)
    const bool nosend = (p.mode == 1 || ((p.dbg & 4) && rank != 0) || ((p.dbg & 8) && rank == 0));
    int ev = nosend ? NEV : 0;
    bool bad = false;

    auto dst_rsrc = [&](int e, unsigned& off, unsigned*& fl) {
        const int g = unit + 256 * e, tau = g / 56, j = g - tau * 56, sp = j / 7, pr = (p.dbg & 1) ? rank : (rank + 1 + j - sp * 7) & 7;
        off = (unsigned)(WS_GATHER + par * GATHER_BYTES) + (unsigned)(((rank * 2 + (sp >> 2)) * NT + tau) * 32768 + (sp & 3) * 8192);
        fl = (unsigned*)(p.ws[pr] + WS_FLAGS) + ((rank * 2 + (sp >> 2)) * NT + tau) * 32 + (sp & 3) * 8;
        return __builtin_amdgcn_make_buffer_rsrc((void*)p.ws[pr], 0, (int)WS_NEED, 0x00020000);
    };
    auto ev_tau = [&](int e) { return (unit + 256 * e) / 56; };
    auto ev_time_due = [&](int e) { return (__builtin_amdgcn_s_memrealtime() - t_start) >= (unsigned long long)(ev_tau(e) * p.slot); };

    f32x4 acc[2][2][4][2];
#pragma unroll
    for (int a = 0; a < 2; ++a)
#pragma unroll
        for (int b = 0; b < 2; ++b)
#pragma unroll
            for (int m = 0; m < 4; ++m)
#pragma unroll
                for (int n = 0; n < 2; ++n) acc[a][b][m][n] = (f32x4){0.f, 0.f, 0.f, 0.f};
    bf16x8 At[4][2]; s16x4 B0[2][2][2], B1[2][2][2];

    STAGE(SB(0, 0), cB, voffB, voffBx); STAGE(SB(0, 1), cB + hstepB, voffB, voffBx);
    STAGE(SB(1, 0), cB + kstepB, voffB, voffBx); STAGE(SB(1, 1), cB + kstepB + hstepB, voffB, voffBx);
    STAMP(33);
    while (ev < NEV && ev_tau(ev) <= 5) {
        while (!ev_time_due(ev)) __builtin_amdgcn_s_sleep(1);
        WAIT_V(0);
        const u32x4 d = *(const LAS u32x4*)(lds + LDS_SHARE + (ev & 1) * 8192 + ldsw + lane * 16);
        unsigned off; unsigned* fl; const __amdgpu_buffer_rsrc_t rs = dst_rsrc(ev, off, fl);
        __builtin_amdgcn_raw_buffer_store_b128(d, rs, (int)(lane * 16), (int)(off + ldsw), 17);
        if (ev + 1 < NEV) GLDS(src_of(ev + 1, wid), lds + LDS_SHARE + ((ev + 1) & 1) * 8192 + ldsw);
        WAIT_V(0);
        if (lane == 0 && !(p.dbg & 2)) __hip_atomic_store(fl + wid, seq, __ATOMIC_RELAXED, __HIP_MEMORY_SCOPE_SYSTEM);
        ++ev;
    }
    auto slow_wait = [&](int kt) {
        const unsigned long long t0 = __builtin_amdgcn_s_memrealtime();
        for (;;) {
            const unsigned v = __hip_atomic_load(flagp + kt * 32, __ATOMIC_RELAXED, __HIP_MEMORY_SCOPE_SYSTEM);
            if (__all(v >= seq)) break;
            if ((unsigned)__builtin_amdgcn_readfirstlane(__hip_atomic_load(tmo, __ATOMIC_RELAXED, __HIP_MEMORY_SCOPE_AGENT)) == seq || __builtin_amdgcn_s_memrealtime() - t0 > DEADLINE_TICKS) {
                __hip_atomic_store(tmo, seq, __ATOMIC_RELAXED, __HIP_MEMORY_SCOPE_AGENT); bad = true; break; }
            __builtin_amdgcn_s_sleep(2);
        }
        __builtin_amdgcn_fence(__ATOMIC_ACQUIRE, "");
    };
    const bool poller = (wid == 2) && !nocomm;
    if (poller) slow_wait(0);
    STAMP(34);
    WAIT_V(0);
    BAR;
    STAGE(SA(0, 0), cA, voffA, voffAx); STAGE(SA(0, 1), cA + hstepA, voffA, voffAx);
    if (wr == 1) BAR;
    STAGE(SA(1, 0), cA + kstepA, voffA, voffAx);
    if (wid < 2) WAIT_V(3); else WAIT_V(2);
    BAR;
    BAR;
    unsigned fv = 0;
    bool pend = false;
    for (int t = 0; t < NT; t += 2) {
        const bool last = (t == NT - 2);
        const char* a1 = cA + (size_t)(t + 1) * kstepA;
        const char* a2 = last ? cA : cA + (size_t)(t + 2) * kstepA; const char* b2 = last ? cB : cB + (size_t)(t + 2) * kstepB;
        const char* a3 = a2 + kstepA; const char* b3 = b2 + kstepB;
        if (poller && !last) { if (!__all(fv >= seq)) slow_wait(t + 2); }
        bool go = false;
        if (wid == 3 && ev < NEV) go = (t >= ev_tau(ev) - 7) || ev_time_due(ev);
        u32x4 d0, d1; unsigned off = 0; unsigned* fl = nullptr; __amdgpu_buffer_rsrc_t rs = __builtin_amdgcn_make_buffer_rsrc((void*)wsl, 0, 0, 0x00020000);
        const LAS unsigned char* shp = lds + LDS_SHARE + (ev & 1) * 8192 + lane * 16;
        if (go) {
            WAIT_V(8);
            rs = dst_rsrc(ev, off, fl);
            if (ev + 1 < NEV) {
#pragma unroll
                for (int s = 0; s < 8; ++s) GLDS(src_of(ev + 1, s), lds + LDS_SHARE + ((ev + 1) & 1) * 8192 + s * 1024);
            }
            d0 = *(const LAS u32x4*)(shp); d1 = *(const LAS u32x4*)(shp + 1024);
        }
#define SEND2(s) do { if (go) { __builtin_amdgcn_raw_buffer_store_b128(d0, rs, (int)(lane * 16), (int)(off + (s) * 1024), 17); __builtin_amdgcn_raw_buffer_store_b128(d1, rs, (int)(lane * 16), (int)(off + (s) * 1024 + 1024), 17); \
            if ((s) < 6) { d0 = *(const LAS u32x4*)(shp + ((s) + 2) * 1024); d1 = *(const LAS u32x4*)(shp + ((s) + 3) * 1024); } } } while (0)
        STAMP(t >> 1);
        LDB(B0, 0, 0); SCHED; LDA(At, 0, 0); STAGE(SA(1, 1), a1 + hstepA, voffA, voffAx);
        WAIT_L(8); BAR; WAIT_LB(B0); MMA(0, 0, At, B0); BAR; SCHED;
        LDB(B1, 0, 1); STAGE(SB(0, 0), b2, voffB, voffBx); SEND2(0);
        BAR; WAIT_LB(B1); MMA(0, 1, At, B1); BAR;
        LDA(At, 0, 1); STAGE(SA(0, 0), a2, voffA, voffAx); SEND2(2);
        BAR; WAIT_L(0); MMA(1, 0, At, B0); BAR; SCHED;
        STAGE(SB(0, 1), b2 + hstepB, voffB, voffBx); SEND2(4);
        WAIT_STAGE();
        BAR; MMA(1, 1, At, B1); BAR;
        LDB(B0, 1, 0); SCHED; LDA(At, 1, 0); STAGE(SA(0, 1), a2 + hstepA, voffA, voffAx); SEND2(6);
        if (poller) { const unsigned* fa = flagp + ((t + 4 < NT) ? (t + 4) : 0) * 32;
            asm volatile("global_load_dword %0, %1, off sc0 sc1" : "=v"(fv) : "v"(fa) : "memory"); }
        WAIT_L(8); BAR; WAIT_LB(B0); MMA(0, 0, At, B0); BAR; SCHED;
        LDB(B1, 1, 1); STAGE(SB(1, 0), b3, voffB, voffBx);
        if (pend) {
            if (!go) WAIT_V(0); else if (ev + 1 < NEV) WAIT_V(16); else WAIT_V(8);
            unsigned o2; unsigned* f2; (void)dst_rsrc(ev - 1, o2, f2);
            if (lane < 8 && !(p.dbg & 2)) __hip_atomic_store(f2 + lane, seq, __ATOMIC_RELAXED, __HIP_MEMORY_SCOPE_SYSTEM);
            pend = false;
        }
        if (go) { pend = true; ++ev; }
        BAR; WAIT_LB(B1); MMA(0, 1, At, B1); BAR;
        LDA(At, 1, 1); STAGE(SA(1, 0), a3, voffA, voffAx);
        BAR; WAIT_L(0); MMA(1, 0, At, B0); BAR; SCHED;
        STAGE(SB(1, 1), b3 + hstepB, voffB, voffBx);
        WAIT_STAGE(); asm volatile("" : "+v"(fv));
        BAR; MMA(1, 1, At, B1); BAR;
    }
    WAIT_V(0);
    if (wr == 0) BAR;
    STAMP(32);
    if (pend) {
        unsigned o2; unsigned* f2; (void)dst_rsrc(ev - 1, o2, f2);
        if (lane < 8 && !(p.dbg & 2)) __hip_atomic_store(f2 + lane, seq, __ATOMIC_RELAXED, __HIP_MEMORY_SCOPE_SYSTEM);
    }

    if (p.mode == 0) bad = ((unsigned)__builtin_amdgcn_readfirstlane(__hip_atomic_load(tmo, __ATOMIC_RELAXED, __HIP_MEMORY_SCOPE_AGENT)) == seq);
    {
        const int colb = pn * 256 + wc * 32 + ((fq & 1) * 16 + (fq >> 1) * 8);
        char* orow = p.out + ((size_t)(pm * 256 + wr * 64 + fr) * GN + colb) * 2;
#pragma unroll
        for (int ai = 0; ai < 2; ++ai)
#pragma unroll
            for (int m = 0; m < 4; ++m)
#pragma unroll
                for (int bj = 0; bj < 2; ++bj) {
                    const f32x4 v0 = acc[ai][bj][m][0], v1 = acc[ai][bj][m][1];
                    unsigned a0 = cvt_pk_bf16(silu1(v0[0]), silu1(v0[1])), a1 = cvt_pk_bf16(silu1(v0[2]), silu1(v0[3]));
                    unsigned b0 = cvt_pk_bf16(silu1(v1[0]), silu1(v1[1])), b1 = cvt_pk_bf16(silu1(v1[2]), silu1(v1[3]));
                    const auto s0 = __builtin_amdgcn_permlane16_swap(a0, b0, false, false);
                    const auto s1 = __builtin_amdgcn_permlane16_swap(a1, b1, false, false);
                    u32x4 o; o.x = s0[0]; o.y = s1[0]; o.z = s0[1]; o.w = s1[1];
                    if (bad) o = (u32x4){0x7fc07fc0u, 0x7fc07fc0u, 0x7fc07fc0u, 0x7fc07fc0u};
                    *(u32x4*)(orow + ((size_t)(ai * 128 + m * 16) * GN + bj * 128) * 2) = o;
                }
    }
    if (tid == 0) __hip_atomic_fetch_add((unsigned*)(wsl + WS_CTR), 1u, __ATOMIC_RELAXED, __HIP_MEMORY_SCOPE_AGENT);
}

static int g_mode = 0, g_slot = 140, g_dbg = 0;

extern "C" void kernel_launch(int rank, int nranks, ncclComm_t comm,
                              hipStream_t stream,
                              void* const* d_in, const int* in_sizes, int n_in,
                              void* d_out, int out_size,
                              const int* in_dtypes, int out_dtype,
                              void* const* d_out_all,
                              void* const* d_ws_all, size_t ws_bytes) {
    if (nranks != NR || ws_bytes < WS_NEED || (unsigned)rank >= (unsigned)NR) { fprintf(stderr, "unsupported configuration\n"); return; }
    static bool init = false;
    if (!init) {
        HIP_CHECK(hipFuncSetAttribute((const void*)ag_gemm_silu, hipFuncAttributeMaxDynamicSharedMemorySize, LDS_BYTES));
        init = true;
    }
    Params p;
    p.x = (const char*)d_in[0]; p.w = (const char*)d_in[1]; p.out = (char*)d_out;
    for (int i = 0; i < NR; ++i) p.ws[i] = (char*)d_ws_all[i];
    p.rank = rank; p.mode = g_mode; p.slot = g_slot; p.dbg = g_dbg;
    hipLaunchKernelGGL(ag_gemm_silu, dim3(NWG), dim3(512), LDS_BYTES, stream, p);
    HIP_CHECK(hipGetLastError());
}
```

```cpp
#include <hip/hip_runtime.h>
#include <rccl/rccl.h>
#include <cstdio>
#include <cstdint>
#include <type_traits>

#define HIP_CHECK(expr) do { hipError_t _e = (expr); if (_e != hipSuccess) { \
    fprintf(stderr, "HIP error %s at %s:%d\n", hipGetErrorString(_e), __FILE__, __LINE__); \
    return; } } while (0)

#define LAS __attribute__((address_space(3)))
#define GAS __attribute__((address_space(1)))
typedef short bf16x8 __attribute__((ext_vector_type(8)));
typedef short s16x4 __attribute__((ext_vector_type(4)));
typedef float f32x4 __attribute__((ext_vector_type(4)));
typedef unsigned u32x4 __attribute__((ext_vector_type(4)));
typedef unsigned u32x2 __attribute__((ext_vector_type(2)));

constexpr int NR = 8;
constexpr int GM = 4096, GK = 4096, GN = 4096;
constexpr int BK = 64, NT = GK / BK;
constexpr int HTB = 128 * BK * 2;
constexpr int LDS_SHARE = 8 * HTB;
constexpr int LDS_BYTES = 8 * HTB + 3 * 8192;
constexpr int NWG = 256;

constexpr size_t WS_CTR = 0;
constexpr size_t WS_TMO = 256;
constexpr size_t WS_FLAGS = 4096;
constexpr size_t WS_STAMPS = 512 * 1024;
constexpr int LDS_PAD = 0;
constexpr int NEV = 14;
constexpr size_t WS_GATHER = (size_t)1 << 20;
constexpr size_t GATHER_BYTES = (size_t)GM * GK * 2;
constexpr size_t WS_NEED = WS_GATHER + 2 * GATHER_BYTES;
constexpr unsigned long long DEADLINE_TICKS = 20000000ull;

struct Params {
    const char* x;
    const char* w;
    char* out;
    char* ws[NR];
    int rank;
    int mode;
    int dbg;
    int pit;
    int pslot;
    int slot;
};

__device__ __forceinline__ int lds_byte(int r, int c) { const int st = (r >> 4) * 2 + (c >> 5), rr = r & 15, cc = c & 31, ob = rr * 64 + cc * 2; return st * 1024 + (ob ^ (((ob >> 9) & 1) << 5)); }
__device__ __forceinline__ void stage_rc(int b, int& R, int& C) { const int st = b / 1024, sb = b % 1024, swz = sb ^ (((sb >> 9) & 1) << 5); R = (st >> 1) * 16 + swz / 64; C = (st & 1) * 32 + (swz % 64) / 2; }

__device__ __forceinline__ unsigned cvt_pk_bf16(float lo, float hi) { unsigned r; asm volatile("v_cvt_pk_bf16_f32 %0, %1, %2" : "=v"(r) : "v"(lo), "v"(hi)); return r; }
__device__ __forceinline__ float silu1(float y) {
    const float c = __builtin_amdgcn_fmed3f(y, -60.0f, 60.0f);
    const float e = __builtin_amdgcn_exp2f(c * -1.4426950408889634f);
    return y * __builtin_amdgcn_rcpf(1.0f + e);
}

__global__ __launch_bounds__(512, 2) void ag_gemm_silu(const Params p) {
    extern __shared__ __attribute__((aligned(16))) unsigned char smem_raw[];
    LAS unsigned char* lds = (LAS unsigned char*)smem_raw;
    const int tid = threadIdx.x, wid = __builtin_amdgcn_readfirstlane(tid >> 6), lane = tid & 63, wr = wid >> 2, wc = wid & 3, fr = lane & 15, fq = lane >> 4;
    const unsigned long long t_start = __builtin_amdgcn_s_memrealtime();
    const int bid = blockIdx.x, xcd = bid & 7, idx = bid >> 3;
    const int pm = (xcd >> 1) * 4 + (idx & 3), pn = (xcd & 1) * 8 + (idx >> 2);
    const int rank = p.rank;
    const bool local = (pm >> 1) == rank;
    const bool nocomm = local || p.mode >= 1;
    const int unit = (((pm - 2 * rank - 2) & 15) << 4) + pn;
    char* const wsl = p.ws[rank];

#define WAIT_V(n) asm volatile("s_waitcnt vmcnt(" #n ")" ::: "memory")
#define WAIT_L(n) asm volatile("s_waitcnt lgkmcnt(" #n ")" ::: "memory")
#define BAR __builtin_amdgcn_s_barrier()
#define SCHED __builtin_amdgcn_sched_barrier(0)
#define BLDS(rs, l, voff, soff) __builtin_amdgcn_raw_ptr_buffer_load_lds(rs, (LAS void*)(l), 16, (int)(voff), (int)(soff), 0, 0)
#define STAGE(bufoff, rs, soff, voff, voffx) do { \
        if (wid != 3) { BLDS(rs, lds + (bufoff) + ldsw, (voff)[0], soff); BLDS(rs, lds + (bufoff) + ldsw + 8192, (voff)[1], soff); } \
        if (wid < 2) BLDS(rs, lds + (bufoff) + 3072 + wid * 8192, voffx, soff); } while (0)
#define WAIT_STAGE() do { if (wid < 2) WAIT_V(9); else if (wid != 3) WAIT_V(6); } while (0)
    const unsigned ldsw = (unsigned)wid * 1024u;
    const __amdgpu_buffer_rsrc_t rsX = __builtin_amdgcn_make_buffer_rsrc((void*)p.x, 0, 512 * GK * 2, 0x00020000);
    const unsigned vsrc = (unsigned)((lane >> 3) * (GK * 2) + (lane & 7) * 16);
    auto src_off = [&](int e) { const int g = unit + 256 * e, tau = g / 56, sp = (g - tau * 56) / 7; return (unsigned)(sp * 64 * (GK * 2) + tau * 128); };
    BLDS(rsX, lds + LDS_SHARE + ldsw, vsrc, src_off(0) + wid * 8 * (GK * 2));
    const unsigned seq = (unsigned)__builtin_amdgcn_readfirstlane(__hip_atomic_load((const unsigned*)(wsl + WS_CTR), __ATOMIC_RELAXED, __HIP_MEMORY_SCOPE_AGENT)) / NWG + 1u;
    const unsigned long long wsv = (unsigned long long)(uintptr_t)p.ws[lane & 7];
    unsigned ws_lo = (unsigned)wsv, ws_hi = (unsigned)(wsv >> 32);
    asm volatile("" : "+v"(ws_lo), "+v"(ws_hi));

    const unsigned rsA = local ? (unsigned)(GK * 2) : 128u;
    unsigned voffA[2], voffB[2], voffAx, voffBx;
#pragma unroll
    for (int i = 0; i < 3; ++i) {
        const int tt = i < 2 ? tid : 192 + lane, ii = i < 2 ? i : (wid & 1);
        int R, C; stage_rc(tt * 16 + ii * 8192, R, C);
        const unsigned va = (unsigned)R * rsA + (unsigned)C * 2u;
        const int k = (tt >> 4) + 32 * ii, c = tt & 15, f = (k & 3) | (((k >> 3) & 1) << 2);
        const unsigned vb = (unsigned)(k * (GN * 2) + ((c ^ (f << 1)) * 16));
        if (i < 2) { voffA[i] = va; voffB[i] = vb; } else { voffAx = va; voffBx = vb; }
    }
    const unsigned kstepA = local ? 128u : 32768u, hstepA = 128u * rsA;
    const unsigned kstepB = (unsigned)(BK * GN * 2), hstepB = 256u;
    const __amdgpu_buffer_rsrc_t rsB = __builtin_amdgcn_make_buffer_rsrc((void*)(p.w + pn * 512), 0, GK * GN * 2 - pn * 512, 0x00020000);
    const int aoff = lds_byte(wr * 64 + fr, fq * 8);
    int boff[2];
    { const int q = (lane >> 2) & 3, pp = lane & 3, g = fq, f = q | ((g & 1) << 2);
#pragma unroll
      for (int n = 0; n < 2; ++n) boff[n] = (int)(unsigned)(uintptr_t)lds + 4 * HTB + (8 * g + q) * 256 + (((wc * 2 + n) ^ f) * 32) + 8 * pp; }

#define SA(b, h) (((b) * 2 + (h)) * HTB)
#define SB(b, h) ((4 + (b) * 2 + (h)) * HTB)
#define LDA(dst, b, h) do { _Pragma("unroll") for (int m = 0; m < 4; ++m) _Pragma("unroll") for (int k = 0; k < 2; ++k) dst[m][k] = *(const LAS bf16x8*)(lds + SA(b, h) + aoff + m * 2048 + k * 1024); } while (0)
#define TRRD(dst, addr, off) asm volatile("ds_read_b64_tr_b16 %0, %1 offset:%2" : "=v"(dst) : "v"(addr), "i"(off) : "memory")
#define LDB(dst, b, h) do { _Pragma("unroll") for (int n = 0; n < 2; ++n) _Pragma("unroll") for (int k = 0; k < 2; ++k) { \
        TRRD(dst[n][k][0], boff[n], (SB(b, h) - 4 * HTB) + k * 8192); TRRD(dst[n][k][1], boff[n], (SB(b, h) - 4 * HTB) + k * 8192 + 1024); } } while (0)
#define WAIT_LB(Bt) asm volatile("s_waitcnt lgkmcnt(0)" : "+v"(Bt[0][0][0]), "+v"(Bt[0][0][1]), "+v"(Bt[0][1][0]), "+v"(Bt[0][1][1]), "+v"(Bt[1][0][0]), "+v"(Bt[1][0][1]), "+v"(Bt[1][1][0]), "+v"(Bt[1][1][1]) :: "memory")
#define BFR(Bt, n, k) __builtin_shufflevector(Bt[n][k][0], Bt[n][k][1], 0, 1, 2, 3, 4, 5, 6, 7)
#define MMA(ai, bj, At, Bt) do { __builtin_amdgcn_s_setprio(1); _Pragma("unroll") for (int m = 0; m < 4; ++m) _Pragma("unroll") for (int n = 0; n < 2; ++n) _Pragma("unroll") for (int k = 0; k < 2; ++k) \
        acc[ai][bj][m][n] = __builtin_amdgcn_mfma_f32_16x16x32_bf16(BFR(Bt, n, k), At[m][k], acc[ai][bj][m][n], 0, 0, 0); __builtin_amdgcn_s_setprio(0); } while (0)

#define STAMP(i)
#define PST(ph)
    const bool nosend = (p.mode == 1 || ((p.dbg & 4) && rank != 0) || ((p.dbg & 8) && rank == 0));
    int ev = nosend ? NEV : 0;
    bool bad = false;
    auto ev_tau = [&](int e) { return (unit + 256 * e) / 56; };

    f32x4 acc[2][2][4][2];
#pragma unroll
    for (int a = 0; a < 2; ++a)
#pragma unroll
        for (int b = 0; b < 2; ++b)
#pragma unroll
            for (int m = 0; m < 4; ++m)
#pragma unroll
                for (int n = 0; n < 2; ++n) acc[a][b][m][n] = (f32x4){0.f, 0.f, 0.f, 0.f};
    bf16x8 At[4][2]; s16x4 B0[2][2][2], B1[2][2][2];

    STAGE(SB(0, 0), rsB, 0, voffB, voffBx); STAGE(SB(0, 1), rsB, hstepB, voffB, voffBx);
    STAGE(SB(1, 0), rsB, kstepB, voffB, voffBx); STAGE(SB(1, 1), rsB, kstepB + hstepB, voffB, voffBx);
    const unsigned par = (p.dbg & 128) ? (seq & 7u) : (seq & 1u);
    const __amdgpu_buffer_rsrc_t rsAr = __builtin_amdgcn_make_buffer_rsrc(local ? (void*)(p.x + (size_t)(pm & 1) * 256 * (GK * 2)) : (void*)(wsl + WS_GATHER + par * GATHER_BYTES + (size_t)pm * (NT * 32768)), 0, NT * 32768, 0x00020000);
    const unsigned* flagp = (const unsigned*)(wsl + WS_FLAGS) + pm * (NT * 32) + lane;
    unsigned* tmo = (unsigned*)(wsl + WS_TMO);
    auto dst_rsrc = [&](int e, unsigned& off, GAS unsigned*& fl) {
        const int g = unit + 256 * e, tau = g / 56, j = g - tau * 56, sp = j / 7, pr = (p.dbg & 1) ? rank : (rank + 1 + j - sp * 7) & 7;
        char* base = (char*)(uintptr_t)(((unsigned long long)(unsigned)__builtin_amdgcn_readlane((int)ws_hi, pr) << 32) | (unsigned)__builtin_amdgcn_readlane((int)ws_lo, pr));
        off = (unsigned)(WS_GATHER + par * GATHER_BYTES) + (unsigned)(((rank * 2 + (sp >> 2)) * NT + tau) * 32768 + (sp & 3) * 8192);
        fl = (GAS unsigned*)(base + WS_FLAGS) + ((rank * 2 + (sp >> 2)) * NT + tau) * 32 + (sp & 3) * 8;
        return __builtin_amdgcn_make_buffer_rsrc((void*)base, 0, (int)WS_NEED, 0x00020000);
    };
    STAMP(33);
    if (wid < 2) WAIT_V(12); else if (wid != 3) WAIT_V(8); else WAIT_V(0);
    while (ev < NEV && ev_tau(ev) <= 5) {
        while ((__builtin_amdgcn_s_memrealtime() - t_start) < (unsigned long long)(ev_tau(ev) * p.pslot)) __builtin_amdgcn_s_sleep(1);
        const u32x4 d = *(const LAS u32x4*)(lds + LDS_SHARE + (ev % 3) * 8192 + ldsw + lane * 16);
        unsigned off; GAS unsigned* fl; const __amdgpu_buffer_rsrc_t rs = dst_rsrc(ev, off, fl);
        __builtin_amdgcn_raw_buffer_store_b128(d, rs, (int)(lane * 16), (int)(off + ldsw), 17);
        BLDS(rsX, lds + LDS_SHARE + ((ev + 1) % 3) * 8192 + ldsw, vsrc, src_off(ev + 1) + wid * 8 * (GK * 2));
        WAIT_V(0);
        if (lane == 0 && !(p.dbg & 2)) __hip_atomic_store(fl + wid, seq, __ATOMIC_RELAXED, __HIP_MEMORY_SCOPE_SYSTEM);
        ++ev;
    }
    if (nosend) {} else BLDS(rsX, lds + LDS_SHARE + ((ev + 1) % 3) * 8192 + ldsw, vsrc, src_off(ev + 1) + wid * 8 * (GK * 2));
    auto slow_wait = [&](int kt) {
        const unsigned long long t0 = __builtin_amdgcn_s_memrealtime();
        for (;;) {
            const unsigned v = __hip_atomic_load(flagp + kt * 32, __ATOMIC_RELAXED, __HIP_MEMORY_SCOPE_SYSTEM);
            if (__all(v >= seq)) break;
            if ((unsigned)__builtin_amdgcn_readfirstlane(__hip_atomic_load(tmo, __ATOMIC_RELAXED, __HIP_MEMORY_SCOPE_AGENT)) == seq || __builtin_amdgcn_s_memrealtime() - t0 > DEADLINE_TICKS) {
                __hip_atomic_store(tmo, seq, __ATOMIC_RELAXED, __HIP_MEMORY_SCOPE_AGENT); bad = true; break; }
            __builtin_amdgcn_s_sleep(2);
        }
        __builtin_amdgcn_fence(__ATOMIC_ACQUIRE, "");
    };
    const bool poller = (wid == 2) && !nocomm;
    if (poller) slow_wait(0);
    STAMP(34);
    WAIT_V(0);
    BAR;
    STAGE(SA(0, 0), rsAr, 0, voffA, voffAx); STAGE(SA(0, 1), rsAr, hstepA, voffA, voffAx);
    if (wr == 1) BAR;
    STAGE(SA(1, 0), rsAr, kstepA, voffA, voffAx);
    if (wid < 2) WAIT_V(3); else WAIT_V(2);
    BAR;
    BAR;
    unsigned fv = 0;
    GAS unsigned* fl_cur = (GAS unsigned*)wsl;
    bool pend = false;
    unsigned long long now = t_start;
    auto main_loop = [&](auto role_c) __attribute__((always_inline)) {
    constexpr int ROLE = decltype(role_c)::value;
#define RSTAGE(bufoff, rs, soff, voff, voffx) do { \
        if constexpr (ROLE != 2) { BLDS(rs, lds + (bufoff) + ldsw, (voff)[0], soff); BLDS(rs, lds + (bufoff) + ldsw + 8192, (voff)[1], soff); } \
        if constexpr (ROLE == 0) BLDS(rs, lds + (bufoff) + 3072 + wid * 8192, voffx, soff); } while (0)
#define RWAIT_STAGE() do { if constexpr (ROLE == 0) WAIT_V(9); else if constexpr (ROLE == 1) WAIT_V(6); } while (0)
    for (int t = 0; t < NT; t += 2) {
        const bool last = (t == NT - 2);
        const unsigned a1 = (unsigned)(t + 1) * kstepA;
        const unsigned a2 = last ? 0u : (unsigned)(t + 2) * kstepA, b2 = last ? 0u : (unsigned)(t + 2) * kstepB;
        const unsigned a3 = a2 + kstepA, b3 = b2 + kstepB;
        if constexpr (ROLE == 1) {
        if (poller && !last) { if (!__all(fv >= seq)) slow_wait(t + 2); }
        }
        bool go = false;
        if constexpr (ROLE == 2) { if (ev < NEV) { const int tau = ev_tau(ev); go = (t >= tau - 7) || ((now - t_start) >= (unsigned long long)(tau * p.slot)); } }
        u32x4 d0, d1; unsigned off = 0, so = 0; __amdgpu_buffer_rsrc_t rs = rsX;
        const unsigned shp = (unsigned)(uintptr_t)lds + LDS_SHARE + (ev % 3) * 8192 + lane * 16;
        LAS unsigned char* const sl = lds + LDS_SHARE + ((ev + 2) % 3) * 8192;
#define SHRD(s) asm volatile("ds_read_b128 %0, %2 offset:%3\n\tds_read_b128 %1, %2 offset:%4" : "=v"(d0), "=v"(d1) : "v"(shp), "i"((s) * 1024), "i"((s) * 1024 + 1024) : "memory")
        GAS unsigned* const fl_prev = fl_cur;
        if constexpr (ROLE == 2) { if (go) {
            WAIT_V(16);
            rs = dst_rsrc(ev, off, fl_cur);
            so = src_off(ev + 2);
            SHRD(0);
        } }
#define PREF(s) do { if constexpr (ROLE == 2) { if (go) BLDS(rsX, sl + (s) * 1024, vsrc, so + (s) * 8 * (GK * 2)); } } while (0)
#define SEND2(s) do { if constexpr (ROLE == 2) { if (go) { asm volatile("s_waitcnt lgkmcnt(0)" : "+v"(d0), "+v"(d1) :: "memory"); \
            __builtin_amdgcn_raw_buffer_store_b128(d0, rs, (int)(lane * 16), (int)(off + (s) * 1024), 17); __builtin_amdgcn_raw_buffer_store_b128(d1, rs, (int)(lane * 16), (int)(off + (s) * 1024 + 1024), 17); \
            if ((s) < 6) SHRD((s) + 2); } } } while (0)
        STAMP(t >> 1);
        PST(0); LDB(B0, 0, 0); SCHED; LDA(At, 0, 0); RSTAGE(SA(1, 1), rsAr, a1 + hstepA, voffA, voffAx); PREF(0);
        WAIT_L(8); BAR; WAIT_LB(B0); MMA(0, 0, At, B0); BAR; SCHED;
        PST(1); SEND2(0); LDB(B1, 0, 1); RSTAGE(SB(0, 0), rsB, b2, voffB, voffBx); PREF(1);
        BAR; WAIT_LB(B1); MMA(0, 1, At, B1); BAR;
        PST(2); SEND2(2); LDA(At, 0, 1); RSTAGE(SA(0, 0), rsAr, a2, voffA, voffAx); PREF(2);
        BAR; WAIT_L(0); MMA(1, 0, At, B0); BAR; SCHED;
        PST(3); SEND2(4); RSTAGE(SB(0, 1), rsB, b2 + hstepB, voffB, voffBx); PREF(3);
        RWAIT_STAGE();
        BAR; MMA(1, 1, At, B1); BAR;
        PST(4); SEND2(6); LDB(B0, 1, 0); SCHED; LDA(At, 1, 0); RSTAGE(SA(0, 1), rsAr, a2 + hstepA, voffA, voffAx); PREF(4);
        if constexpr (ROLE == 1) { if (poller) { const unsigned* fa = flagp + ((t + 4 < NT) ? (t + 4) : 0) * 32;
            asm volatile("global_load_dword %0, %1, off sc0 sc1" : "=v"(fv) : "v"(fa) : "memory"); } }
        WAIT_L(8); BAR; WAIT_LB(B0); MMA(0, 0, At, B0); BAR; SCHED;
        PST(5); LDB(B1, 1, 1); RSTAGE(SB(1, 0), rsB, b3, voffB, voffBx);
        if constexpr (ROLE == 2) {
        if (pend) {
            if (go) WAIT_V(17); else WAIT_V(4);
            if (lane < 8 && !(p.dbg & 2)) __hip_atomic_store(fl_prev + lane, seq, __ATOMIC_RELAXED, __HIP_MEMORY_SCOPE_SYSTEM);
            pend = false;
        }
        PREF(5);
        if (go) { pend = true; ++ev; }
        }
        BAR; WAIT_LB(B1); MMA(0, 1, At, B1); BAR;
        PST(6); LDA(At, 1, 1); RSTAGE(SA(1, 0), rsAr, a3, voffA, voffAx); PREF(6);
        BAR; WAIT_L(0); MMA(1, 0, At, B0); BAR; SCHED;
        PST(7); RSTAGE(SB(1, 1), rsB, b3 + hstepB, voffB, voffBx); PREF(7);
        if constexpr (ROLE == 2) now = __builtin_amdgcn_s_memrealtime();
        RWAIT_STAGE();
        if constexpr (ROLE == 1) asm volatile("" : "+v"(fv));
        BAR; MMA(1, 1, At, B1); BAR;
        PST(8);
    }
    };
    if (wid < 2) main_loop(std::integral_constant<int, 0>{});
    else if (wid != 3) main_loop(std::integral_constant<int, 1>{});
    else main_loop(std::integral_constant<int, 2>{});
    WAIT_V(0);
    if (wr == 0) BAR;
    STAMP(32);
    if (pend) {
        if (lane < 8 && !(p.dbg & 2)) __hip_atomic_store(fl_cur + lane, seq, __ATOMIC_RELAXED, __HIP_MEMORY_SCOPE_SYSTEM);
    }

    if (p.mode == 0) bad = ((unsigned)__builtin_amdgcn_readfirstlane(__hip_atomic_load(tmo, __ATOMIC_RELAXED, __HIP_MEMORY_SCOPE_AGENT)) == seq);
    {
        const int colb = pn * 256 + wc * 32 + ((fq & 1) * 16 + (fq >> 1) * 8);
        char* orow = p.out + ((size_t)(pm * 256 + wr * 64 + fr) * GN + colb) * 2;
#pragma unroll
        for (int ai = 0; ai < 2; ++ai)
#pragma unroll
            for (int m = 0; m < 4; ++m)
#pragma unroll
                for (int bj = 0; bj < 2; ++bj) {
                    const f32x4 v0 = acc[ai][bj][m][0], v1 = acc[ai][bj][m][1];
                    unsigned a0 = cvt_pk_bf16(silu1(v0[0]), silu1(v0[1])), a1 = cvt_pk_bf16(silu1(v0[2]), silu1(v0[3]));
                    unsigned b0 = cvt_pk_bf16(silu1(v1[0]), silu1(v1[1])), b1 = cvt_pk_bf16(silu1(v1[2]), silu1(v1[3]));
                    const auto s0 = __builtin_amdgcn_permlane16_swap(a0, b0, false, false);
                    const auto s1 = __builtin_amdgcn_permlane16_swap(a1, b1, false, false);
                    u32x4 o; o.x = s0[0]; o.y = s1[0]; o.z = s0[1]; o.w = s1[1];
                    if (bad) o = (u32x4){0x7fc07fc0u, 0x7fc07fc0u, 0x7fc07fc0u, 0x7fc07fc0u};
                    *(u32x4*)(orow + ((size_t)(ai * 128 + m * 16) * GN + bj * 128) * 2) = o;
                }
    }
    if (tid == 0) __hip_atomic_fetch_add((unsigned*)(wsl + WS_CTR), 1u, __ATOMIC_RELAXED, __HIP_MEMORY_SCOPE_AGENT);
}

static int g_mode = 0, g_slot = 145, g_pslot = 115, g_dbg = 0, g_pit = 10;

extern "C" void kernel_launch(int rank, int nranks, ncclComm_t comm,
                              hipStream_t stream,
                              void* const* d_in, const int* in_sizes, int n_in,
                              void* d_out, int out_size,
                              const int* in_dtypes, int out_dtype,
                              void* const* d_out_all,
                              void* const* d_ws_all, size_t ws_bytes) {
    if (nranks != NR || ws_bytes < WS_NEED || (unsigned)rank >= (unsigned)NR) { fprintf(stderr, "unsupported configuration\n"); return; }
    static bool init = false;
    if (!init) {
        HIP_CHECK(hipFuncSetAttribute((const void*)ag_gemm_silu, hipFuncAttributeMaxDynamicSharedMemorySize, LDS_BYTES));
        init = true;
    }
    Params p;
    p.x = (const char*)d_in[0]; p.w = (const char*)d_in[1]; p.out = (char*)d_out;
    for (int i = 0; i < NR; ++i) p.ws[i] = (char*)d_ws_all[i];
    p.rank = rank; p.mode = g_mode; p.slot = g_slot; p.pslot = g_pslot; p.pit = g_pit; p.dbg = g_dbg;
    hipLaunchKernelGGL(ag_gemm_silu, dim3(NWG), dim3(512), LDS_BYTES, stream, p);
    HIP_CHECK(hipGetLastError());
}
```

```cpp
#include <hip/hip_runtime.h>
#include <rccl/rccl.h>
#include <cstdio>
#include <cstdint>
#include <type_traits>

#define HIP_CHECK(expr) do { hipError_t _e = (expr); if (_e != hipSuccess) { \
    fprintf(stderr, "HIP error %s at %s:%d\n", hipGetErrorString(_e), __FILE__, __LINE__); \
    return; } } while (0)

#define LAS __attribute__((address_space(3)))
#define GAS __attribute__((address_space(1)))
typedef short bf16x8 __attribute__((ext_vector_type(8)));
typedef short s16x4 __attribute__((ext_vector_type(4)));
typedef float f32x4 __attribute__((ext_vector_type(4)));
typedef unsigned u32x4 __attribute__((ext_vector_type(4)));
typedef unsigned u32x2 __attribute__((ext_vector_type(2)));

constexpr int NR = 8;
constexpr int GM = 4096, GK = 4096, GN = 4096;
constexpr int BK = 64, NT = GK / BK;
constexpr int HTB = 128 * BK * 2;
constexpr int LDS_CTL = 8 * HTB;
constexpr int LDS_BYTES = 8 * HTB + 1024;
constexpr int NWG = 256;

constexpr size_t WS_CTR = 0;
constexpr size_t WS_TMO = 256;
constexpr size_t WS_FLAGS = 4096;
constexpr size_t WS_STAMPS = 512 * 1024;
constexpr int NEV = 16;
constexpr size_t WS_GATHER = (size_t)1 << 20;
constexpr size_t GATHER_BYTES = (size_t)GM * GK * 2;
constexpr size_t WS_NEED = WS_GATHER + 2 * GATHER_BYTES;
constexpr unsigned long long DEADLINE_TICKS = 20000000ull;

struct Params {
    const char* x;
    const char* w;
    char* out;
    char* ws[NR];
    int rank;
    int mode;
    int dbg;
    int pit;
    int pslot;
    int slot;
};

__device__ __forceinline__ int lds_byte(int r, int c) { const int st = (r >> 4) * 2 + (c >> 5), rr = r & 15, cc = c & 31, ob = rr * 64 + cc * 2; return st * 1024 + (ob ^ (((ob >> 9) & 1) << 5)); }
__device__ __forceinline__ void stage_rc(int b, int& R, int& C) { const int st = b / 1024, sb = b % 1024, swz = sb ^ (((sb >> 9) & 1) << 5); R = (st >> 1) * 16 + swz / 64; C = (st & 1) * 32 + (swz % 64) / 2; }

__device__ __forceinline__ unsigned cvt_pk_bf16(float lo, float hi) { unsigned r; asm volatile("v_cvt_pk_bf16_f32 %0, %1, %2" : "=v"(r) : "v"(lo), "v"(hi)); return r; }
__device__ __forceinline__ float silu1(float y) {
    const float c = __builtin_amdgcn_fmed3f(y, -60.0f, 60.0f);
    const float e = __builtin_amdgcn_exp2f(c * -1.4426950408889634f);
    return y * __builtin_amdgcn_rcpf(1.0f + e);
}

__global__ __launch_bounds__(512, 2) void ag_gemm_silu(const Params p) {
    extern __shared__ __attribute__((aligned(16))) unsigned char smem_raw[];
    LAS unsigned char* lds = (LAS unsigned char*)smem_raw;
    const int tid = threadIdx.x, wid = __builtin_amdgcn_readfirstlane(tid >> 6), lane = tid & 63, wr = wid >> 2, wc = wid & 3, fr = lane & 15, fq = lane >> 4;
    const unsigned long long t_start = __builtin_amdgcn_s_memrealtime();
    const int bid = blockIdx.x, xcd = bid & 7, idx = bid >> 3;
    const int pm = (xcd >> 1) * 4 + (idx & 3), pn = (xcd & 1) * 8 + (idx >> 2);
    const int rank = p.rank;
    const bool local = (pm >> 1) == rank;
    const bool nocomm = local || p.mode >= 1;
    const int unit = (((pm - 2 * rank - 2) & 15) << 4) + pn;
    char* const wsl = p.ws[rank];

#define WAIT_V(n) asm volatile("s_waitcnt vmcnt(" #n ")" ::: "memory")
#define WAIT_L(n) asm volatile("s_waitcnt lgkmcnt(" #n ")" ::: "memory")
#define BAR __builtin_amdgcn_s_barrier()
#define SCHED __builtin_amdgcn_sched_barrier(0)
#define BLDS(rs, l, voff, soff) __builtin_amdgcn_raw_ptr_buffer_load_lds(rs, (LAS void*)(l), 16, (int)(voff), (int)(soff), 0, 0)
#define STAGE(bufoff, rs, soff, voff, voffx) do { \
        if (wid != 3) { BLDS(rs, lds + (bufoff) + ldsw, (voff)[0], soff); BLDS(rs, lds + (bufoff) + ldsw + 8192, (voff)[1], soff); } \
        if (wid < 2) BLDS(rs, lds + (bufoff) + 3072 + wid * 8192, voffx, soff); } while (0)
#define WAIT_STAGE() do { if (wid < 2) WAIT_V(9); else if (wid != 3) WAIT_V(6); } while (0)
    const unsigned ldsw = (unsigned)wid * 1024u;
    const __amdgpu_buffer_rsrc_t rsX = __builtin_amdgcn_make_buffer_rsrc((void*)p.x, 0, 512 * GK * 2, 0x00020000);
    const unsigned vsrc = (unsigned)((lane >> 3) * (GK * 2) + (lane & 7) * 16);
    const bool nosend = (p.mode == 1 || ((p.dbg & 4) && rank != 0) || ((p.dbg & 8) && rank == 0));
    int ev = nosend ? NEV : 0;
    const unsigned so0 = (unsigned)((unit & 63) * 8 * (GK * 2) + (unit >> 6) * 128);
    u32x4 d0 = {0u, 0u, 0u, 0u}, dn = {0u, 0u, 0u, 0u};
    if (wid == 3 && ev < NEV) dn = __builtin_amdgcn_raw_buffer_load_b128(rsX, (int)vsrc, (int)so0, 0);
    const unsigned seqv = __hip_atomic_load((const unsigned*)(wsl + WS_CTR), __ATOMIC_RELAXED, __HIP_MEMORY_SCOPE_AGENT);
    unsigned long long wsv = (unsigned long long)(uintptr_t)p.ws[0];
#pragma unroll
    for (int i = 1; i < NR; ++i) { unsigned long long w = (unsigned long long)(uintptr_t)p.ws[i]; asm volatile("" : "+s"(w)); wsv = ((lane & 7) == i) ? w : wsv; }
    const unsigned ws_lo = (unsigned)wsv, ws_hi = (unsigned)(wsv >> 32);

    const unsigned rsA = local ? (unsigned)(GK * 2) : 128u;
    unsigned voffA[2], voffB[2], voffAx, voffBx;
#pragma unroll
    for (int i = 0; i < 3; ++i) {
        const int tt = i < 2 ? tid : 192 + lane, ii = i < 2 ? i : (wid & 1);
        int R, C; stage_rc(tt * 16 + ii * 8192, R, C);
        const unsigned va = (unsigned)R * rsA + (unsigned)C * 2u;
        const int k = (tt >> 4) + 32 * ii, c = tt & 15, f = (k & 3) | (((k >> 3) & 1) << 2);
        const unsigned vb = (unsigned)(k * (GN * 2) + ((c ^ (f << 1)) * 16));
        if (i < 2) { voffA[i] = va; voffB[i] = vb; } else { voffAx = va; voffBx = vb; }
    }
    const unsigned kstepA = local ? 128u : 32768u, hstepA = 128u * rsA;
    const unsigned kstepB = (unsigned)(BK * GN * 2), hstepB = 256u;
    const __amdgpu_buffer_rsrc_t rsB = __builtin_amdgcn_make_buffer_rsrc((void*)(p.w + pn * 512), 0, GK * GN * 2 - pn * 512, 0x00020000);
    const int aoff = lds_byte(wr * 64 + fr, fq * 8);
    int boff[2];
    { const int q = (lane >> 2) & 3, pp = lane & 3, g = fq, f = q | ((g & 1) << 2);
#pragma unroll
      for (int n = 0; n < 2; ++n) boff[n] = (int)(unsigned)(uintptr_t)lds + 4 * HTB + (8 * g + q) * 256 + (((wc * 2 + n) ^ f) * 32) + 8 * pp; }

#define SA(b, h) (((b) * 2 + (h)) * HTB)
#define SB(b, h) ((4 + (b) * 2 + (h)) * HTB)
#define LDA(dst, b, h) do { _Pragma("unroll") for (int m = 0; m < 4; ++m) _Pragma("unroll") for (int k = 0; k < 2; ++k) dst[m][k] = *(const LAS bf16x8*)(lds + SA(b, h) + aoff + m * 2048 + k * 1024); } while (0)
#define TRRD(dst, addr, off) asm volatile("ds_read_b64_tr_b16 %0, %1 offset:%2" : "=v"(dst) : "v"(addr), "i"(off) : "memory")
#define LDB(dst, b, h) do { _Pragma("unroll") for (int n = 0; n < 2; ++n) _Pragma("unroll") for (int k = 0; k < 2; ++k) { \
        TRRD(dst[n][k][0], boff[n], (SB(b, h) - 4 * HTB) + k * 8192); TRRD(dst[n][k][1], boff[n], (SB(b, h) - 4 * HTB) + k * 8192 + 1024); } } while (0)
#define WAIT_LB(Bt) asm volatile("s_waitcnt lgkmcnt(0)" : "+v"(Bt[0][0][0]), "+v"(Bt[0][0][1]), "+v"(Bt[0][1][0]), "+v"(Bt[0][1][1]), "+v"(Bt[1][0][0]), "+v"(Bt[1][0][1]), "+v"(Bt[1][1][0]), "+v"(Bt[1][1][1]) :: "memory")
#define BFR(Bt, n, k) __builtin_shufflevector(Bt[n][k][0], Bt[n][k][1], 0, 1, 2, 3, 4, 5, 6, 7)
#define MMA(ai, bj, At, Bt) do { __builtin_amdgcn_s_setprio(1); _Pragma("unroll") for (int m = 0; m < 4; ++m) _Pragma("unroll") for (int n = 0; n < 2; ++n) _Pragma("unroll") for (int k = 0; k < 2; ++k) \
        acc[ai][bj][m][n] = __builtin_amdgcn_mfma_f32_16x16x32_bf16(BFR(Bt, n, k), At[m][k], acc[ai][bj][m][n], 0, 0, 0); __builtin_amdgcn_s_setprio(0); } while (0)

#define STAMP(i)
#define PST(ph)
    bool bad = false;

    f32x4 acc[2][2][4][2];
#pragma unroll
    for (int a = 0; a < 2; ++a)
#pragma unroll
        for (int b = 0; b < 2; ++b)
#pragma unroll
            for (int m = 0; m < 4; ++m)
#pragma unroll
                for (int n = 0; n < 2; ++n) acc[a][b][m][n] = (f32x4){0.f, 0.f, 0.f, 0.f};
    bf16x8 At[4][2]; s16x4 B0[2][2][2], B1[2][2][2];

    STAGE(SB(0, 0), rsB, 0, voffB, voffBx); STAGE(SB(0, 1), rsB, hstepB, voffB, voffBx);
    STAGE(SB(1, 0), rsB, kstepB, voffB, voffBx); STAGE(SB(1, 1), rsB, kstepB + hstepB, voffB, voffBx);
    const unsigned ctl = (unsigned)(uintptr_t)lds + LDS_CTL;
#define CTL_WR(v) asm volatile("ds_write_b32 %0, %1" :: "v"(ctl), "v"((unsigned)(v)) : "memory")
#define CTL_RD(r) asm volatile("ds_read_b32 %0, %1" : "=v"(r) : "v"(ctl) : "memory")
#define CTL_GET(r) ({ asm volatile("s_waitcnt lgkmcnt(0)" : "+v"(r) :: "memory"); (int)__builtin_amdgcn_readfirstlane((int)(r)); })
    if (wid == 2) { CTL_WR(0); WAIT_L(0); }
    BAR;
    const unsigned seq = (unsigned)__builtin_amdgcn_readfirstlane((int)seqv) / NWG + 1u;
    const unsigned par = seq & 1u;
    const __amdgpu_buffer_rsrc_t rsAr = __builtin_amdgcn_make_buffer_rsrc(local ? (void*)(p.x + (size_t)(pm & 1) * 256 * (GK * 2)) : (void*)(wsl + WS_GATHER + par * GATHER_BYTES + (size_t)pm * (NT * 32768)), 0, NT * 32768, 0x00020000);
    const unsigned* flagp = (const unsigned*)(wsl + WS_FLAGS) + pm * (NT * 32) + lane;
    unsigned* tmo = (unsigned*)(wsl + WS_TMO);
    STAMP(33);
    const unsigned ui0 = (unsigned)((((rank * 2 + ((unit >> 5) & 1)) * NT + (unit >> 6)) << 5) + (unit & 31));
    const unsigned goff = (unsigned)(WS_GATHER + par * GATHER_BYTES);
    const int pj0 = unit % 7;
    int st = (wid == 3 && ev < NEV) ? 1 : 0;
    bool flp = false;
    unsigned long long nowr = t_start;
    auto pump = [&]() __attribute__((always_inline)) {
        if (st != 0) {
            if (st < 3) {
                if (st == 1) {
                    unsigned r; asm volatile("s_getreg_b32 %0, hwreg(HW_REG_IB_STS)" : "=s"(r) :: "memory");
                    if ((r & 0xC0000Fu) == 0u) {
                        asm volatile("v_mov_b32 %0, %4\n\tv_mov_b32 %1, %5\n\tv_mov_b32 %2, %6\n\tv_mov_b32 %3, %7" : "=&v"(d0.x), "=&v"(d0.y), "=&v"(d0.z), "=&v"(d0.w) : "v"(dn.x), "v"(dn.y), "v"(dn.z), "v"(dn.w));
                        if (flp) {
                            const unsigned long long fa = wsv + WS_FLAGS + (unsigned long long)(ui0 + 128u * (unsigned)(ev - 1)) * 4u;
                            if (lane < 8 && lane != rank && !(p.dbg & 2)) asm volatile("global_store_dword %0, %1, off sc0 sc1" :: "v"(fa), "v"(seq) : "memory");
                        }
                        st = ev < NEV ? 2 : 0;
                    }
                }
                if (st == 2 && (unsigned)nowr - (unsigned)t_start >= (((unsigned)(unit + 256 * ev) * (unsigned)p.slot) >> 6)) st = 3;
                nowr = __builtin_amdgcn_s_memrealtime();
            }
            if (st >= 3) {
                int j = pj0 + st - 3; j = j >= 7 ? j - 7 : j;
                const int pr = (p.dbg & 1) ? rank : (rank + 1 + j) & 7;
                char* base = (char*)(uintptr_t)(((unsigned long long)(unsigned)__builtin_amdgcn_readlane((int)ws_hi, pr) << 32) | (unsigned)__builtin_amdgcn_readlane((int)ws_lo, pr));
                const __amdgpu_buffer_rsrc_t rs = __builtin_amdgcn_make_buffer_rsrc((void*)base, 0, (int)WS_NEED, 0x00020000);
                __builtin_amdgcn_raw_buffer_store_b128(d0, rs, (int)(lane * 16), (int)(goff + (ui0 + 128u * (unsigned)ev) * 1024u), 17);
                if (++st == 10) {
                    flp = true; ++ev; st = 1;
                    if (ev < NEV) dn = __builtin_amdgcn_raw_buffer_load_b128(rsX, (int)vsrc, (int)(so0 + 512u * (unsigned)ev), 0);
                }
            }
        }
    };
    auto slow_wait = [&](int kt) {
        const unsigned long long t0 = __builtin_amdgcn_s_memrealtime();
        for (;;) {
            const unsigned v = __hip_atomic_load(flagp + kt * 32, __ATOMIC_RELAXED, __HIP_MEMORY_SCOPE_SYSTEM);
            if (__all(v >= seq)) break;
            if ((unsigned)__builtin_amdgcn_readfirstlane(__hip_atomic_load(tmo, __ATOMIC_RELAXED, __HIP_MEMORY_SCOPE_AGENT)) == seq || __builtin_amdgcn_s_memrealtime() - t0 > DEADLINE_TICKS) {
                __hip_atomic_store(tmo, seq, __ATOMIC_RELAXED, __HIP_MEMORY_SCOPE_AGENT); bad = true; break; }
            __builtin_amdgcn_s_sleep(2);
        }
        __builtin_amdgcn_fence(__ATOMIC_ACQUIRE, "");
        CTL_WR(kt + 2);
    };
    unsigned rv = 0;
    auto pump_until = [&](int need) __attribute__((always_inline)) {
        if (CTL_GET(rv) < need) {
            const unsigned long long t0 = __builtin_amdgcn_s_memrealtime();
            do { pump(); CTL_RD(rv);
                if (__builtin_amdgcn_s_memrealtime() - t0 > 2 * DEADLINE_TICKS) { __hip_atomic_store(tmo, seq, __ATOMIC_RELAXED, __HIP_MEMORY_SCOPE_AGENT); break; }
            } while (CTL_GET(rv) < need);
        }
    };
    const bool poller = (wid == 2) && !nocomm;
    if (poller) slow_wait(0);
    if (wid == 3) {
        pump(); pump();
        if (!nocomm) { CTL_RD(rv); pump_until(2); }
    }
    STAMP(34);
    BAR;
    STAGE(SA(0, 0), rsAr, 0, voffA, voffAx); STAGE(SA(0, 1), rsAr, hstepA, voffA, voffAx);
    if (wr == 1) BAR;
    STAGE(SA(1, 0), rsAr, kstepA, voffA, voffAx);
    if (wid != 3) WAIT_V(0); else { pump(); CTL_RD(rv); }
    BAR;
    BAR;
    unsigned fv = 0; bool ok = false;
    auto main_loop = [&](auto role_c) __attribute__((always_inline)) {
    constexpr int ROLE = decltype(role_c)::value;
#define RSTAGE(bufoff, rs_, soff, voff, voffx) do { \
        if constexpr (ROLE != 2) { BLDS(rs_, lds + (bufoff) + ldsw, (voff)[0], soff); BLDS(rs_, lds + (bufoff) + ldsw + 8192, (voff)[1], soff); } \
        if constexpr (ROLE == 0) BLDS(rs_, lds + (bufoff) + 3072 + wid * 8192, voffx, soff); } while (0)
#define RWAIT_STAGE() do { if constexpr (ROLE == 0) WAIT_V(15); else if constexpr (ROLE == 1) WAIT_V(10); } while (0)
#define PUMP() do { if constexpr (ROLE == 2) pump(); } while (0)
    for (int t = 0; t < NT; t += 2) {
        const bool last = (t == NT - 2);
        const unsigned a1 = (unsigned)(t + 1) * kstepA;
        const unsigned a2 = last ? 0u : (unsigned)(t + 2) * kstepA, b2 = last ? 0u : (unsigned)(t + 2) * kstepB;
        const unsigned a3 = a2 + kstepA, b3 = b2 + kstepB;
        if constexpr (ROLE == 1) {
        if (poller && !last) { if (!ok) slow_wait(t + 2); }
        }
        if constexpr (ROLE == 2) { if (!nocomm && !last) pump_until(t + 4); }
        STAMP(t >> 1);
        PST(0); PUMP(); LDB(B0, 0, 0); SCHED; LDA(At, 0, 0); RSTAGE(SA(1, 1), rsAr, a1 + hstepA, voffA, voffAx);
        RWAIT_STAGE(); WAIT_L(8); BAR; WAIT_LB(B0); MMA(0, 0, At, B0); BAR; SCHED;
        PST(1); PUMP(); LDB(B1, 0, 1); RSTAGE(SB(0, 0), rsB, b2, voffB, voffBx);
        RWAIT_STAGE();
        if constexpr (ROLE == 1) { if (poller) { const unsigned* fa = flagp + ((t + 4 < NT) ? (t + 4) : 0) * 32;
            asm volatile("global_load_dword %0, %1, off sc0 sc1" : "=v"(fv) : "v"(fa) : "memory"); } }
        BAR; WAIT_LB(B1); MMA(0, 1, At, B1); BAR;
        PST(2); PUMP(); LDA(At, 0, 1); RSTAGE(SA(0, 0), rsAr, a2, voffA, voffAx);
        BAR; WAIT_L(0); MMA(1, 0, At, B0); BAR; SCHED;
        PST(3); PUMP(); RSTAGE(SB(0, 1), rsB, b2 + hstepB, voffB, voffBx);
        RWAIT_STAGE();
        BAR; MMA(1, 1, At, B1); BAR;
        PST(4); PUMP(); LDB(B0, 1, 0); SCHED; LDA(At, 1, 0); RSTAGE(SA(0, 1), rsAr, a2 + hstepA, voffA, voffAx);
        RWAIT_STAGE(); WAIT_L(8); BAR; WAIT_LB(B0); MMA(0, 0, At, B0); BAR; SCHED;
        PST(5); PUMP(); LDB(B1, 1, 1); RSTAGE(SB(1, 0), rsB, b3, voffB, voffBx);
        RWAIT_STAGE();
        BAR; WAIT_LB(B1); MMA(0, 1, At, B1); BAR;
        PST(6); PUMP(); LDA(At, 1, 1); RSTAGE(SA(1, 0), rsAr, a3, voffA, voffAx);
        BAR; WAIT_L(0); MMA(1, 0, At, B0); BAR; SCHED;
        PST(7); PUMP(); RSTAGE(SB(1, 1), rsB, b3 + hstepB, voffB, voffBx);
        RWAIT_STAGE();
        if constexpr (ROLE == 1) { asm volatile("" : "+v"(fv)); if (poller) { ok = __all(fv >= seq); if (ok) { CTL_WR(t + 6); WAIT_L(0); } } }
        BAR;
        if constexpr (ROLE == 2) CTL_RD(rv);
        MMA(1, 1, At, B1); BAR;
        PST(8);
    }
    };
    if (wid < 2) main_loop(std::integral_constant<int, 0>{});
    else if (wid != 3) main_loop(std::integral_constant<int, 1>{});
    else main_loop(std::integral_constant<int, 2>{});
    if (wid == 3) { const unsigned long long t0 = __builtin_amdgcn_s_memrealtime(); while (st != 0 && __builtin_amdgcn_s_memrealtime() - t0 < DEADLINE_TICKS) pump(); }
    WAIT_V(0);
    if (wr == 0) BAR;
    STAMP(32);

    if (p.mode == 0) bad = ((unsigned)__builtin_amdgcn_readfirstlane(__hip_atomic_load(tmo, __ATOMIC_RELAXED, __HIP_MEMORY_SCOPE_AGENT)) == seq);
    {
        const int colb = pn * 256 + wc * 32 + ((fq & 1) * 16 + (fq >> 1) * 8);
        char* orow = p.out + ((size_t)(pm * 256 + wr * 64 + fr) * GN + colb) * 2;
#pragma unroll
        for (int ai = 0; ai < 2; ++ai)
#pragma unroll
            for (int m = 0; m < 4; ++m)
#pragma unroll
                for (int bj = 0; bj < 2; ++bj) {
                    const f32x4 v0 = acc[ai][bj][m][0], v1 = acc[ai][bj][m][1];
                    unsigned a0 = cvt_pk_bf16(silu1(v0[0]), silu1(v0[1])), a1 = cvt_pk_bf16(silu1(v0[2]), silu1(v0[3]));
                    unsigned b0 = cvt_pk_bf16(silu1(v1[0]), silu1(v1[1])), b1 = cvt_pk_bf16(silu1(v1[2]), silu1(v1[3]));
                    const auto s0 = __builtin_amdgcn_permlane16_swap(a0, b0, false, false);
                    const auto s1 = __builtin_amdgcn_permlane16_swap(a1, b1, false, false);
                    u32x4 o; o.x = s0[0]; o.y = s1[0]; o.z = s0[1]; o.w = s1[1];
                    if (bad) o = (u32x4){0x7fc07fc0u, 0x7fc07fc0u, 0x7fc07fc0u, 0x7fc07fc0u};
                    *(u32x4*)(orow + ((size_t)(ai * 128 + m * 16) * GN + bj * 128) * 2) = o;
                }
    }
    if (tid == 0) __hip_atomic_fetch_add((unsigned*)(wsl + WS_CTR), 1u, __ATOMIC_RELAXED, __HIP_MEMORY_SCOPE_AGENT);
}

static int g_mode = 0, g_slot = 136, g_pslot = 100, g_dbg = 0, g_pit = 10, g_late = 0;

extern "C" void kernel_launch(int rank, int nranks, ncclComm_t comm,
                              hipStream_t stream,
                              void* const* d_in, const int* in_sizes, int n_in,
                              void* d_out, int out_size,
                              const int* in_dtypes, int out_dtype,
                              void* const* d_out_all,
                              void* const* d_ws_all, size_t ws_bytes) {
    if (nranks != NR || ws_bytes < WS_NEED || (unsigned)rank >= (unsigned)NR) { fprintf(stderr, "unsupported configuration\n"); return; }
    static bool init = false;
    if (!init) {
        HIP_CHECK(hipFuncSetAttribute((const void*)ag_gemm_silu, hipFuncAttributeMaxDynamicSharedMemorySize, LDS_BYTES));
        init = true;
    }
    Params p;
    p.x = (const char*)d_in[0]; p.w = (const char*)d_in[1]; p.out = (char*)d_out;
    for (int i = 0; i < NR; ++i) p.ws[i] = (char*)d_ws_all[i];
    p.rank = rank; p.mode = g_mode; p.slot = g_slot; p.pslot = g_pslot; p.pit = g_pit; p.dbg = g_dbg;
    hipLaunchKernelGGL(ag_gemm_silu, dim3(NWG), dim3(512), LDS_BYTES, stream, p);
    HIP_CHECK(hipGetLastError());
}
```

```cpp
#include <hip/hip_runtime.h>
#include <rccl/rccl.h>
#include <cstdio>
#include <cstdint>
#include <type_traits>

#define HIP_CHECK(expr) do { hipError_t _e = (expr); if (_e != hipSuccess) { \
    fprintf(stderr, "HIP error %s at %s:%d\n", hipGetErrorString(_e), __FILE__, __LINE__); \
    return; } } while (0)

#define LAS __attribute__((address_space(3)))
#define GAS __attribute__((address_space(1)))
typedef short bf16x8 __attribute__((ext_vector_type(8)));
typedef short s16x4 __attribute__((ext_vector_type(4)));
typedef float f32x4 __attribute__((ext_vector_type(4)));
typedef unsigned u32x4 __attribute__((ext_vector_type(4)));
typedef unsigned u32x2 __attribute__((ext_vector_type(2)));

constexpr int NR = 8;
constexpr int GM = 4096, GK = 4096, GN = 4096;
constexpr int BK = 64, NT = GK / BK;
constexpr int HTB = 128 * BK * 2;
constexpr int LDS_CTL = 8 * HTB;
constexpr int LDS_BYTES = 8 * HTB + 1024;
constexpr int NWG = 256;

constexpr size_t WS_CTR = 0;
constexpr size_t WS_TMO = 256;
constexpr size_t WS_FLAGS = 4096;
constexpr size_t WS_STAMPS = 512 * 1024;
constexpr int NEV = 16;
constexpr size_t WS_GATHER = (size_t)1 << 20;
constexpr size_t GATHER_BYTES = (size_t)GM * GK * 2;
constexpr size_t WS_NEED = WS_GATHER + 2 * GATHER_BYTES;
constexpr unsigned long long DEADLINE_TICKS = 20000000ull;

struct Params {
    const char* x;
    const char* w;
    char* out;
    char* ws[NR];
    int rank;
    int mode;
    int dbg;
    int pit;
    int pslot;
    int slot;
};

__device__ __forceinline__ int lds_byte(int r, int c) { const int st = (r >> 4) * 2 + (c >> 5), rr = r & 15, cc = c & 31, ob = rr * 64 + cc * 2; return st * 1024 + (ob ^ (((ob >> 9) & 1) << 5)); }
__device__ __forceinline__ void stage_rc(int b, int& R, int& C) { const int st = b / 1024, sb = b % 1024, swz = sb ^ (((sb >> 9) & 1) << 5); R = (st >> 1) * 16 + swz / 64; C = (st & 1) * 32 + (swz % 64) / 2; }

__device__ __forceinline__ unsigned cvt_pk_bf16(float lo, float hi) { unsigned r; asm volatile("v_cvt_pk_bf16_f32 %0, %1, %2" : "=v"(r) : "v"(lo), "v"(hi)); return r; }
typedef float f32x2 __attribute__((ext_vector_type(2)));
__device__ __forceinline__ unsigned silu2(float y0, float y1) {
    const f32x2 y = {y0, y1};
    const f32x2 a = y * -1.4426950408889634f;
    const f32x2 d = (f32x2){__builtin_amdgcn_exp2f(__builtin_amdgcn_fmed3f(a.x, -86.5617f, 86.5617f)), __builtin_amdgcn_exp2f(__builtin_amdgcn_fmed3f(a.y, -86.5617f, 86.5617f))} + 1.0f;
    const f32x2 r = y * (f32x2){__builtin_amdgcn_rcpf(d.x), __builtin_amdgcn_rcpf(d.y)};
    return cvt_pk_bf16(r.x, r.y);
}

__global__ __launch_bounds__(512, 2) void ag_gemm_silu(const Params p) {
    extern __shared__ __attribute__((aligned(16))) unsigned char smem_raw[];
    LAS unsigned char* lds = (LAS unsigned char*)smem_raw;
    const int tid = threadIdx.x, wid = __builtin_amdgcn_readfirstlane(tid >> 6), lane = tid & 63, wr = wid >> 2, wc = wid & 3, fr = lane & 15, fq = lane >> 4;
    const unsigned long long t_start = __builtin_amdgcn_s_memrealtime();
    const int bid = blockIdx.x, xcd = bid & 7, idx = bid >> 3;
    const int pm = (xcd >> 1) * 4 + (idx & 3), pn = (xcd & 1) * 8 + (idx >> 2);
    const int rank = p.rank;
    const bool local = (pm >> 1) == rank;
    const bool nocomm = local || p.mode >= 1;
    const int unit = (((pm - 2 * rank - 2) & 15) << 4) + pn;
    char* const wsl = p.ws[rank];

#define WAIT_V(n) asm volatile("s_waitcnt vmcnt(" #n ")" ::: "memory")
#define WAIT_L(n) asm volatile("s_waitcnt lgkmcnt(" #n ")" ::: "memory")
#define BAR __builtin_amdgcn_s_barrier()
#define SCHED __builtin_amdgcn_sched_barrier(0)
#define BLDS(rs, l, voff, soff) __builtin_amdgcn_raw_ptr_buffer_load_lds(rs, (LAS void*)(l), 16, (int)(voff), (int)(soff), 0, 0)
#define STAGE(bufoff, rs, soff, voff, voffx) do { \
        if (wid != 3) { BLDS(rs, lds + (bufoff) + ldsw, (voff)[0], soff); BLDS(rs, lds + (bufoff) + ldsw + 8192, (voff)[1], soff); } \
        if (wid < 2) BLDS(rs, lds + (bufoff) + 3072 + wid * 8192, voffx, soff); } while (0)
#define WAIT_STAGE() do { if (wid < 2) WAIT_V(9); else if (wid != 3) WAIT_V(6); } while (0)
    const unsigned ldsw = (unsigned)wid * 1024u;
    const __amdgpu_buffer_rsrc_t rsX = __builtin_amdgcn_make_buffer_rsrc((void*)p.x, 0, 512 * GK * 2, 0x00020000);
    const unsigned vsrc = (unsigned)((lane >> 3) * (GK * 2) + (lane & 7) * 16);
    const bool nosend = (p.mode == 1 || ((p.dbg & 4) && rank != 0) || ((p.dbg & 8) && rank == 0));
    int ev = nosend ? NEV : 0;
    const unsigned so0 = (unsigned)((unit & 63) * 8 * (GK * 2) + (unit >> 6) * 128);
    u32x4 d0 = {0u, 0u, 0u, 0u}, dn = {0u, 0u, 0u, 0u};
    if (wid == 3 && ev < NEV) dn = __builtin_amdgcn_raw_buffer_load_b128(rsX, (int)vsrc, (int)so0, 0);
    const unsigned seqv = __hip_atomic_load((const unsigned*)(wsl + WS_CTR), __ATOMIC_RELAXED, __HIP_MEMORY_SCOPE_AGENT);
    unsigned long long wsv = (unsigned long long)(uintptr_t)p.ws[0];
#pragma unroll
    for (int i = 1; i < NR; ++i) { unsigned long long w = (unsigned long long)(uintptr_t)p.ws[i]; asm volatile("" : "+s"(w)); wsv = ((lane & 7) == i) ? w : wsv; }
    const unsigned ws_lo = (unsigned)wsv, ws_hi = (unsigned)(wsv >> 32);

    const unsigned rsA = local ? (unsigned)(GK * 2) : 128u;
    unsigned voffA[2], voffB[2], voffAx, voffBx;
#pragma unroll
    for (int i = 0; i < 3; ++i) {
        const int tt = i < 2 ? tid : 192 + lane, ii = i < 2 ? i : (wid & 1);
        int R, C; stage_rc(tt * 16 + ii * 8192, R, C);
        const unsigned va = (unsigned)R * rsA + (unsigned)C * 2u;
        const int k = (tt >> 4) + 32 * ii, c = tt & 15, f = (k & 3) | (((k >> 3) & 1) << 2);
        const unsigned vb = (unsigned)(k * (GN * 2) + ((c ^ (f << 1)) * 16));
        if (i < 2) { voffA[i] = va; voffB[i] = vb; } else { voffAx = va; voffBx = vb; }
    }
    const unsigned kstepA = local ? 128u : 32768u, hstepA = 128u * rsA;
    const unsigned kstepB = (unsigned)(BK * GN * 2), hstepB = 256u;
    const __amdgpu_buffer_rsrc_t rsB = __builtin_amdgcn_make_buffer_rsrc((void*)(p.w + pn * 512), 0, GK * GN * 2 - pn * 512, 0x00020000);
    const int aoff = lds_byte(wr * 64 + fr, fq * 8);
    int boff[2];
    { const int q = (lane >> 2) & 3, pp = lane & 3, g = fq, f = q | ((g & 1) << 2);
#pragma unroll
      for (int n = 0; n < 2; ++n) boff[n] = (int)(unsigned)(uintptr_t)lds + 4 * HTB + (8 * g + q) * 256 + (((wc * 2 + n) ^ f) * 32) + 8 * pp; }

#define SA(b, h) (((b) * 2 + (h)) * HTB)
#define SB(b, h) ((4 + (b) * 2 + (h)) * HTB)
#define LDA(dst, b, h) do { _Pragma("unroll") for (int m = 0; m < 4; ++m) _Pragma("unroll") for (int k = 0; k < 2; ++k) dst[m][k] = *(const LAS bf16x8*)(lds + SA(b, h) + aoff + m * 2048 + k * 1024); } while (0)
#define TRRD(dst, addr, off) asm volatile("ds_read_b64_tr_b16 %0, %1 offset:%2" : "=v"(dst) : "v"(addr), "i"(off) : "memory")
#define LDB(dst, b, h) do { _Pragma("unroll") for (int n = 0; n < 2; ++n) _Pragma("unroll") for (int k = 0; k < 2; ++k) { \
        TRRD(dst[n][k][0], boff[n], (SB(b, h) - 4 * HTB) + k * 8192); TRRD(dst[n][k][1], boff[n], (SB(b, h) - 4 * HTB) + k * 8192 + 1024); } } while (0)
#define WAIT_LB(Bt) asm volatile("s_waitcnt lgkmcnt(0)" : "+v"(Bt[0][0][0]), "+v"(Bt[0][0][1]), "+v"(Bt[0][1][0]), "+v"(Bt[0][1][1]), "+v"(Bt[1][0][0]), "+v"(Bt[1][0][1]), "+v"(Bt[1][1][0]), "+v"(Bt[1][1][1]) :: "memory")
#define BFR(Bt, n, k) __builtin_shufflevector(Bt[n][k][0], Bt[n][k][1], 0, 1, 2, 3, 4, 5, 6, 7)
#define MMA(ai, bj, At, Bt) do { __builtin_amdgcn_s_setprio(1); _Pragma("unroll") for (int m = 0; m < 4; ++m) _Pragma("unroll") for (int n = 0; n < 2; ++n) _Pragma("unroll") for (int k = 0; k < 2; ++k) \
        acc[ai][bj][m][n] = __builtin_amdgcn_mfma_f32_16x16x32_bf16(BFR(Bt, n, k), At[m][k], acc[ai][bj][m][n], 0, 0, 0); __builtin_amdgcn_s_setprio(0); } while (0)

#define STAMP(i)
#define PST(ph)
    bool bad = false;

    f32x4 acc[2][2][4][2];
#pragma unroll
    for (int a = 0; a < 2; ++a)
#pragma unroll
        for (int b = 0; b < 2; ++b)
#pragma unroll
            for (int m = 0; m < 4; ++m)
#pragma unroll
                for (int n = 0; n < 2; ++n) acc[a][b][m][n] = (f32x4){0.f, 0.f, 0.f, 0.f};
    bf16x8 At[4][2]; s16x4 B0[2][2][2], B1[2][2][2];

    STAGE(SB(0, 0), rsB, 0, voffB, voffBx); STAGE(SB(0, 1), rsB, hstepB, voffB, voffBx);
    STAGE(SB(1, 0), rsB, kstepB, voffB, voffBx); STAGE(SB(1, 1), rsB, kstepB + hstepB, voffB, voffBx);
    const unsigned ctl = (unsigned)(uintptr_t)lds + LDS_CTL;
#define CTL_WR(v) asm volatile("ds_write_b32 %0, %1" :: "v"(ctl), "v"((unsigned)(v)) : "memory")
#define CTL_RD(r) asm volatile("ds_read_b32 %0, %1" : "=v"(r) : "v"(ctl) : "memory")
#define CTL_GET(r) ({ asm volatile("s_waitcnt lgkmcnt(0)" : "+v"(r) :: "memory"); (int)__builtin_amdgcn_readfirstlane((int)(r)); })
    if (wid == 2) { CTL_WR(0); WAIT_L(0); }
    BAR;
    const unsigned seq = (unsigned)__builtin_amdgcn_readfirstlane((int)seqv) / NWG + 1u;
    const unsigned par = seq & 1u;
    const __amdgpu_buffer_rsrc_t rsAr = __builtin_amdgcn_make_buffer_rsrc(local ? (void*)(p.x + (size_t)(pm & 1) * 256 * (GK * 2)) : (void*)(wsl + WS_GATHER + par * GATHER_BYTES + (size_t)pm * (NT * 32768)), 0, NT * 32768, 0x00020000);
    const unsigned* flagp = (const unsigned*)(wsl + WS_FLAGS) + pm * (NT * 32) + lane;
    unsigned* tmo = (unsigned*)(wsl + WS_TMO);
    STAMP(33);
    const unsigned ui0 = (unsigned)((((rank * 2 + ((unit >> 5) & 1)) * NT + (unit >> 6)) << 5) + (unit & 31));
    const unsigned goff = (unsigned)(WS_GATHER + par * GATHER_BYTES);
    const int pj0 = unit % 7;
    int st = (wid == 3 && ev < NEV) ? 1 : 0;
    bool flp = false;
    unsigned long long nowr = t_start;
    auto pump = [&]() __attribute__((always_inline)) {
        if (st != 0) {
            if (st < 3) {
                if (st == 1) {
                    unsigned r; asm volatile("s_getreg_b32 %0, hwreg(HW_REG_IB_STS)" : "=s"(r) :: "memory");
                    if ((r & 0xC0000Fu) == 0u) {
                        asm volatile("v_mov_b32 %0, %4\n\tv_mov_b32 %1, %5\n\tv_mov_b32 %2, %6\n\tv_mov_b32 %3, %7" : "=&v"(d0.x), "=&v"(d0.y), "=&v"(d0.z), "=&v"(d0.w) : "v"(dn.x), "v"(dn.y), "v"(dn.z), "v"(dn.w));
                        if (flp) {
                            const unsigned long long fa = wsv + WS_FLAGS + (unsigned long long)(ui0 + 128u * (unsigned)(ev - 1)) * 4u;
                            if (lane < 8 && lane != rank && !(p.dbg & 2)) asm volatile("global_store_dword %0, %1, off sc0 sc1" :: "v"(fa), "v"(seq) : "memory");
                        }
                        st = ev < NEV ? 2 : 0;
                    }
                }
                if (st == 2 && (unsigned)nowr - (unsigned)t_start >= (((unsigned)(unit + 256 * ev) * (unsigned)p.slot) >> 6)) st = 3;
                nowr = __builtin_amdgcn_s_memrealtime();
            }
            if (st >= 3) {
                int j = pj0 + st - 3; j = j >= 7 ? j - 7 : j;
                const int pr = (p.dbg & 1) ? rank : (rank + 1 + j) & 7;
                char* base = (char*)(uintptr_t)(((unsigned long long)(unsigned)__builtin_amdgcn_readlane((int)ws_hi, pr) << 32) | (unsigned)__builtin_amdgcn_readlane((int)ws_lo, pr));
                const __amdgpu_buffer_rsrc_t rs = __builtin_amdgcn_make_buffer_rsrc((void*)base, 0, (int)WS_NEED, 0x00020000);
                __builtin_amdgcn_raw_buffer_store_b128(d0, rs, (int)(lane * 16), (int)(goff + (ui0 + 128u * (unsigned)ev) * 1024u), 17);
                if (++st == 10) {
                    flp = true; ++ev; st = 1;
                    if (ev < NEV) dn = __builtin_amdgcn_raw_buffer_load_b128(rsX, (int)vsrc, (int)(so0 + 512u * (unsigned)ev), 0);
                }
            }
        }
    };
    auto slow_wait = [&](int kt) {
        const unsigned long long t0 = __builtin_amdgcn_s_memrealtime();
        for (;;) {
            const unsigned v = __hip_atomic_load(flagp + kt * 32, __ATOMIC_RELAXED, __HIP_MEMORY_SCOPE_SYSTEM);
            if (__all(v >= seq)) break;
            if ((unsigned)__builtin_amdgcn_readfirstlane(__hip_atomic_load(tmo, __ATOMIC_RELAXED, __HIP_MEMORY_SCOPE_AGENT)) == seq || __builtin_amdgcn_s_memrealtime() - t0 > DEADLINE_TICKS) {
                __hip_atomic_store(tmo, seq, __ATOMIC_RELAXED, __HIP_MEMORY_SCOPE_AGENT); bad = true; break; }
            __builtin_amdgcn_s_sleep(2);
        }
        __builtin_amdgcn_fence(__ATOMIC_ACQUIRE, "");
        CTL_WR(kt + 2);
    };
    unsigned rv = 0;
    auto pump_until = [&](int need) __attribute__((always_inline)) {
        if (CTL_GET(rv) < need) {
            const unsigned long long t0 = __builtin_amdgcn_s_memrealtime();
            do { pump(); CTL_RD(rv);
                if (__builtin_amdgcn_s_memrealtime() - t0 > 2 * DEADLINE_TICKS) { __hip_atomic_store(tmo, seq, __ATOMIC_RELAXED, __HIP_MEMORY_SCOPE_AGENT); break; }
            } while (CTL_GET(rv) < need);
        }
    };
    const bool poller = (wid == 2) && !nocomm;
    if (poller) slow_wait(0);
    if (wid == 3) {
        pump(); pump();
        if (!nocomm) { CTL_RD(rv); pump_until(2); }
    }
    STAMP(34);
    BAR;
    STAGE(SA(0, 0), rsAr, 0, voffA, voffAx); STAGE(SA(0, 1), rsAr, hstepA, voffA, voffAx);
    if (wr == 1) BAR;
    STAGE(SA(1, 0), rsAr, kstepA, voffA, voffAx);
    if (wid != 3) WAIT_V(0); else { pump(); CTL_RD(rv); }
    BAR;
    BAR;
    unsigned fv = 0; bool ok = false;
    auto main_loop = [&](auto role_c) __attribute__((always_inline)) {
    constexpr int ROLE = decltype(role_c)::value;
#define RSTAGE(bufoff, rs_, soff, voff, voffx) do { \
        if constexpr (ROLE != 2) { BLDS(rs_, lds + (bufoff) + ldsw, (voff)[0], soff); BLDS(rs_, lds + (bufoff) + ldsw + 8192, (voff)[1], soff); } \
        if constexpr (ROLE == 0) BLDS(rs_, lds + (bufoff) + 3072 + wid * 8192, voffx, soff); } while (0)
#define RWAIT_STAGE() do { if constexpr (ROLE == 0) WAIT_V(15); else if constexpr (ROLE == 1) WAIT_V(10); } while (0)
#define PUMP() do { if constexpr (ROLE == 2) pump(); } while (0)
    for (int t = 0; t < NT; t += 2) {
        const bool last = (t == NT - 2);
        const unsigned a1 = (unsigned)(t + 1) * kstepA;
        const unsigned a2 = last ? 0u : (unsigned)(t + 2) * kstepA, b2 = last ? 0u : (unsigned)(t + 2) * kstepB;
        const unsigned a3 = a2 + kstepA, b3 = b2 + kstepB;
        if constexpr (ROLE == 1) {
        if (poller && !last) { if (!ok) slow_wait(t + 2); }
        }
        if constexpr (ROLE == 2) { if (!nocomm && !last) pump_until(t + 4); }
        STAMP(t >> 1);
        PST(0); PUMP(); LDB(B0, 0, 0); SCHED; LDA(At, 0, 0); RSTAGE(SA(1, 1), rsAr, a1 + hstepA, voffA, voffAx);
        RWAIT_STAGE(); WAIT_L(8); BAR; WAIT_LB(B0); MMA(0, 0, At, B0); BAR; SCHED;
        PST(1); PUMP(); LDB(B1, 0, 1); RSTAGE(SB(0, 0), rsB, b2, voffB, voffBx);
        RWAIT_STAGE();
        if constexpr (ROLE == 1) { if (poller) { const unsigned* fa = flagp + ((t + 4 < NT) ? (t + 4) : 0) * 32;
            asm volatile("global_load_dword %0, %1, off sc0 sc1" : "=v"(fv) : "v"(fa) : "memory"); } }
        BAR; WAIT_LB(B1); MMA(0, 1, At, B1); BAR;
        PST(2); PUMP(); LDA(At, 0, 1); RSTAGE(SA(0, 0), rsAr, a2, voffA, voffAx);
        BAR; WAIT_L(0); MMA(1, 0, At, B0); BAR; SCHED;
        PST(3); PUMP(); RSTAGE(SB(0, 1), rsB, b2 + hstepB, voffB, voffBx);
        RWAIT_STAGE();
        BAR; MMA(1, 1, At, B1); BAR;
        PST(4); PUMP(); LDB(B0, 1, 0); SCHED; LDA(At, 1, 0); RSTAGE(SA(0, 1), rsAr, a2 + hstepA, voffA, voffAx);
        RWAIT_STAGE(); WAIT_L(8); BAR; WAIT_LB(B0); MMA(0, 0, At, B0); BAR; SCHED;
        PST(5); PUMP(); LDB(B1, 1, 1); RSTAGE(SB(1, 0), rsB, b3, voffB, voffBx);
        RWAIT_STAGE();
        BAR; WAIT_LB(B1); MMA(0, 1, At, B1); BAR;
        PST(6); PUMP(); LDA(At, 1, 1); RSTAGE(SA(1, 0), rsAr, a3, voffA, voffAx);
        BAR; WAIT_L(0); MMA(1, 0, At, B0); BAR; SCHED;
        PST(7); PUMP(); RSTAGE(SB(1, 1), rsB, b3 + hstepB, voffB, voffBx);
        RWAIT_STAGE();
        if constexpr (ROLE == 1) { asm volatile("" : "+v"(fv)); if (poller) { ok = __all(fv >= seq); if (ok) { CTL_WR(t + 6); WAIT_L(0); } } }
        BAR;
        if constexpr (ROLE == 2) CTL_RD(rv);
        MMA(1, 1, At, B1); BAR;
        PST(8);
    }
    };
    if (wid < 2) main_loop(std::integral_constant<int, 0>{});
    else if (wid != 3) main_loop(std::integral_constant<int, 1>{});
    else main_loop(std::integral_constant<int, 2>{});
    if (wid == 3) { const unsigned long long t0 = __builtin_amdgcn_s_memrealtime(); while (st != 0 && __builtin_amdgcn_s_memrealtime() - t0 < DEADLINE_TICKS) pump(); }
    WAIT_V(0);
    if (wr == 0) BAR;
    STAMP(32);

    if (p.mode == 0) bad = ((unsigned)__builtin_amdgcn_readfirstlane(__hip_atomic_load(tmo, __ATOMIC_RELAXED, __HIP_MEMORY_SCOPE_AGENT)) == seq);
    {
        const int colb = pn * 256 + wc * 32 + ((fq & 1) * 16 + (fq >> 1) * 8);
        char* orow = p.out + ((size_t)(pm * 256 + wr * 64 + fr) * GN + colb) * 2;
#pragma unroll
        for (int ai = 0; ai < 2; ++ai)
#pragma unroll
            for (int m = 0; m < 4; ++m)
#pragma unroll
                for (int bj = 0; bj < 2; ++bj) {
                    const f32x4 v0 = acc[ai][bj][m][0], v1 = acc[ai][bj][m][1];
                    unsigned a0 = silu2(v0[0], v0[1]), a1 = silu2(v0[2], v0[3]);
                    unsigned b0 = silu2(v1[0], v1[1]), b1 = silu2(v1[2], v1[3]);
                    const auto s0 = __builtin_amdgcn_permlane16_swap(a0, b0, false, false);
                    const auto s1 = __builtin_amdgcn_permlane16_swap(a1, b1, false, false);
                    u32x4 o; o.x = s0[0]; o.y = s1[0]; o.z = s0[1]; o.w = s1[1];
                    if (bad) o = (u32x4){0x7fc07fc0u, 0x7fc07fc0u, 0x7fc07fc0u, 0x7fc07fc0u};
                    *(u32x4*)(orow + ((size_t)(ai * 128 + m * 16) * GN + bj * 128) * 2) = o;
                }
    }
    if (tid == 0) __hip_atomic_fetch_add((unsigned*)(wsl + WS_CTR), 1u, __ATOMIC_RELAXED, __HIP_MEMORY_SCOPE_AGENT);
}

static int g_mode = 0, g_slot = 136, g_pslot = 100, g_dbg = 0, g_pit = 10, g_late = 0;

extern "C" void kernel_launch(int rank, int nranks, ncclComm_t comm,
                              hipStream_t stream,
                              void* const* d_in, const int* in_sizes, int n_in,
                              void* d_out, int out_size,
                              const int* in_dtypes, int out_dtype,
                              void* const* d_out_all,
                              void* const* d_ws_all, size_t ws_bytes) {
    if (nranks != NR || ws_bytes < WS_NEED || (unsigned)rank >= (unsigned)NR) { fprintf(stderr, "unsupported configuration\n"); return; }
    static bool init = false;
    if (!init) {
        HIP_CHECK(hipFuncSetAttribute((const void*)ag_gemm_silu, hipFuncAttributeMaxDynamicSharedMemorySize, LDS_BYTES));
        init = true;
    }
    Params p;
    p.x = (const char*)d_in[0]; p.w = (const char*)d_in[1]; p.out = (char*)d_out;
    for (int i = 0; i < NR; ++i) p.ws[i] = (char*)d_ws_all[i];
    p.rank = rank; p.mode = g_mode; p.slot = g_slot; p.pslot = g_pslot; p.pit = g_pit; p.dbg = g_dbg;
    hipLaunchKernelGGL(ag_gemm_silu, dim3(NWG), dim3(512), LDS_BYTES, stream, p);
    HIP_CHECK(hipGetLastError());
}
```

```cpp
#include <hip/hip_runtime.h>
#include <rccl/rccl.h>
#include <cstdio>
#include <cstdint>
#include <type_traits>

#define HIP_CHECK(expr) do { hipError_t _e = (expr); if (_e != hipSuccess) { \
    fprintf(stderr, "HIP error %s at %s:%d\n", hipGetErrorString(_e), __FILE__, __LINE__); \
    return; } } while (0)

#define LAS __attribute__((address_space(3)))
#define GAS __attribute__((address_space(1)))
typedef short bf16x8 __attribute__((ext_vector_type(8)));
typedef short s16x4 __attribute__((ext_vector_type(4)));
typedef float f32x4 __attribute__((ext_vector_type(4)));
typedef unsigned u32x4 __attribute__((ext_vector_type(4)));
typedef unsigned u32x2 __attribute__((ext_vector_type(2)));

constexpr int NR = 8;
constexpr int GM = 4096, GK = 4096, GN = 4096;
constexpr int BK = 64, NT = GK / BK;
constexpr int HTB = 128 * BK * 2;
constexpr int LDS_CTL = 8 * HTB;
constexpr int LDS_BYTES = 8 * HTB + 1024;
constexpr int NWG = 256;

constexpr size_t WS_CTR = 0;
constexpr size_t WS_TMO = 256;
constexpr size_t WS_FLAGS = 4096;
constexpr size_t WS_STAMPS = 512 * 1024;
constexpr int NEV = 16;
constexpr size_t WS_GATHER = (size_t)1 << 20;
constexpr size_t GATHER_BYTES = (size_t)GM * GK * 2;
constexpr size_t WS_NEED = WS_GATHER + 2 * GATHER_BYTES;
constexpr unsigned long long DEADLINE_TICKS = 20000000ull;

struct Params {
    const char* x;
    const char* w;
    char* out;
    char* ws[NR];
    int rank;
    int mode;
    int dbg;
    int pit;
    int pslot;
    int slot;
};

__device__ __forceinline__ int lds_byte(int r, int c) { const int st = (r >> 4) * 2 + (c >> 5), rr = r & 15, cc = c & 31, ob = rr * 64 + cc * 2; return st * 1024 + (ob ^ (((ob >> 9) & 1) << 5)); }
__device__ __forceinline__ void stage_rc(int b, int& R, int& C) { const int st = b / 1024, sb = b % 1024, swz = sb ^ (((sb >> 9) & 1) << 5); R = (st >> 1) * 16 + swz / 64; C = (st & 1) * 32 + (swz % 64) / 2; }

__device__ __forceinline__ unsigned cvt_pk_bf16(float lo, float hi) { unsigned r; asm volatile("v_cvt_pk_bf16_f32 %0, %1, %2" : "=v"(r) : "v"(lo), "v"(hi)); return r; }
typedef float f32x2 __attribute__((ext_vector_type(2)));
__device__ __forceinline__ unsigned silu2(float y0, float y1) {
    const f32x2 y = {y0, y1};
    const f32x2 a = y * -1.4426950408889634f;
    const f32x2 d = (f32x2){__builtin_amdgcn_exp2f(__builtin_amdgcn_fmed3f(a.x, -86.5617f, 86.5617f)), __builtin_amdgcn_exp2f(__builtin_amdgcn_fmed3f(a.y, -86.5617f, 86.5617f))} + 1.0f;
    const f32x2 r = y * (f32x2){__builtin_amdgcn_rcpf(d.x), __builtin_amdgcn_rcpf(d.y)};
    return cvt_pk_bf16(r.x, r.y);
}

__global__ __launch_bounds__(512, 2) void ag_gemm_silu(const Params p) {
    extern __shared__ __attribute__((aligned(16))) unsigned char smem_raw[];
    LAS unsigned char* lds = (LAS unsigned char*)smem_raw;
    const int tid = threadIdx.x, wid = __builtin_amdgcn_readfirstlane(tid >> 6), lane = tid & 63, wr = wid >> 2, wc = wid & 3, fr = lane & 15, fq = lane >> 4;
    const unsigned long long t_start = __builtin_amdgcn_s_memrealtime();
    const int bid = blockIdx.x, xcd = bid & 7, idx = bid >> 3;
    const int pm = (xcd >> 1) * 4 + (idx & 3), pn = (xcd & 1) * 8 + (idx >> 2);
    const int rank = p.rank;
    const bool local = (pm >> 1) == rank;
    const bool nocomm = local || p.mode >= 1;
    const int unit = (((pm - 2 * rank - 2) & 15) << 4) + pn;
    char* const wsl = p.ws[rank];

#define WAIT_V(n) asm volatile("s_waitcnt vmcnt(" #n ")" ::: "memory")
#define WAIT_L(n) asm volatile("s_waitcnt lgkmcnt(" #n ")" ::: "memory")
#define BAR __builtin_amdgcn_s_barrier()
#define SCHED __builtin_amdgcn_sched_barrier(0)
#define BLDS(rs, l, voff, soff) __builtin_amdgcn_raw_ptr_buffer_load_lds(rs, (LAS void*)(l), 16, (int)(voff), (int)(soff), 0, 0)
#define STAGE(bufoff, rs, soff, voff, voffx) do { \
        if (wid != 3) { BLDS(rs, lds + (bufoff) + ldsw, (voff)[0], soff); BLDS(rs, lds + (bufoff) + ldsw + 8192, (voff)[1], soff); } \
        if (wid < 2) BLDS(rs, lds + (bufoff) + 3072 + wid * 8192, voffx, soff); } while (0)
#define WAIT_STAGE() do { if (wid < 2) WAIT_V(9); else if (wid != 3) WAIT_V(6); } while (0)
    const unsigned ldsw = (unsigned)wid * 1024u;
    const __amdgpu_buffer_rsrc_t rsX = __builtin_amdgcn_make_buffer_rsrc((void*)p.x, 0, 512 * GK * 2, 0x00020000);
    const unsigned vsrc = (unsigned)((lane >> 3) * (GK * 2) + (lane & 7) * 16);
    const bool nosend = (p.mode == 1 || ((p.dbg & 4) && rank != 0) || ((p.dbg & 8) && rank == 0));
    int ev = nosend ? NEV : 0;
    const unsigned so0 = (unsigned)((unit & 63) * 8 * (GK * 2) + (unit >> 6) * 128);
    u32x4 d0 = {0u, 0u, 0u, 0u}, dn = {0u, 0u, 0u, 0u};
    if (wid == 3 && ev < NEV) dn = __builtin_amdgcn_raw_buffer_load_b128(rsX, (int)vsrc, (int)so0, 0);
    const unsigned seqv = __hip_atomic_load((const unsigned*)(wsl + WS_CTR), __ATOMIC_RELAXED, __HIP_MEMORY_SCOPE_AGENT);
    unsigned long long wsv = (unsigned long long)(uintptr_t)p.ws[0];
#pragma unroll
    for (int i = 1; i < NR; ++i) { unsigned long long w = (unsigned long long)(uintptr_t)p.ws[i]; asm volatile("" : "+s"(w)); wsv = ((lane & 7) == i) ? w : wsv; }
    const unsigned ws_lo = (unsigned)wsv, ws_hi = (unsigned)(wsv >> 32);

    const unsigned rsA = local ? (unsigned)(GK * 2) : 128u;
    unsigned voffA[2], voffB[2], voffAx, voffBx;
#pragma unroll
    for (int i = 0; i < 3; ++i) {
        const int tt = i < 2 ? tid : 192 + lane, ii = i < 2 ? i : (wid & 1);
        int R, C; stage_rc(tt * 16 + ii * 8192, R, C);
        const unsigned va = (unsigned)R * rsA + (unsigned)C * 2u;
        const int k = (tt >> 4) + 32 * ii, c = tt & 15, f = (k & 3) | (((k >> 3) & 1) << 2);
        const unsigned vb = (unsigned)(k * (GN * 2) + ((c ^ (f << 1)) * 16));
        if (i < 2) { voffA[i] = va; voffB[i] = vb; } else { voffAx = va; voffBx = vb; }
    }
    const unsigned kstepA = local ? 128u : 32768u, hstepA = 128u * rsA;
    const unsigned kstepB = (unsigned)(BK * GN * 2), hstepB = 256u;
    const __amdgpu_buffer_rsrc_t rsB = __builtin_amdgcn_make_buffer_rsrc((void*)(p.w + pn * 512), 0, GK * GN * 2 - pn * 512, 0x00020000);
    const int aoff = lds_byte(wr * 64 + fr, fq * 8);
    int boff[2];
    { const int q = (lane >> 2) & 3, pp = lane & 3, g = fq, f = q | ((g & 1) << 2);
#pragma unroll
      for (int n = 0; n < 2; ++n) boff[n] = (int)(unsigned)(uintptr_t)lds + 4 * HTB + (8 * g + q) * 256 + (((wc * 2 + n) ^ f) * 32) + 8 * pp; }

#define SA(b, h) (((b) * 2 + (h)) * HTB)
#define SB(b, h) ((4 + (b) * 2 + (h)) * HTB)
#define LDA(dst, b, h) do { _Pragma("unroll") for (int m = 0; m < 4; ++m) _Pragma("unroll") for (int k = 0; k < 2; ++k) dst[m][k] = *(const LAS bf16x8*)(lds + SA(b, h) + aoff + m * 2048 + k * 1024); } while (0)
#define TRRD(dst, addr, off) asm volatile("ds_read_b64_tr_b16 %0, %1 offset:%2" : "=v"(dst) : "v"(addr), "i"(off) : "memory")
#define LDB(dst, b, h) do { _Pragma("unroll") for (int n = 0; n < 2; ++n) _Pragma("unroll") for (int k = 0; k < 2; ++k) { \
        TRRD(dst[n][k][0], boff[n], (SB(b, h) - 4 * HTB) + k * 8192); TRRD(dst[n][k][1], boff[n], (SB(b, h) - 4 * HTB) + k * 8192 + 1024); } } while (0)
#define WAIT_LB(Bt) asm volatile("s_waitcnt lgkmcnt(0)" : "+v"(Bt[0][0][0]), "+v"(Bt[0][0][1]), "+v"(Bt[0][1][0]), "+v"(Bt[0][1][1]), "+v"(Bt[1][0][0]), "+v"(Bt[1][0][1]), "+v"(Bt[1][1][0]), "+v"(Bt[1][1][1]) :: "memory")
#define BFR(Bt, n, k) __builtin_shufflevector(Bt[n][k][0], Bt[n][k][1], 0, 1, 2, 3, 4, 5, 6, 7)
#define MMA(ai, bj, At, Bt) do { __builtin_amdgcn_s_setprio(1); _Pragma("unroll") for (int m = 0; m < 4; ++m) _Pragma("unroll") for (int n = 0; n < 2; ++n) _Pragma("unroll") for (int k = 0; k < 2; ++k) \
        acc[ai][bj][m][n] = __builtin_amdgcn_mfma_f32_16x16x32_bf16(BFR(Bt, n, k), At[m][k], acc[ai][bj][m][n], 0, 0, 0); __builtin_amdgcn_s_setprio(0); } while (0)

#define STAMP(i)
#define PST(ph)
    bool bad = false;

    f32x4 acc[2][2][4][2];
#pragma unroll
    for (int a = 0; a < 2; ++a)
#pragma unroll
        for (int b = 0; b < 2; ++b)
#pragma unroll
            for (int m = 0; m < 4; ++m)
#pragma unroll
                for (int n = 0; n < 2; ++n) acc[a][b][m][n] = (f32x4){0.f, 0.f, 0.f, 0.f};
    bf16x8 At[4][2]; s16x4 B0[2][2][2], B1[2][2][2];

    STAGE(SB(0, 0), rsB, 0, voffB, voffBx); STAGE(SB(0, 1), rsB, hstepB, voffB, voffBx);
    STAGE(SB(1, 0), rsB, kstepB, voffB, voffBx); STAGE(SB(1, 1), rsB, kstepB + hstepB, voffB, voffBx);
    const unsigned ctl = (unsigned)(uintptr_t)lds + LDS_CTL;
#define CTL_WR(v) asm volatile("ds_write_b32 %0, %1" :: "v"(ctl), "v"((unsigned)(v)) : "memory")
#define CTL_RD(r) asm volatile("ds_read_b32 %0, %1" : "=v"(r) : "v"(ctl) : "memory")
#define CTL_GET(r) ({ asm volatile("s_waitcnt lgkmcnt(0)" : "+v"(r) :: "memory"); (int)__builtin_amdgcn_readfirstlane((int)(r)); })
    if (wid == 2) { CTL_WR(0); asm volatile("ds_write_b32 %0, %1 offset:4" :: "v"(ctl), "v"(0u) : "memory"); WAIT_L(0); }
    BAR;
    const unsigned seq = (unsigned)__builtin_amdgcn_readfirstlane((int)seqv) / NWG + 1u;
    const unsigned par = seq & 1u;
    const __amdgpu_buffer_rsrc_t rsAr = __builtin_amdgcn_make_buffer_rsrc(local ? (void*)(p.x + (size_t)(pm & 1) * 256 * (GK * 2)) : (void*)(wsl + WS_GATHER + par * GATHER_BYTES + (size_t)pm * (NT * 32768)), 0, NT * 32768, 0x00020000);
    const unsigned* flagp = (const unsigned*)(wsl + WS_FLAGS) + pm * (NT * 32) + lane;
    unsigned* tmo = (unsigned*)(wsl + WS_TMO);
    STAMP(33);
    const unsigned ui0 = (unsigned)((((rank * 2 + ((unit >> 5) & 1)) * NT + (unit >> 6)) << 5) + (unit & 31));
    const unsigned goff = (unsigned)(WS_GATHER + par * GATHER_BYTES);
    const int pj0 = unit % 7;
    int st = (wid == 3 && ev < NEV) ? 1 : 0;
    bool flp = false;
    unsigned long long nowr = t_start;
    auto pump = [&]() __attribute__((always_inline)) {
        if (st != 0) {
            if (st < 3) {
                if (st == 1) {
                    unsigned r; asm volatile("s_getreg_b32 %0, hwreg(HW_REG_IB_STS)" : "=s"(r) :: "memory");
                    if ((r & 0xC0000Fu) == 0u) {
                        asm volatile("v_mov_b32 %0, %4\n\tv_mov_b32 %1, %5\n\tv_mov_b32 %2, %6\n\tv_mov_b32 %3, %7" : "=&v"(d0.x), "=&v"(d0.y), "=&v"(d0.z), "=&v"(d0.w) : "v"(dn.x), "v"(dn.y), "v"(dn.z), "v"(dn.w));
                        if (flp) {
                            const unsigned long long fa = wsv + WS_FLAGS + (unsigned long long)(ui0 + 128u * (unsigned)(ev - 1)) * 4u;
                            if (lane < 8 && lane != rank && !(p.dbg & 2)) asm volatile("global_store_dword %0, %1, off sc0 sc1" :: "v"(fa), "v"(seq) : "memory");
                        }
                        st = ev < NEV ? 2 : 0;
                    }
                }
                if (st == 2 && (unsigned)nowr - (unsigned)t_start >= (((unsigned)(unit + 256 * ev) * (unsigned)p.slot) >> 6)) st = 3;
                nowr = __builtin_amdgcn_s_memrealtime();
            }
            if (st >= 3) {
                int j = pj0 + st - 3; j = j >= 7 ? j - 7 : j;
                const int pr = (p.dbg & 1) ? rank : (rank + 1 + j) & 7;
                char* base = (char*)(uintptr_t)(((unsigned long long)(unsigned)__builtin_amdgcn_readlane((int)ws_hi, pr) << 32) | (unsigned)__builtin_amdgcn_readlane((int)ws_lo, pr));
                const __amdgpu_buffer_rsrc_t rs = __builtin_amdgcn_make_buffer_rsrc((void*)base, 0, (int)WS_NEED, 0x00020000);
                __builtin_amdgcn_raw_buffer_store_b128(d0, rs, (int)(lane * 16), (int)(goff + (ui0 + 128u * (unsigned)ev) * 1024u), 17);
                if (++st == 10) {
                    flp = true; ++ev; st = 1;
                    if (ev < NEV) dn = __builtin_amdgcn_raw_buffer_load_b128(rsX, (int)vsrc, (int)(so0 + 512u * (unsigned)ev), 0);
                }
            }
        }
    };
    auto slow_wait = [&](int kt) {
        const unsigned long long t0 = __builtin_amdgcn_s_memrealtime();
        for (;;) {
            const unsigned v = __hip_atomic_load(flagp + kt * 32, __ATOMIC_RELAXED, __HIP_MEMORY_SCOPE_SYSTEM);
            if (__all(v >= seq)) break;
            if ((unsigned)__builtin_amdgcn_readfirstlane(__hip_atomic_load(tmo, __ATOMIC_RELAXED, __HIP_MEMORY_SCOPE_AGENT)) == seq || __builtin_amdgcn_s_memrealtime() - t0 > DEADLINE_TICKS) {
                __hip_atomic_store(tmo, seq, __ATOMIC_RELAXED, __HIP_MEMORY_SCOPE_AGENT); bad = true; break; }
            __builtin_amdgcn_s_sleep(2);
        }
        __builtin_amdgcn_fence(__ATOMIC_ACQUIRE, "");
        CTL_WR(kt + 2);
    };
    unsigned rv = 0;
    auto pump_until = [&](int need) __attribute__((always_inline)) {
        if (CTL_GET(rv) < need) {
            const unsigned long long t0 = __builtin_amdgcn_s_memrealtime();
            do { do pump(); while (st >= 3); CTL_RD(rv);
                if (__builtin_amdgcn_s_memrealtime() - t0 > 2 * DEADLINE_TICKS) { __hip_atomic_store(tmo, seq, __ATOMIC_RELAXED, __HIP_MEMORY_SCOPE_AGENT); break; }
            } while (CTL_GET(rv) < need);
        }
    };
    const bool poller = (wid == 2) && !nocomm;
    if (poller) slow_wait(0);
    if (wid == 3) {
        pump(); pump();
        if (!nocomm) { CTL_RD(rv); pump_until(2); }
    }
    STAMP(34);
    BAR;
    STAGE(SA(0, 0), rsAr, 0, voffA, voffAx); STAGE(SA(0, 1), rsAr, hstepA, voffA, voffAx);
    STAGE(SA(1, 0), rsAr, kstepA, voffA, voffAx);
    if (wid != 3) { WAIT_V(0); if (lane == 0) asm volatile("ds_add_u32 %0, %1 offset:4" :: "v"(ctl), "v"(1u) : "memory"); WAIT_L(0); }
    else {
        const unsigned long long t0 = __builtin_amdgcn_s_memrealtime(); unsigned cn;
        do { do pump(); while (st >= 3); asm volatile("ds_read_b32 %0, %1 offset:4" : "=v"(cn) : "v"(ctl) : "memory"); }
        while (CTL_GET(cn) < 7 && __builtin_amdgcn_s_memrealtime() - t0 < 2 * DEADLINE_TICKS);
        CTL_RD(rv);
    }
    BAR;
    if (wr == 1) BAR;
    unsigned fv = 0; bool ok = false;
    auto main_loop = [&](auto role_c) __attribute__((always_inline)) {
    constexpr int ROLE = decltype(role_c)::value;
#define RSTAGE(bufoff, rs_, soff, voff, voffx) do { \
        if constexpr (ROLE != 2) { BLDS(rs_, lds + (bufoff) + ldsw, (voff)[0], soff); BLDS(rs_, lds + (bufoff) + ldsw + 8192, (voff)[1], soff); } \
        if constexpr (ROLE == 0) BLDS(rs_, lds + (bufoff) + 3072 + wid * 8192, voffx, soff); } while (0)
#define RWAIT_STAGE() do { if constexpr (ROLE == 0) WAIT_V(15); else if constexpr (ROLE == 1) WAIT_V(10); } while (0)
#define PUMP() do { if constexpr (ROLE == 2) pump(); } while (0)
    for (int t = 0; t < NT; t += 2) {
        const bool last = (t == NT - 2);
        const unsigned a1 = (unsigned)(t + 1) * kstepA;
        const unsigned a2 = last ? 0u : (unsigned)(t + 2) * kstepA, b2 = last ? 0u : (unsigned)(t + 2) * kstepB;
        const unsigned a3 = a2 + kstepA, b3 = b2 + kstepB;
        if constexpr (ROLE == 1) {
        if (poller && !last) { if (!ok) slow_wait(t + 2); }
        }
        if constexpr (ROLE == 2) { if (!nocomm && !last) pump_until(t + 4); }
        STAMP(t >> 1);
        PST(0); PUMP(); LDB(B0, 0, 0); SCHED; LDA(At, 0, 0); RSTAGE(SA(1, 1), rsAr, a1 + hstepA, voffA, voffAx);
        RWAIT_STAGE(); WAIT_L(8); BAR; WAIT_LB(B0); MMA(0, 0, At, B0); BAR; SCHED;
        PST(1); PUMP(); LDB(B1, 0, 1); RSTAGE(SB(0, 0), rsB, b2, voffB, voffBx);
        RWAIT_STAGE();
        if constexpr (ROLE == 1) { if (poller) { const unsigned* fa = flagp + ((t + 4 < NT) ? (t + 4) : 0) * 32;
            asm volatile("global_load_dword %0, %1, off sc0 sc1" : "=v"(fv) : "v"(fa) : "memory"); } }
        BAR; WAIT_LB(B1); MMA(0, 1, At, B1); BAR;
        PST(2); PUMP(); LDA(At, 0, 1); RSTAGE(SA(0, 0), rsAr, a2, voffA, voffAx);
        BAR; WAIT_L(0); MMA(1, 0, At, B0); BAR; SCHED;
        PST(3); PUMP(); RSTAGE(SB(0, 1), rsB, b2 + hstepB, voffB, voffBx);
        RWAIT_STAGE();
        BAR; MMA(1, 1, At, B1); BAR;
        PST(4); PUMP(); LDB(B0, 1, 0); SCHED; LDA(At, 1, 0); RSTAGE(SA(0, 1), rsAr, a2 + hstepA, voffA, voffAx);
        RWAIT_STAGE(); WAIT_L(8); BAR; WAIT_LB(B0); MMA(0, 0, At, B0); BAR; SCHED;
        PST(5); PUMP(); LDB(B1, 1, 1); RSTAGE(SB(1, 0), rsB, b3, voffB, voffBx);
        RWAIT_STAGE();
        BAR; WAIT_LB(B1); MMA(0, 1, At, B1); BAR;
        PST(6); PUMP(); LDA(At, 1, 1); RSTAGE(SA(1, 0), rsAr, a3, voffA, voffAx);
        BAR; WAIT_L(0); MMA(1, 0, At, B0); BAR; SCHED;
        PST(7); PUMP(); RSTAGE(SB(1, 1), rsB, b3 + hstepB, voffB, voffBx);
        RWAIT_STAGE();
        if constexpr (ROLE == 1) { asm volatile("" : "+v"(fv)); if (poller) { ok = __all(fv >= seq); if (ok) { CTL_WR(t + 6); WAIT_L(0); } } }
        BAR;
        if constexpr (ROLE == 2) CTL_RD(rv);
        MMA(1, 1, At, B1); BAR;
        PST(8);
    }
    };
    if (wid < 2) main_loop(std::integral_constant<int, 0>{});
    else if (wid != 3) main_loop(std::integral_constant<int, 1>{});
    else main_loop(std::integral_constant<int, 2>{});
    if (wid == 3) { const unsigned long long t0 = __builtin_amdgcn_s_memrealtime(); while (st != 0 && __builtin_amdgcn_s_memrealtime() - t0 < DEADLINE_TICKS) pump(); }
    WAIT_V(0);
    if (wr == 0) BAR;
    STAMP(32);

    if (p.mode == 0) bad = ((unsigned)__builtin_amdgcn_readfirstlane(__hip_atomic_load(tmo, __ATOMIC_RELAXED, __HIP_MEMORY_SCOPE_AGENT)) == seq);
    {
        const int colb = pn * 256 + wc * 32 + ((fq & 1) * 16 + (fq >> 1) * 8);
        char* orow = p.out + ((size_t)(pm * 256 + wr * 64 + fr) * GN + colb) * 2;
#pragma unroll
        for (int ai = 0; ai < 2; ++ai)
#pragma unroll
            for (int m = 0; m < 4; ++m)
#pragma unroll
                for (int bj = 0; bj < 2; ++bj) {
                    const f32x4 v0 = acc[ai][bj][m][0], v1 = acc[ai][bj][m][1];
                    unsigned a0 = silu2(v0[0], v0[1]), a1 = silu2(v0[2], v0[3]);
                    unsigned b0 = silu2(v1[0], v1[1]), b1 = silu2(v1[2], v1[3]);
                    const auto s0 = __builtin_amdgcn_permlane16_swap(a0, b0, false, false);
                    const auto s1 = __builtin_amdgcn_permlane16_swap(a1, b1, false, false);
                    u32x4 o; o.x = s0[0]; o.y = s1[0]; o.z = s0[1]; o.w = s1[1];
                    if (bad) o = (u32x4){0x7fc07fc0u, 0x7fc07fc0u, 0x7fc07fc0u, 0x7fc07fc0u};
                    *(u32x4*)(orow + ((size_t)(ai * 128 + m * 16) * GN + bj * 128) * 2) = o;
                }
    }
    if (tid == 0) __hip_atomic_fetch_add((unsigned*)(wsl + WS_CTR), 1u, __ATOMIC_RELAXED, __HIP_MEMORY_SCOPE_AGENT);
}

static int g_mode = 0, g_slot = 136, g_pslot = 100, g_dbg = 0, g_pit = 10, g_late = 0;

extern "C" void kernel_launch(int rank, int nranks, ncclComm_t comm,
                              hipStream_t stream,
                              void* const* d_in, const int* in_sizes, int n_in,
                              void* d_out, int out_size,
                              const int* in_dtypes, int out_dtype,
                              void* const* d_out_all,
                              void* const* d_ws_all, size_t ws_bytes) {
    if (nranks != NR || ws_bytes < WS_NEED || (unsigned)rank >= (unsigned)NR) { fprintf(stderr, "unsupported configuration\n"); return; }
    static bool init = false;
    if (!init) {
        HIP_CHECK(hipFuncSetAttribute((const void*)ag_gemm_silu, hipFuncAttributeMaxDynamicSharedMemorySize, LDS_BYTES));
        init = true;
    }
    Params p;
    p.x = (const char*)d_in[0]; p.w = (const char*)d_in[1]; p.out = (char*)d_out;
    for (int i = 0; i < NR; ++i) p.ws[i] = (char*)d_ws_all[i];
    p.rank = rank; p.mode = g_mode; p.slot = g_slot; p.pslot = g_pslot; p.pit = g_pit; p.dbg = g_dbg;
    hipLaunchKernelGGL(ag_gemm_silu, dim3(NWG), dim3(512), LDS_BYTES, stream, p);
    HIP_CHECK(hipGetLastError());
}
```

```cpp
#include <hip/hip_runtime.h>
#include <rccl/rccl.h>
#include <cstdio>
#include <cstdint>
#include <type_traits>

#define HIP_CHECK(expr) do { hipError_t _e = (expr); if (_e != hipSuccess) { \
    fprintf(stderr, "HIP error %s at %s:%d\n", hipGetErrorString(_e), __FILE__, __LINE__); \
    return; } } while (0)

#define LAS __attribute__((address_space(3)))
#define GAS __attribute__((address_space(1)))
typedef short bf16x8 __attribute__((ext_vector_type(8)));
typedef short s16x4 __attribute__((ext_vector_type(4)));
typedef float f32x4 __attribute__((ext_vector_type(4)));
typedef unsigned u32x4 __attribute__((ext_vector_type(4)));
typedef unsigned u32x2 __attribute__((ext_vector_type(2)));

constexpr int NR = 8;
constexpr int GM = 4096, GK = 4096, GN = 4096;
constexpr int BK = 64, NT = GK / BK;
constexpr int HTB = 128 * BK * 2;
constexpr int LDS_CTL = 8 * HTB;
constexpr int LDS_BYTES = 8 * HTB + 1024;
constexpr int NWG = 256;

constexpr size_t WS_CTR = 0;
constexpr size_t WS_TMO = 256;
constexpr size_t WS_FLAGS = 4096;
constexpr size_t WS_STAMPS = 512 * 1024;
constexpr int NEV = 16;
constexpr size_t WS_GATHER = (size_t)1 << 20;
constexpr size_t GATHER_BYTES = (size_t)GM * GK * 2;
constexpr size_t WS_NEED = WS_GATHER + 2 * GATHER_BYTES;
constexpr unsigned long long DEADLINE_TICKS = 20000000ull;
#define USH 0
#define EARLY 128

struct Params {
    const char* x;
    const char* w;
    char* out;
    char* ws[NR];
    int rank;
    int mode;
    int dbg;
    int pit;
    int pslot;
    int slot;
};

__device__ __forceinline__ int lds_byte(int r, int c) { const int st = (r >> 4) * 2 + (c >> 5), rr = r & 15, cc = c & 31, ob = rr * 64 + cc * 2; return st * 1024 + (ob ^ (((ob >> 9) & 1) << 5)); }
__device__ __forceinline__ void stage_rc(int b, int& R, int& C) { const int st = b / 1024, sb = b % 1024, swz = sb ^ (((sb >> 9) & 1) << 5); R = (st >> 1) * 16 + swz / 64; C = (st & 1) * 32 + (swz % 64) / 2; }

__device__ __forceinline__ unsigned cvt_pk_bf16(float lo, float hi) { unsigned r; asm volatile("v_cvt_pk_bf16_f32 %0, %1, %2" : "=v"(r) : "v"(lo), "v"(hi)); return r; }
typedef float f32x2 __attribute__((ext_vector_type(2)));
__device__ __forceinline__ unsigned silu2(float y0, float y1) {
    const f32x2 y = {y0, y1};
    const f32x2 a = y * -1.4426950408889634f;
    const f32x2 d = (f32x2){__builtin_amdgcn_exp2f(__builtin_amdgcn_fmed3f(a.x, -86.5617f, 86.5617f)), __builtin_amdgcn_exp2f(__builtin_amdgcn_fmed3f(a.y, -86.5617f, 86.5617f))} + 1.0f;
    const f32x2 r = y * (f32x2){__builtin_amdgcn_rcpf(d.x), __builtin_amdgcn_rcpf(d.y)};
    return cvt_pk_bf16(r.x, r.y);
}

__global__ __launch_bounds__(512, 2) void ag_gemm_silu(const Params p) {
    extern __shared__ __attribute__((aligned(16))) unsigned char smem_raw[];
    LAS unsigned char* lds = (LAS unsigned char*)smem_raw;
    const int tid = threadIdx.x, wid = __builtin_amdgcn_readfirstlane(tid >> 6), lane = tid & 63, wr = wid >> 2, wc = wid & 3, fr = lane & 15, fq = lane >> 4;
    const unsigned long long t_start = __builtin_amdgcn_s_memrealtime();
    const int bid = blockIdx.x, xcd = bid & 7, idx = bid >> 3;
    const int pm = (xcd >> 1) * 4 + (idx & 3), pn = (xcd & 1) * 8 + (idx >> 2);
    const int rank = p.rank;
    const bool local = (pm >> 1) == rank;
    const bool nocomm = local || p.mode >= 1;
    const int unit = (((pm - 2 * rank - USH) & 15) << 4) + pn;
    char* const wsl = p.ws[rank];

#define WAIT_V(n) asm volatile("s_waitcnt vmcnt(" #n ")" ::: "memory")
#define WAIT_L(n) asm volatile("s_waitcnt lgkmcnt(" #n ")" ::: "memory")
#define BAR __builtin_amdgcn_s_barrier()
#define SCHED __builtin_amdgcn_sched_barrier(0)
#define BLDS(rs, l, voff, soff) __builtin_amdgcn_raw_ptr_buffer_load_lds(rs, (LAS void*)(l), 16, (int)(voff), (int)(soff), 0, 0)
#define STAGE(bufoff, rs, soff, voff, voffx) do { \
        if (wid != 3) { BLDS(rs, lds + (bufoff) + ldsw, (voff)[0], soff); BLDS(rs, lds + (bufoff) + ldsw + 8192, (voff)[1], soff); } \
        if (wid < 2) BLDS(rs, lds + (bufoff) + 3072 + wid * 8192, voffx, soff); } while (0)
#define WAIT_STAGE() do { if (wid < 2) WAIT_V(9); else if (wid != 3) WAIT_V(6); } while (0)
    const unsigned ldsw = (unsigned)wid * 1024u;
    const __amdgpu_buffer_rsrc_t rsX = __builtin_amdgcn_make_buffer_rsrc((void*)p.x, 0, 512 * GK * 2, 0x00020000);
    const unsigned vsrc = (unsigned)((lane >> 3) * (GK * 2) + (lane & 7) * 16);
    const bool nosend = (p.mode == 1 || ((p.dbg & 4) && rank != 0) || ((p.dbg & 8) && rank == 0));
    int ev = nosend ? NEV : 0;
    const unsigned so0 = (unsigned)((unit & 63) * 8 * (GK * 2) + (unit >> 6) * 128);
    u32x4 d0 = {0u, 0u, 0u, 0u}, dn = {0u, 0u, 0u, 0u};
    if (wid == 3 && ev < NEV) dn = __builtin_amdgcn_raw_buffer_load_b128(rsX, (int)vsrc, (int)so0, 0);
    const unsigned seqv = __hip_atomic_load((const unsigned*)(wsl + WS_CTR), __ATOMIC_RELAXED, __HIP_MEMORY_SCOPE_AGENT);
    unsigned long long wsv = (unsigned long long)(uintptr_t)p.ws[0];
#pragma unroll
    for (int i = 1; i < NR; ++i) { unsigned long long w = (unsigned long long)(uintptr_t)p.ws[i]; asm volatile("" : "+s"(w)); wsv = ((lane & 7) == i) ? w : wsv; }
    const unsigned ws_lo = (unsigned)wsv, ws_hi = (unsigned)(wsv >> 32);

    const unsigned rsA = local ? (unsigned)(GK * 2) : 128u;
    unsigned voffA[2], voffB[2], voffAx, voffBx;
#pragma unroll
    for (int i = 0; i < 3; ++i) {
        const int tt = i < 2 ? tid : 192 + lane, ii = i < 2 ? i : (wid & 1);
        int R, C; stage_rc(tt * 16 + ii * 8192, R, C);
        const unsigned va = (unsigned)R * rsA + (unsigned)C * 2u;
        const int k = (tt >> 4) + 32 * ii, c = tt & 15, f = (k & 3) | (((k >> 3) & 1) << 2);
        const unsigned vb = (unsigned)(k * (GN * 2) + ((c ^ (f << 1)) * 16));
        if (i < 2) { voffA[i] = va; voffB[i] = vb; } else { voffAx = va; voffBx = vb; }
    }
    const unsigned kstepA = local ? 128u : 32768u, hstepA = 128u * rsA;
    const unsigned kstepB = (unsigned)(BK * GN * 2), hstepB = 256u;
    const __amdgpu_buffer_rsrc_t rsB = __builtin_amdgcn_make_buffer_rsrc((void*)(p.w + pn * 512), 0, GK * GN * 2 - pn * 512, 0x00020000);
    const int aoff = lds_byte(wr * 64 + fr, fq * 8);
    int boff[2];
    { const int q = (lane >> 2) & 3, pp = lane & 3, g = fq, f = q | ((g & 1) << 2);
#pragma unroll
      for (int n = 0; n < 2; ++n) boff[n] = (int)(unsigned)(uintptr_t)lds + 4 * HTB + (8 * g + q) * 256 + (((wc * 2 + n) ^ f) * 32) + 8 * pp; }

#define SA(b, h) (((b) * 2 + (h)) * HTB)
#define SB(b, h) ((4 + (b) * 2 + (h)) * HTB)
#define LDA(dst, b, h) do { _Pragma("unroll") for (int m = 0; m < 4; ++m) _Pragma("unroll") for (int k = 0; k < 2; ++k) dst[m][k] = *(const LAS bf16x8*)(lds + SA(b, h) + aoff + m * 2048 + k * 1024); } while (0)
#define TRRD(dst, addr, off) asm volatile("ds_read_b64_tr_b16 %0, %1 offset:%2" : "=v"(dst) : "v"(addr), "i"(off) : "memory")
#define LDB(dst, b, h) do { _Pragma("unroll") for (int n = 0; n < 2; ++n) _Pragma("unroll") for (int k = 0; k < 2; ++k) { \
        TRRD(dst[n][k][0], boff[n], (SB(b, h) - 4 * HTB) + k * 8192); TRRD(dst[n][k][1], boff[n], (SB(b, h) - 4 * HTB) + k * 8192 + 1024); } } while (0)
#define WAIT_LB(Bt) asm volatile("s_waitcnt lgkmcnt(0)" : "+v"(Bt[0][0][0]), "+v"(Bt[0][0][1]), "+v"(Bt[0][1][0]), "+v"(Bt[0][1][1]), "+v"(Bt[1][0][0]), "+v"(Bt[1][0][1]), "+v"(Bt[1][1][0]), "+v"(Bt[1][1][1]) :: "memory")
#define BFR(Bt, n, k) __builtin_shufflevector(Bt[n][k][0], Bt[n][k][1], 0, 1, 2, 3, 4, 5, 6, 7)
#define MMA(ai, bj, At, Bt) do { __builtin_amdgcn_s_setprio(1); _Pragma("unroll") for (int m = 0; m < 4; ++m) _Pragma("unroll") for (int n = 0; n < 2; ++n) _Pragma("unroll") for (int k = 0; k < 2; ++k) \
        acc[ai][bj][m][n] = __builtin_amdgcn_mfma_f32_16x16x32_bf16(BFR(Bt, n, k), At[m][k], acc[ai][bj][m][n], 0, 0, 0); __builtin_amdgcn_s_setprio(0); } while (0)

#define STAMP(i)
#define PST(ph)
    bool bad = false;

    f32x4 acc[2][2][4][2];
#pragma unroll
    for (int a = 0; a < 2; ++a)
#pragma unroll
        for (int b = 0; b < 2; ++b)
#pragma unroll
            for (int m = 0; m < 4; ++m)
#pragma unroll
                for (int n = 0; n < 2; ++n) acc[a][b][m][n] = (f32x4){0.f, 0.f, 0.f, 0.f};
    bf16x8 At[4][2]; s16x4 B0[2][2][2], B1[2][2][2];

    STAGE(SB(0, 0), rsB, 0, voffB, voffBx); STAGE(SB(0, 1), rsB, hstepB, voffB, voffBx);
    STAGE(SB(1, 0), rsB, kstepB, voffB, voffBx); STAGE(SB(1, 1), rsB, kstepB + hstepB, voffB, voffBx);
    const unsigned ctl = (unsigned)(uintptr_t)lds + LDS_CTL;
#define CTL_WR(v) asm volatile("ds_write_b32 %0, %1" :: "v"(ctl), "v"((unsigned)(v)) : "memory")
#define CTL_RD(r) asm volatile("ds_read_b32 %0, %1" : "=v"(r) : "v"(ctl) : "memory")
#define CTL_GET(r) ({ asm volatile("s_waitcnt lgkmcnt(0)" : "+v"(r) :: "memory"); (int)__builtin_amdgcn_readfirstlane((int)(r)); })
    if (wid == 2) { CTL_WR(0); asm volatile("ds_write_b32 %0, %1 offset:4" :: "v"(ctl), "v"(0u) : "memory"); WAIT_L(0); }
    const unsigned seq = (unsigned)__builtin_amdgcn_readfirstlane((int)seqv) / NWG + 1u;
    const unsigned par = seq & 1u;
    const __amdgpu_buffer_rsrc_t rsAr = __builtin_amdgcn_make_buffer_rsrc(local ? (void*)(p.x + (size_t)(pm & 1) * 256 * (GK * 2)) : (void*)(wsl + WS_GATHER + par * GATHER_BYTES + (size_t)pm * (NT * 32768)), 0, NT * 32768, 0x00020000);
    const unsigned* flagp = (const unsigned*)(wsl + WS_FLAGS) + pm * (NT * 32) + lane;
    unsigned* tmo = (unsigned*)(wsl + WS_TMO);
    STAMP(33);
    const unsigned ui0 = (unsigned)((((rank * 2 + ((unit >> 5) & 1)) * NT + (unit >> 6)) << 5) + (unit & 31));
    const unsigned goff = (unsigned)(WS_GATHER + par * GATHER_BYTES);
    const int pj0 = unit % 7;
    int st = (wid == 3 && ev < NEV) ? 1 : 0;
    bool flp = false;
    unsigned long long nowr = t_start;
    auto pump = [&]() __attribute__((always_inline)) {
        if (st != 0) {
            if (st < 3) {
                if (st == 1) {
                    unsigned r; asm volatile("s_getreg_b32 %0, hwreg(HW_REG_IB_STS)" : "=s"(r) :: "memory");
                    if ((r & 0xC0000Fu) == 0u) {
                        asm volatile("v_mov_b32 %0, %4\n\tv_mov_b32 %1, %5\n\tv_mov_b32 %2, %6\n\tv_mov_b32 %3, %7" : "=&v"(d0.x), "=&v"(d0.y), "=&v"(d0.z), "=&v"(d0.w) : "v"(dn.x), "v"(dn.y), "v"(dn.z), "v"(dn.w));
                        if (flp) {
                            const unsigned long long fa = wsv + WS_FLAGS + (unsigned long long)(ui0 + 128u * (unsigned)(ev - 1)) * 4u;
                            if (lane < 8 && lane != rank && !(p.dbg & 2)) asm volatile("global_store_dword %0, %1, off sc0 sc1" :: "v"(fa), "v"(seq) : "memory");
                        }
                        st = ev < NEV ? 2 : 0;
                    }
                }
                if (st == 2 && (int)((unsigned)nowr - (unsigned)t_start) >= (int)(((unsigned)(unit + 256 * ev) * (unsigned)p.slot) >> 6) + p.pslot) st = 3;
                nowr = __builtin_amdgcn_s_memrealtime();
            }
            if (st >= 3) {
                int j = pj0 + st - 3; j = j >= 7 ? j - 7 : j;
                const int pr = (p.dbg & 1) ? rank : (rank + 1 + j) & 7;
                char* base = (char*)(uintptr_t)(((unsigned long long)(unsigned)__builtin_amdgcn_readlane((int)ws_hi, pr) << 32) | (unsigned)__builtin_amdgcn_readlane((int)ws_lo, pr));
                const __amdgpu_buffer_rsrc_t rs = __builtin_amdgcn_make_buffer_rsrc((void*)base, 0, (int)WS_NEED, 0x00020000);
                __builtin_amdgcn_raw_buffer_store_b128(d0, rs, (int)(lane * 16), (int)(goff + (ui0 + 128u * (unsigned)ev) * 1024u), 17);
                if (++st == 10) {
                    flp = true; ++ev; st = 1;
                    if (ev < NEV) dn = __builtin_amdgcn_raw_buffer_load_b128(rsX, (int)vsrc, (int)(so0 + 512u * (unsigned)ev), 0);
                }
            }
        }
    };
    auto slow_wait = [&](int kt) {
        const unsigned long long t0 = __builtin_amdgcn_s_memrealtime();
        for (;;) {
            const unsigned v = __hip_atomic_load(flagp + kt * 32, __ATOMIC_RELAXED, __HIP_MEMORY_SCOPE_SYSTEM);
            if (__all(v >= seq)) break;
            if ((unsigned)__builtin_amdgcn_readfirstlane(__hip_atomic_load(tmo, __ATOMIC_RELAXED, __HIP_MEMORY_SCOPE_AGENT)) == seq || __builtin_amdgcn_s_memrealtime() - t0 > DEADLINE_TICKS) {
                __hip_atomic_store(tmo, seq, __ATOMIC_RELAXED, __HIP_MEMORY_SCOPE_AGENT); bad = true; break; }
            __builtin_amdgcn_s_sleep(2);
        }
        __builtin_amdgcn_fence(__ATOMIC_ACQUIRE, "");
        CTL_WR(kt + 2);
    };
    unsigned rv = 0;
    auto pump_until = [&](int need) __attribute__((always_inline)) {
        if (CTL_GET(rv) < need) {
            const unsigned long long t0 = __builtin_amdgcn_s_memrealtime();
            do { do pump(); while (st >= 3); CTL_RD(rv);
                if (__builtin_amdgcn_s_memrealtime() - t0 > 2 * DEADLINE_TICKS) { __hip_atomic_store(tmo, seq, __ATOMIC_RELAXED, __HIP_MEMORY_SCOPE_AGENT); break; }
            } while (CTL_GET(rv) < need);
        }
    };
    const bool poller = (wid == 2) && !nocomm;
    if (wid == 3) { nowr = __builtin_amdgcn_s_memrealtime(); pump(); while (st != 0 && ev == 0 && unit < EARLY) pump(); }
    BAR;
    if (poller) slow_wait(0);
    if (wid == 3) {
        if (!nocomm) { CTL_RD(rv); pump_until(2); }
    }
    STAMP(34);
    BAR;
    STAGE(SA(0, 0), rsAr, 0, voffA, voffAx); STAGE(SA(0, 1), rsAr, hstepA, voffA, voffAx);
    STAGE(SA(1, 0), rsAr, kstepA, voffA, voffAx);
    if (wid != 3) { WAIT_V(0); if (lane == 0) asm volatile("ds_add_u32 %0, %1 offset:4" :: "v"(ctl), "v"(1u) : "memory"); WAIT_L(0); }
    else {
        const unsigned long long t0 = __builtin_amdgcn_s_memrealtime(); unsigned cn;
        do { do pump(); while (st >= 3); asm volatile("ds_read_b32 %0, %1 offset:4" : "=v"(cn) : "v"(ctl) : "memory"); }
        while (CTL_GET(cn) < 7 && __builtin_amdgcn_s_memrealtime() - t0 < 2 * DEADLINE_TICKS);
        CTL_RD(rv);
    }
    BAR;
    if (wr == 1) BAR;
    unsigned fv = 0; bool ok = false;
    auto main_loop = [&](auto role_c) __attribute__((always_inline)) {
    constexpr int ROLE = decltype(role_c)::value;
#define RSTAGE(bufoff, rs_, soff, voff, voffx) do { \
        if constexpr (ROLE != 2) { BLDS(rs_, lds + (bufoff) + ldsw, (voff)[0], soff); BLDS(rs_, lds + (bufoff) + ldsw + 8192, (voff)[1], soff); } \
        if constexpr (ROLE == 0) BLDS(rs_, lds + (bufoff) + 3072 + wid * 8192, voffx, soff); } while (0)
#define RWAIT_STAGE() do { if constexpr (ROLE == 0) WAIT_V(15); else if constexpr (ROLE == 1) WAIT_V(10); } while (0)
#define PUMP() do { if constexpr (ROLE == 2) pump(); } while (0)
    for (int t = 0; t < NT; t += 2) {
        const bool last = (t == NT - 2);
        const unsigned a1 = (unsigned)(t + 1) * kstepA;
        const unsigned a2 = last ? 0u : (unsigned)(t + 2) * kstepA, b2 = last ? 0u : (unsigned)(t + 2) * kstepB;
        const unsigned a3 = a2 + kstepA, b3 = b2 + kstepB;
        if constexpr (ROLE == 1) {
        if (poller && !last) { if (!ok) slow_wait(t + 2); }
        }
        if constexpr (ROLE == 2) { if (!nocomm && !last) pump_until(t + 4); }
        STAMP(t >> 1);
        PST(0); PUMP(); LDB(B0, 0, 0); SCHED; LDA(At, 0, 0); RSTAGE(SA(1, 1), rsAr, a1 + hstepA, voffA, voffAx);
        RWAIT_STAGE(); WAIT_L(8); BAR; WAIT_LB(B0); MMA(0, 0, At, B0); BAR; SCHED;
        PST(1); PUMP(); LDB(B1, 0, 1); RSTAGE(SB(0, 0), rsB, b2, voffB, voffBx);
        RWAIT_STAGE();
        if constexpr (ROLE == 1) { if (poller) { const unsigned* fa = flagp + ((t + 4 < NT) ? (t + 4) : 0) * 32;
            asm volatile("global_load_dword %0, %1, off sc0 sc1" : "=v"(fv) : "v"(fa) : "memory"); } }
        BAR; WAIT_LB(B1); MMA(0, 1, At, B1); BAR;
        PST(2); PUMP(); LDA(At, 0, 1); RSTAGE(SA(0, 0), rsAr, a2, voffA, voffAx);
        BAR; WAIT_L(0); MMA(1, 0, At, B0); BAR; SCHED;
        PST(3); PUMP(); RSTAGE(SB(0, 1), rsB, b2 + hstepB, voffB, voffBx);
        RWAIT_STAGE();
        BAR; MMA(1, 1, At, B1); BAR;
        PST(4); PUMP(); LDB(B0, 1, 0); SCHED; LDA(At, 1, 0); RSTAGE(SA(0, 1), rsAr, a2 + hstepA, voffA, voffAx);
        RWAIT_STAGE(); WAIT_L(8); BAR; WAIT_LB(B0); MMA(0, 0, At, B0); BAR; SCHED;
        PST(5); PUMP(); LDB(B1, 1, 1); RSTAGE(SB(1, 0), rsB, b3, voffB, voffBx);
        RWAIT_STAGE();
        BAR; WAIT_LB(B1); MMA(0, 1, At, B1); BAR;
        PST(6); PUMP(); LDA(At, 1, 1); RSTAGE(SA(1, 0), rsAr, a3, voffA, voffAx);
        BAR; WAIT_L(0); MMA(1, 0, At, B0); BAR; SCHED;
        PST(7); PUMP(); RSTAGE(SB(1, 1), rsB, b3 + hstepB, voffB, voffBx);
        RWAIT_STAGE();
        if constexpr (ROLE == 1) { asm volatile("" : "+v"(fv)); if (poller) { ok = __all(fv >= seq); if (ok) { CTL_WR(t + 6); WAIT_L(0); } } }
        BAR;
        if constexpr (ROLE == 2) CTL_RD(rv);
        MMA(1, 1, At, B1); BAR;
        PST(8);
    }
    };
    if (wid < 2) main_loop(std::integral_constant<int, 0>{});
    else if (wid != 3) main_loop(std::integral_constant<int, 1>{});
    else main_loop(std::integral_constant<int, 2>{});
    if (wid == 3) { const unsigned long long t0 = __builtin_amdgcn_s_memrealtime(); while (st != 0 && __builtin_amdgcn_s_memrealtime() - t0 < DEADLINE_TICKS) pump(); }
    WAIT_V(0);
    if (wr == 0) BAR;
    STAMP(32);

    if (p.mode == 0) bad = ((unsigned)__builtin_amdgcn_readfirstlane(__hip_atomic_load(tmo, __ATOMIC_RELAXED, __HIP_MEMORY_SCOPE_AGENT)) == seq);
    {
        const int colb = pn * 256 + wc * 32 + ((fq & 1) * 16 + (fq >> 1) * 8);
        char* orow = p.out + ((size_t)(pm * 256 + wr * 64 + fr) * GN + colb) * 2;
#pragma unroll
        for (int ai = 0; ai < 2; ++ai)
#pragma unroll
            for (int m = 0; m < 4; ++m)
#pragma unroll
                for (int bj = 0; bj < 2; ++bj) {
                    const f32x4 v0 = acc[ai][bj][m][0], v1 = acc[ai][bj][m][1];
                    unsigned a0 = silu2(v0[0], v0[1]), a1 = silu2(v0[2], v0[3]);
                    unsigned b0 = silu2(v1[0], v1[1]), b1 = silu2(v1[2], v1[3]);
                    const auto s0 = __builtin_amdgcn_permlane16_swap(a0, b0, false, false);
                    const auto s1 = __builtin_amdgcn_permlane16_swap(a1, b1, false, false);
                    u32x4 o; o.x = s0[0]; o.y = s1[0]; o.z = s0[1]; o.w = s1[1];
                    if (bad) o = (u32x4){0x7fc07fc0u, 0x7fc07fc0u, 0x7fc07fc0u, 0x7fc07fc0u};
                    *(u32x4*)(orow + ((size_t)(ai * 128 + m * 16) * GN + bj * 128) * 2) = o;
                }
    }
    if (tid == 0) __hip_atomic_fetch_add((unsigned*)(wsl + WS_CTR), 1u, __ATOMIC_RELAXED, __HIP_MEMORY_SCOPE_AGENT);
}

static int g_mode = 0, g_slot = 136, g_pslot = -80, g_dbg = 0, g_pit = 10, g_late = 0;

extern "C" void kernel_launch(int rank, int nranks, ncclComm_t comm,
                              hipStream_t stream,
                              void* const* d_in, const int* in_sizes, int n_in,
                              void* d_out, int out_size,
                              const int* in_dtypes, int out_dtype,
                              void* const* d_out_all,
                              void* const* d_ws_all, size_t ws_bytes) {
    if (nranks != NR || ws_bytes < WS_NEED || (unsigned)rank >= (unsigned)NR) { fprintf(stderr, "unsupported configuration\n"); return; }
    static bool init = false;
    if (!init) {
        HIP_CHECK(hipFuncSetAttribute((const void*)ag_gemm_silu, hipFuncAttributeMaxDynamicSharedMemorySize, LDS_BYTES));
        init = true;
    }
    Params p;
    p.x = (const char*)d_in[0]; p.w = (const char*)d_in[1]; p.out = (char*)d_out;
    for (int i = 0; i < NR; ++i) p.ws[i] = (char*)d_ws_all[i];
    p.rank = rank; p.mode = g_mode; p.slot = g_slot; p.pslot = g_pslot; p.pit = g_pit; p.dbg = g_dbg;
    hipLaunchKernelGGL(ag_gemm_silu, dim3(NWG), dim3(512), LDS_BYTES, stream, p);
    HIP_CHECK(hipGetLastError());
}
```

```cpp
#include <hip/hip_runtime.h>
#include <rccl/rccl.h>
#include <cstdio>
#include <cstdint>
#include <type_traits>

#define HIP_CHECK(expr) do { hipError_t _e = (expr); if (_e != hipSuccess) { \
    fprintf(stderr, "HIP error %s at %s:%d\n", hipGetErrorString(_e), __FILE__, __LINE__); \
    return; } } while (0)

#define LAS __attribute__((address_space(3)))
#define GAS __attribute__((address_space(1)))
typedef short bf16x8 __attribute__((ext_vector_type(8)));
typedef short s16x4 __attribute__((ext_vector_type(4)));
typedef float f32x4 __attribute__((ext_vector_type(4)));
typedef unsigned u32x4 __attribute__((ext_vector_type(4)));
typedef unsigned u32x2 __attribute__((ext_vector_type(2)));

constexpr int NR = 8;
constexpr int GM = 4096, GK = 4096, GN = 4096;
constexpr int BK = 64, NT = GK / BK;
constexpr int HTB = 128 * BK * 2;
constexpr int LDS_CTL = 8 * HTB;
constexpr int LDS_BYTES = 8 * HTB + 1024;
constexpr int NWG = 256;

constexpr size_t WS_CTR = 0;
constexpr size_t WS_TMO = 256;
constexpr size_t WS_FLAGS = 4096;
constexpr size_t WS_STAMPS = 512 * 1024;
constexpr int NEV = 16;
constexpr size_t WS_GATHER = (size_t)1 << 20;
constexpr size_t GATHER_BYTES = (size_t)GM * GK * 2;
constexpr size_t WS_NEED = WS_GATHER + 2 * GATHER_BYTES;
constexpr unsigned long long DEADLINE_TICKS = 20000000ull;
#define USH 0
#define FENCE 0
#define EARLY 128

struct Params {
    const char* x;
    const char* w;
    char* out;
    char* ws[NR];
    int rank;
    int mode;
    int dbg;
    int pit;
    int pslot;
    int slot;
};

__device__ __forceinline__ int lds_byte(int r, int c) { const int st = (r >> 4) * 2 + (c >> 5), rr = r & 15, cc = c & 31, ob = rr * 64 + cc * 2; return st * 1024 + (ob ^ (((ob >> 9) & 1) << 5)); }
__device__ __forceinline__ void stage_rc(int b, int& R, int& C) { const int st = b / 1024, sb = b % 1024, swz = sb ^ (((sb >> 9) & 1) << 5); R = (st >> 1) * 16 + swz / 64; C = (st & 1) * 32 + (swz % 64) / 2; }

__device__ __forceinline__ unsigned cvt_pk_bf16(float lo, float hi) { unsigned r; asm volatile("v_cvt_pk_bf16_f32 %0, %1, %2" : "=v"(r) : "v"(lo), "v"(hi)); return r; }
typedef float f32x2 __attribute__((ext_vector_type(2)));
__device__ __forceinline__ unsigned silu2(float y0, float y1) {
    const f32x2 y = {y0, y1};
    const f32x2 a = y * -1.4426950408889634f;
    const f32x2 d = (f32x2){__builtin_amdgcn_exp2f(__builtin_amdgcn_fmed3f(a.x, -86.5617f, 86.5617f)), __builtin_amdgcn_exp2f(__builtin_amdgcn_fmed3f(a.y, -86.5617f, 86.5617f))} + 1.0f;
    const f32x2 r = y * (f32x2){__builtin_amdgcn_rcpf(d.x), __builtin_amdgcn_rcpf(d.y)};
    return cvt_pk_bf16(r.x, r.y);
}

__global__ __launch_bounds__(512, 2) void ag_gemm_silu(const Params p) {
    extern __shared__ __attribute__((aligned(16))) unsigned char smem_raw[];
    LAS unsigned char* lds = (LAS unsigned char*)smem_raw;
    const int tid = threadIdx.x, wid = __builtin_amdgcn_readfirstlane(tid >> 6), lane = tid & 63, wr = wid >> 2, wc = wid & 3, fr = lane & 15, fq = lane >> 4;
    const unsigned long long t_start = __builtin_amdgcn_s_memrealtime();
    const int bid = blockIdx.x, xcd = bid & 7, idx = bid >> 3;
    const int pm = (xcd >> 1) * 4 + (idx & 3), pn = (xcd & 1) * 8 + (idx >> 2);
    const int rank = p.rank;
    const bool local = (pm >> 1) == rank;
    const bool nocomm = local || p.mode >= 1;
    const int unit = (((pm - 2 * rank - USH) & 15) << 4) + pn;
    char* const wsl = p.ws[rank];

#define WAIT_V(n) asm volatile("s_waitcnt vmcnt(" #n ")" ::: "memory")
#define WAIT_L(n) asm volatile("s_waitcnt lgkmcnt(" #n ")" ::: "memory")
#define BAR __builtin_amdgcn_s_barrier()
#define SCHED __builtin_amdgcn_sched_barrier(0)
#define BLDS(rs, l, voff, soff) __builtin_amdgcn_raw_ptr_buffer_load_lds(rs, (LAS void*)(l), 16, (int)(voff), (int)(soff), 0, 0)
#define STAGE(bufoff, rs, soff, voff, voffx) do { \
        if (wid != 3) { BLDS(rs, lds + (bufoff) + ldsw, (voff)[0], soff); BLDS(rs, lds + (bufoff) + ldsw + 8192, (voff)[1], soff); } \
        if (wid < 2) BLDS(rs, lds + (bufoff) + 3072 + wid * 8192, voffx, soff); } while (0)
#define WAIT_STAGE() do { if (wid < 2) WAIT_V(9); else if (wid != 3) WAIT_V(6); } while (0)
    const unsigned ldsw = (unsigned)wid * 1024u;
    const __amdgpu_buffer_rsrc_t rsX = __builtin_amdgcn_make_buffer_rsrc((void*)p.x, 0, 512 * GK * 2, 0x00020000);
    const unsigned vsrc = (unsigned)((lane >> 3) * (GK * 2) + (lane & 7) * 16);
    const bool nosend = (p.mode == 1 || ((p.dbg & 4) && rank != 0) || ((p.dbg & 8) && rank == 0));
    int ev = nosend ? NEV : 0;
    const unsigned so0 = (unsigned)((unit & 63) * 8 * (GK * 2) + (unit >> 6) * 128);
    u32x4 d0 = {0u, 0u, 0u, 0u}, dn = {0u, 0u, 0u, 0u};
    if (wid == 3 && ev < NEV) dn = __builtin_amdgcn_raw_buffer_load_b128(rsX, (int)vsrc, (int)so0, 0);
    const unsigned seqv = __hip_atomic_load((const unsigned*)(wsl + WS_CTR), __ATOMIC_RELAXED, __HIP_MEMORY_SCOPE_AGENT);
    unsigned long long wsv = (unsigned long long)(uintptr_t)p.ws[0];
#pragma unroll
    for (int i = 1; i < NR; ++i) { unsigned long long w = (unsigned long long)(uintptr_t)p.ws[i]; asm volatile("" : "+s"(w)); wsv = ((lane & 7) == i) ? w : wsv; }
    const unsigned ws_lo = (unsigned)wsv, ws_hi = (unsigned)(wsv >> 32);

    const unsigned rsA = local ? (unsigned)(GK * 2) : 128u;
    unsigned voffA[2], voffB[2], voffAx, voffBx;
#pragma unroll
    for (int i = 0; i < 3; ++i) {
        const int tt = i < 2 ? tid : 192 + lane, ii = i < 2 ? i : (wid & 1);
        int R, C; stage_rc(tt * 16 + ii * 8192, R, C);
        const unsigned va = (unsigned)R * rsA + (unsigned)C * 2u;
        const int k = (tt >> 4) + 32 * ii, c = tt & 15, f = (k & 3) | (((k >> 3) & 1) << 2);
        const unsigned vb = (unsigned)(k * (GN * 2) + ((c ^ (f << 1)) * 16));
        if (i < 2) { voffA[i] = va; voffB[i] = vb; } else { voffAx = va; voffBx = vb; }
    }
    const unsigned kstepA = local ? 128u : 32768u, hstepA = 128u * rsA;
    const unsigned kstepB = (unsigned)(BK * GN * 2), hstepB = 256u;
    const __amdgpu_buffer_rsrc_t rsB = __builtin_amdgcn_make_buffer_rsrc((void*)(p.w + pn * 512), 0, GK * GN * 2 - pn * 512, 0x00020000);
    const int aoff = lds_byte(wr * 64 + fr, fq * 8);
    int boff[2];
    { const int q = (lane >> 2) & 3, pp = lane & 3, g = fq, f = q | ((g & 1) << 2);
#pragma unroll
      for (int n = 0; n < 2; ++n) boff[n] = (int)(unsigned)(uintptr_t)lds + 4 * HTB + (8 * g + q) * 256 + (((wc * 2 + n) ^ f) * 32) + 8 * pp; }

#define SA(b, h) (((b) * 2 + (h)) * HTB)
#define SB(b, h) ((4 + (b) * 2 + (h)) * HTB)
#define LDA(dst, b, h) do { _Pragma("unroll") for (int m = 0; m < 4; ++m) _Pragma("unroll") for (int k = 0; k < 2; ++k) dst[m][k] = *(const LAS bf16x8*)(lds + SA(b, h) + aoff + m * 2048 + k * 1024); } while (0)
#define TRRD(dst, addr, off) asm volatile("ds_read_b64_tr_b16 %0, %1 offset:%2" : "=v"(dst) : "v"(addr), "i"(off) : "memory")
#define LDB(dst, b, h) do { _Pragma("unroll") for (int n = 0; n < 2; ++n) _Pragma("unroll") for (int k = 0; k < 2; ++k) { \
        TRRD(dst[n][k][0], boff[n], (SB(b, h) - 4 * HTB) + k * 8192); TRRD(dst[n][k][1], boff[n], (SB(b, h) - 4 * HTB) + k * 8192 + 1024); } } while (0)
#define WAIT_LB(Bt) asm volatile("s_waitcnt lgkmcnt(0)" : "+v"(Bt[0][0][0]), "+v"(Bt[0][0][1]), "+v"(Bt[0][1][0]), "+v"(Bt[0][1][1]), "+v"(Bt[1][0][0]), "+v"(Bt[1][0][1]), "+v"(Bt[1][1][0]), "+v"(Bt[1][1][1]) :: "memory")
#define BFR(Bt, n, k) __builtin_shufflevector(Bt[n][k][0], Bt[n][k][1], 0, 1, 2, 3, 4, 5, 6, 7)
#define MMA(ai, bj, At, Bt) do { __builtin_amdgcn_s_setprio(1); _Pragma("unroll") for (int m = 0; m < 4; ++m) _Pragma("unroll") for (int n = 0; n < 2; ++n) _Pragma("unroll") for (int k = 0; k < 2; ++k) \
        acc[ai][bj][m][n] = __builtin_amdgcn_mfma_f32_16x16x32_bf16(BFR(Bt, n, k), At[m][k], acc[ai][bj][m][n], 0, 0, 0); __builtin_amdgcn_s_setprio(0); } while (0)

#define STAMP(i)
#define PST(ph)
    bool bad = false;

    f32x4 acc[2][2][4][2];
#pragma unroll
    for (int a = 0; a < 2; ++a)
#pragma unroll
        for (int b = 0; b < 2; ++b)
#pragma unroll
            for (int m = 0; m < 4; ++m)
#pragma unroll
                for (int n = 0; n < 2; ++n) acc[a][b][m][n] = (f32x4){0.f, 0.f, 0.f, 0.f};
    bf16x8 At[4][2]; s16x4 B0[2][2][2], B1[2][2][2];

    STAGE(SB(0, 0), rsB, 0, voffB, voffBx); STAGE(SB(0, 1), rsB, hstepB, voffB, voffBx);
    STAGE(SB(1, 0), rsB, kstepB, voffB, voffBx); STAGE(SB(1, 1), rsB, kstepB + hstepB, voffB, voffBx);
    const unsigned ctl = (unsigned)(uintptr_t)lds + LDS_CTL;
#define CTL_WR(v) asm volatile("ds_write_b32 %0, %1" :: "v"(ctl), "v"((unsigned)(v)) : "memory")
#define BAD_WR() asm volatile("ds_write_b32 %0, %1 offset:8" :: "v"(ctl), "v"(1u) : "memory")
#define CTL_RD(r) asm volatile("ds_read_b32 %0, %1" : "=v"(r) : "v"(ctl) : "memory")
#define CTL_GET(r) ({ asm volatile("s_waitcnt lgkmcnt(0)" : "+v"(r) :: "memory"); (int)__builtin_amdgcn_readfirstlane((int)(r)); })
    if (wid == 2) { CTL_WR(0); asm volatile("ds_write_b32 %0, %1 offset:4" :: "v"(ctl), "v"(0u) : "memory"); asm volatile("ds_write_b32 %0, %1 offset:8" :: "v"(ctl), "v"(0u) : "memory"); WAIT_L(0); }
    const unsigned seq = (unsigned)__builtin_amdgcn_readfirstlane((int)seqv) / NWG + 1u;
    const unsigned par = seq & 1u;
    const __amdgpu_buffer_rsrc_t rsAr = __builtin_amdgcn_make_buffer_rsrc(local ? (void*)(p.x + (size_t)(pm & 1) * 256 * (GK * 2)) : (void*)(wsl + WS_GATHER + par * GATHER_BYTES + (size_t)pm * (NT * 32768)), 0, NT * 32768, 0x00020000);
    const unsigned* flagp = (const unsigned*)(wsl + WS_FLAGS) + pm * (NT * 32) + lane;
    unsigned* tmo = (unsigned*)(wsl + WS_TMO);
    STAMP(33);
    const unsigned ui0 = (unsigned)((((rank * 2 + ((unit >> 5) & 1)) * NT + (unit >> 6)) << 5) + (unit & 31));
    const unsigned goff = (unsigned)(WS_GATHER + par * GATHER_BYTES);
    const int pj0 = unit % 7;
    int st = (wid == 3 && ev < NEV) ? 1 : 0;
    bool flp = false;
    unsigned long long nowr = t_start;
    auto pump = [&]() __attribute__((always_inline)) {
        if (st != 0) {
            if (st < 3) {
                if (st == 1) {
                    unsigned r; asm volatile("s_getreg_b32 %0, hwreg(HW_REG_IB_STS)" : "=s"(r) :: "memory");
                    if ((r & 0xC0000Fu) == 0u) {
                        asm volatile("v_mov_b32 %0, %4\n\tv_mov_b32 %1, %5\n\tv_mov_b32 %2, %6\n\tv_mov_b32 %3, %7" : "=&v"(d0.x), "=&v"(d0.y), "=&v"(d0.z), "=&v"(d0.w) : "v"(dn.x), "v"(dn.y), "v"(dn.z), "v"(dn.w));
                        if (flp) {
                            const unsigned long long fa = wsv + WS_FLAGS + (unsigned long long)(ui0 + 128u * (unsigned)(ev - 1)) * 4u;
                            if (lane < 8 && lane != rank && !(p.dbg & 2)) asm volatile("global_store_dword %0, %1, off sc0 sc1" :: "v"(fa), "v"(seq) : "memory");
                        }
                        st = ev < NEV ? 2 : 0;
                    }
                }
                if (st == 2 && (int)((unsigned)nowr - (unsigned)t_start) >= (int)(((unsigned)(unit + 256 * ev) * (unsigned)p.slot) >> 6) + p.pslot) st = 3;
                nowr = __builtin_amdgcn_s_memrealtime();
            }
            if (st >= 3) {
                int j = pj0 + st - 3; j = j >= 7 ? j - 7 : j;
                const int pr = (p.dbg & 1) ? rank : (rank + 1 + j) & 7;
                char* base = (char*)(uintptr_t)(((unsigned long long)(unsigned)__builtin_amdgcn_readlane((int)ws_hi, pr) << 32) | (unsigned)__builtin_amdgcn_readlane((int)ws_lo, pr));
                const __amdgpu_buffer_rsrc_t rs = __builtin_amdgcn_make_buffer_rsrc((void*)base, 0, (int)WS_NEED, 0x00020000);
                __builtin_amdgcn_raw_buffer_store_b128(d0, rs, (int)(lane * 16), (int)(goff + (ui0 + 128u * (unsigned)ev) * 1024u), 17);
                if (++st == 10) {
                    flp = true; ++ev; st = 1;
                    if (ev < NEV) dn = __builtin_amdgcn_raw_buffer_load_b128(rsX, (int)vsrc, (int)(so0 + 512u * (unsigned)ev), 0);
                }
            }
        }
    };
    auto slow_wait = [&](int kt) {
        const unsigned long long t0 = __builtin_amdgcn_s_memrealtime();
        for (unsigned it = 1;; ++it) {
            const unsigned v = __hip_atomic_load(flagp + kt * 32, __ATOMIC_RELAXED, __HIP_MEMORY_SCOPE_SYSTEM);
            if (__all(v >= seq)) break;
            if (((it & 63u) == 0u && (unsigned)__builtin_amdgcn_readfirstlane(__hip_atomic_load(tmo, __ATOMIC_RELAXED, __HIP_MEMORY_SCOPE_AGENT)) == seq) || __builtin_amdgcn_s_memrealtime() - t0 > DEADLINE_TICKS) {
                __hip_atomic_store(tmo, seq, __ATOMIC_RELAXED, __HIP_MEMORY_SCOPE_AGENT); BAD_WR(); break; }
        }
        CTL_WR(kt + 2);
    };
    unsigned rv = 0;
    auto pump_until = [&](int need) __attribute__((always_inline)) {
        if (CTL_GET(rv) < need) {
            const unsigned long long t0 = __builtin_amdgcn_s_memrealtime();
            do { do pump(); while (st >= 3); CTL_RD(rv);
                if (__builtin_amdgcn_s_memrealtime() - t0 > 2 * DEADLINE_TICKS) { __hip_atomic_store(tmo, seq, __ATOMIC_RELAXED, __HIP_MEMORY_SCOPE_AGENT); BAD_WR(); break; }
            } while (CTL_GET(rv) < need);
        }
    };
    const bool poller = (wid == 2) && !nocomm;
    if (wid == 3) { nowr = __builtin_amdgcn_s_memrealtime(); pump(); while (st != 0 && ev == 0 && unit < EARLY) pump(); }
    BAR;
    if (poller) slow_wait(0);
    if (wid == 3) {
        if (!nocomm) { CTL_RD(rv); pump_until(2); }
    }
    STAMP(34);
    BAR;
    STAGE(SA(0, 0), rsAr, 0, voffA, voffAx); STAGE(SA(0, 1), rsAr, hstepA, voffA, voffAx);
    STAGE(SA(1, 0), rsAr, kstepA, voffA, voffAx);
    if (wid != 3) { WAIT_V(0); if (lane == 0) asm volatile("ds_add_u32 %0, %1 offset:4" :: "v"(ctl), "v"(1u) : "memory"); WAIT_L(0); }
    else {
        const unsigned long long t0 = __builtin_amdgcn_s_memrealtime(); unsigned cn;
        do { do pump(); while (st >= 3); asm volatile("ds_read_b32 %0, %1 offset:4" : "=v"(cn) : "v"(ctl) : "memory"); }
        while (CTL_GET(cn) < 7 && __builtin_amdgcn_s_memrealtime() - t0 < 2 * DEADLINE_TICKS);
        CTL_RD(rv);
    }
    BAR;
    if (wr == 1) BAR;
    unsigned fv = 0; bool ok = false;
    auto main_loop = [&](auto role_c) __attribute__((always_inline)) {
    constexpr int ROLE = decltype(role_c)::value;
#define RSTAGE(bufoff, rs_, soff, voff, voffx) do { \
        if constexpr (ROLE != 2) { BLDS(rs_, lds + (bufoff) + ldsw, (voff)[0], soff); BLDS(rs_, lds + (bufoff) + ldsw + 8192, (voff)[1], soff); } \
        if constexpr (ROLE == 0) BLDS(rs_, lds + (bufoff) + 3072 + wid * 8192, voffx, soff); } while (0)
#define RWAIT_STAGE() do { if constexpr (ROLE == 0) WAIT_V(15); else if constexpr (ROLE == 1) WAIT_V(10); } while (0)
#define PUMP() do { if constexpr (ROLE == 2) pump(); } while (0)
    for (int t = 0; t < NT; t += 2) {
        const bool last = (t == NT - 2);
        const unsigned a1 = (unsigned)(t + 1) * kstepA;
        const unsigned a2 = last ? 0u : (unsigned)(t + 2) * kstepA, b2 = last ? 0u : (unsigned)(t + 2) * kstepB;
        const unsigned a3 = a2 + kstepA, b3 = b2 + kstepB;
        if constexpr (ROLE == 1) {
        if (poller && !last) { if (!ok) slow_wait(t + 2); }
        }
        if constexpr (ROLE == 2) { if (!nocomm && !last) pump_until(t + 4); }
        STAMP(t >> 1);
        PST(0); PUMP(); LDB(B0, 0, 0); SCHED; LDA(At, 0, 0); RSTAGE(SA(1, 1), rsAr, a1 + hstepA, voffA, voffAx);
        RWAIT_STAGE(); WAIT_L(8); BAR; WAIT_LB(B0); MMA(0, 0, At, B0); BAR; SCHED;
        PST(1); PUMP(); LDB(B1, 0, 1); RSTAGE(SB(0, 0), rsB, b2, voffB, voffBx);
        RWAIT_STAGE();
        if constexpr (ROLE == 1) { if (poller) { const unsigned* fa = flagp + ((t + 4 < NT) ? (t + 4) : 0) * 32;
            asm volatile("global_load_dword %0, %1, off sc0 sc1" : "=v"(fv) : "v"(fa) : "memory"); } }
        BAR; WAIT_LB(B1); MMA(0, 1, At, B1); BAR;
        PST(2); PUMP(); LDA(At, 0, 1); RSTAGE(SA(0, 0), rsAr, a2, voffA, voffAx);
        BAR; WAIT_L(0); MMA(1, 0, At, B0); BAR; SCHED;
        PST(3); PUMP(); RSTAGE(SB(0, 1), rsB, b2 + hstepB, voffB, voffBx);
        RWAIT_STAGE();
        BAR; MMA(1, 1, At, B1); BAR;
        PST(4); PUMP(); LDB(B0, 1, 0); SCHED; LDA(At, 1, 0); RSTAGE(SA(0, 1), rsAr, a2 + hstepA, voffA, voffAx);
        RWAIT_STAGE(); WAIT_L(8); BAR; WAIT_LB(B0); MMA(0, 0, At, B0); BAR; SCHED;
        PST(5); PUMP(); LDB(B1, 1, 1); RSTAGE(SB(1, 0), rsB, b3, voffB, voffBx);
        RWAIT_STAGE();
        BAR; WAIT_LB(B1); MMA(0, 1, At, B1); BAR;
        PST(6); PUMP(); LDA(At, 1, 1); RSTAGE(SA(1, 0), rsAr, a3, voffA, voffAx);
        BAR; WAIT_L(0); MMA(1, 0, At, B0); BAR; SCHED;
        PST(7); PUMP(); RSTAGE(SB(1, 1), rsB, b3 + hstepB, voffB, voffBx);
        RWAIT_STAGE();
        if constexpr (ROLE == 1) { asm volatile("" : "+v"(fv)); if (poller) { ok = __all(fv >= seq); if (ok) { CTL_WR(t + 6); WAIT_L(0); } } }
        BAR;
        if constexpr (ROLE == 2) CTL_RD(rv);
        MMA(1, 1, At, B1); BAR;
        PST(8);
    }
    };
    if (wid < 2) main_loop(std::integral_constant<int, 0>{});
    else if (wid != 3) main_loop(std::integral_constant<int, 1>{});
    else main_loop(std::integral_constant<int, 2>{});
    if (wid == 3) { const unsigned long long t0 = __builtin_amdgcn_s_memrealtime(); while (st != 0 && __builtin_amdgcn_s_memrealtime() - t0 < DEADLINE_TICKS) pump(); }
    WAIT_V(0);
    WAIT_L(0);
    if (wr == 0) BAR;
    STAMP(32);

    { unsigned bw; asm volatile("ds_read_b32 %0, %1 offset:8" : "=v"(bw) : "v"(ctl) : "memory"); asm volatile("s_waitcnt lgkmcnt(0)" : "+v"(bw) :: "memory"); bad = bw != 0u; }
    {
        const int colb = pn * 256 + wc * 32 + ((fq & 1) * 16 + (fq >> 1) * 8);
        char* orow = p.out + ((size_t)(pm * 256 + wr * 64 + fr) * GN + colb) * 2;
#pragma unroll
        for (int ai = 0; ai < 2; ++ai)
#pragma unroll
            for (int m = 0; m < 4; ++m)
#pragma unroll
                for (int bj = 0; bj < 2; ++bj) {
                    const f32x4 v0 = acc[ai][bj][m][0], v1 = acc[ai][bj][m][1];
                    unsigned a0 = silu2(v0[0], v0[1]), a1 = silu2(v0[2], v0[3]);
                    unsigned b0 = silu2(v1[0], v1[1]), b1 = silu2(v1[2], v1[3]);
                    const auto s0 = __builtin_amdgcn_permlane16_swap(a0, b0, false, false);
                    const auto s1 = __builtin_amdgcn_permlane16_swap(a1, b1, false, false);
                    u32x4 o; o.x = s0[0]; o.y = s1[0]; o.z = s0[1]; o.w = s1[1];
                    if (bad) o = (u32x4){0x7fc07fc0u, 0x7fc07fc0u, 0x7fc07fc0u, 0x7fc07fc0u};
                    *(u32x4*)(orow + ((size_t)(ai * 128 + m * 16) * GN + bj * 128) * 2) = o;
                }
    }
    if (tid == 0) __hip_atomic_fetch_add((unsigned*)(wsl + WS_CTR), 1u, __ATOMIC_RELAXED, __HIP_MEMORY_SCOPE_AGENT);
}

static int g_mode = 0, g_slot = 136, g_pslot = -80, g_dbg = 0, g_pit = 10, g_late = 0;

extern "C" void kernel_launch(int rank, int nranks, ncclComm_t comm,
                              hipStream_t stream,
                              void* const* d_in, const int* in_sizes, int n_in,
                              void* d_out, int out_size,
                              const int* in_dtypes, int out_dtype,
                              void* const* d_out_all,
                              void* const* d_ws_all, size_t ws_bytes) {
    if (nranks != NR || ws_bytes < WS_NEED || (unsigned)rank >= (unsigned)NR) { fprintf(stderr, "unsupported configuration\n"); return; }
    static bool init = false;
    if (!init) {
        HIP_CHECK(hipFuncSetAttribute((const void*)ag_gemm_silu, hipFuncAttributeMaxDynamicSharedMemorySize, LDS_BYTES));
        init = true;
    }
    Params p;
    p.x = (const char*)d_in[0]; p.w = (const char*)d_in[1]; p.out = (char*)d_out;
    for (int i = 0; i < NR; ++i) p.ws[i] = (char*)d_ws_all[i];
    p.rank = rank; p.mode = g_mode; p.slot = g_slot; p.pslot = g_pslot; p.pit = g_pit; p.dbg = g_dbg;
    hipLaunchKernelGGL(ag_gemm_silu, dim3(NWG), dim3(512), LDS_BYTES, stream, p);
    HIP_CHECK(hipGetLastError());
}
```

```cpp
#include <hip/hip_runtime.h>
#include <rccl/rccl.h>
#include <cstdio>
#include <cstdint>
#include <type_traits>

#define HIP_CHECK(expr) do { hipError_t _e = (expr); if (_e != hipSuccess) { \
    fprintf(stderr, "HIP error %s at %s:%d\n", hipGetErrorString(_e), __FILE__, __LINE__); \
    return; } } while (0)

#define LAS __attribute__((address_space(3)))
#define GAS __attribute__((address_space(1)))
typedef short bf16x8 __attribute__((ext_vector_type(8)));
typedef short s16x4 __attribute__((ext_vector_type(4)));
typedef float f32x4 __attribute__((ext_vector_type(4)));
typedef unsigned u32x4 __attribute__((ext_vector_type(4)));
typedef unsigned u32x2 __attribute__((ext_vector_type(2)));

constexpr int NR = 8;
constexpr int GM = 4096, GK = 4096, GN = 4096;
constexpr int BK = 64, NT = GK / BK;
constexpr int HTB = 128 * BK * 2;
constexpr int LDS_CTL = 8 * HTB;
constexpr int LDS_BYTES = 8 * HTB + 1024;
constexpr int NWG = 256;

constexpr size_t WS_CTR = 0;
constexpr size_t WS_TMO = 256;
constexpr size_t WS_FLAGS = 4096;
constexpr size_t WS_STAMPS = 512 * 1024;
constexpr size_t WS_GATHER = (size_t)1 << 20;
constexpr size_t GATHER_BYTES = (size_t)GM * GK * 2;
constexpr size_t WS_NEED = WS_GATHER + 2 * GATHER_BYTES;
constexpr unsigned long long DEADLINE_TICKS = 20000000ull;
#define ASYM 1
#define FENCE 0
#define EARLY 128

struct Params {
    const char* x;
    const char* w;
    char* out;
    char* ws[NR];
    int rank;
    int mode;
    int dbg;
    int pit;
    int pslot;
    int slot;
};

__device__ __forceinline__ int lds_byte(int r, int c) { const int st = (r >> 4) * 2 + (c >> 5), rr = r & 15, cc = c & 31, ob = rr * 64 + cc * 2; return st * 1024 + (ob ^ (((ob >> 9) & 1) << 5)); }
__device__ __forceinline__ void stage_rc(int b, int& R, int& C) { const int st = b / 1024, sb = b % 1024, swz = sb ^ (((sb >> 9) & 1) << 5); R = (st >> 1) * 16 + swz / 64; C = (st & 1) * 32 + (swz % 64) / 2; }

__device__ __forceinline__ unsigned cvt_pk_bf16(float lo, float hi) { unsigned r; asm volatile("v_cvt_pk_bf16_f32 %0, %1, %2" : "=v"(r) : "v"(lo), "v"(hi)); return r; }
typedef float f32x2 __attribute__((ext_vector_type(2)));
__device__ __forceinline__ unsigned silu2(float y0, float y1) {
    const f32x2 y = {y0, y1};
    const f32x2 a = y * -1.4426950408889634f;
    const f32x2 d = (f32x2){__builtin_amdgcn_exp2f(__builtin_amdgcn_fmed3f(a.x, -86.5617f, 86.5617f)), __builtin_amdgcn_exp2f(__builtin_amdgcn_fmed3f(a.y, -86.5617f, 86.5617f))} + 1.0f;
    const f32x2 r = y * (f32x2){__builtin_amdgcn_rcpf(d.x), __builtin_amdgcn_rcpf(d.y)};
    return cvt_pk_bf16(r.x, r.y);
}

__global__ __launch_bounds__(512, 2) void ag_gemm_silu(const Params p) {
    extern __shared__ __attribute__((aligned(16))) unsigned char smem_raw[];
    LAS unsigned char* lds = (LAS unsigned char*)smem_raw;
    const int tid = threadIdx.x, wid = __builtin_amdgcn_readfirstlane(tid >> 6), lane = tid & 63, wr = wid >> 2, wc = wid & 3, fr = lane & 15, fq = lane >> 4;
    const unsigned long long t_start = __builtin_amdgcn_s_memrealtime();
    const int bid = blockIdx.x, xcd = bid & 7, idx = bid >> 3;
    const int pm = (xcd >> 1) * 4 + (idx & 3), pn = (xcd & 1) * 8 + (idx >> 2);
    const int rank = p.rank;
    const bool local = (pm >> 1) == rank;
    const bool nocomm = local || p.mode >= 1;
    const bool evn = (xcd & 1) == 0;
    const int uidx = (((pm - 2 * rank) & 15) << 3) + (pn & 7);
    const int unit = ASYM ? (evn ? 64 + uidx : (uidx < 64 ? uidx : uidx + 128)) : (((pm - 2 * rank) & 15) << 4) + pn;
    const int nev = ASYM ? (evn ? 20 : 12) : 16;
    const int gcyc = ASYM ? (evn ? 5 : 3) : 1, gsp = ASYM ? (evn ? 2 : 1) : 0, gdl = ASYM ? (evn ? 192 : 384) : 256;
    char* const wsl = p.ws[rank];

#define WAIT_V(n) asm volatile("s_waitcnt vmcnt(" #n ")" ::: "memory")
#define WAIT_L(n) asm volatile("s_waitcnt lgkmcnt(" #n ")" ::: "memory")
#define BAR __builtin_amdgcn_s_barrier()
#define SCHED __builtin_amdgcn_sched_barrier(0)
#define BLDS(rs, l, voff, soff) __builtin_amdgcn_raw_ptr_buffer_load_lds(rs, (LAS void*)(l), 16, (int)(voff), (int)(soff), 0, 0)
#define STAGE(bufoff, rs, soff, voff, voffx) do { \
        if (wid != 3) { BLDS(rs, lds + (bufoff) + ldsw, (voff)[0], soff); BLDS(rs, lds + (bufoff) + ldsw + 8192, (voff)[1], soff); } \
        if (wid < 2) BLDS(rs, lds + (bufoff) + 3072 + wid * 8192, voffx, soff); } while (0)
#define WAIT_STAGE() do { if (wid < 2) WAIT_V(9); else if (wid != 3) WAIT_V(6); } while (0)
    const unsigned ldsw = (unsigned)wid * 1024u;
    const __amdgpu_buffer_rsrc_t rsX = __builtin_amdgcn_make_buffer_rsrc((void*)p.x, 0, 512 * GK * 2, 0x00020000);
    const unsigned vsrc = (unsigned)((lane >> 3) * (GK * 2) + (lane & 7) * 16);
    const bool nosend = (p.mode == 1 || ((p.dbg & 4) && rank != 0) || ((p.dbg & 8) && rank == 0));
    int ev = nosend ? nev : 0;
    int g = unit, gr = 0;
#define SO(g_) ((unsigned)(((g_) & 63) * 8 * (GK * 2) + ((g_) >> 6) * 128))
    u32x4 d0 = {0u, 0u, 0u, 0u}, dn = {0u, 0u, 0u, 0u};
    if (wid == 3 && ev < nev) dn = __builtin_amdgcn_raw_buffer_load_b128(rsX, (int)vsrc, (int)SO(g), 0);
    const unsigned seqv = __hip_atomic_load((const unsigned*)(wsl + WS_CTR), __ATOMIC_RELAXED, __HIP_MEMORY_SCOPE_AGENT);
    unsigned long long wsv = (unsigned long long)(uintptr_t)p.ws[0];
#pragma unroll
    for (int i = 1; i < NR; ++i) { unsigned long long w = (unsigned long long)(uintptr_t)p.ws[i]; asm volatile("" : "+s"(w)); wsv = ((lane & 7) == i) ? w : wsv; }
    const unsigned ws_lo = (unsigned)wsv, ws_hi = (unsigned)(wsv >> 32);

    const unsigned rsA = local ? (unsigned)(GK * 2) : 128u;
    unsigned voffA[2], voffB[2], voffAx, voffBx;
#pragma unroll
    for (int i = 0; i < 3; ++i) {
        const int tt = i < 2 ? tid : 192 + lane, ii = i < 2 ? i : (wid & 1);
        int R, C; stage_rc(tt * 16 + ii * 8192, R, C);
        const unsigned va = (unsigned)R * rsA + (unsigned)C * 2u;
        const int k = (tt >> 4) + 32 * ii, c = tt & 15, f = (k & 3) | (((k >> 3) & 1) << 2);
        const unsigned vb = (unsigned)(k * (GN * 2) + ((c ^ (f << 1)) * 16));
        if (i < 2) { voffA[i] = va; voffB[i] = vb; } else { voffAx = va; voffBx = vb; }
    }
    const unsigned kstepA = local ? 128u : 32768u, hstepA = 128u * rsA;
    const unsigned kstepB = (unsigned)(BK * GN * 2), hstepB = 256u;
    const __amdgpu_buffer_rsrc_t rsB = __builtin_amdgcn_make_buffer_rsrc((void*)(p.w + pn * 512), 0, GK * GN * 2 - pn * 512, 0x00020000);
    const int aoff = lds_byte(wr * 64 + fr, fq * 8);
    int boff[2];
    { const int q = (lane >> 2) & 3, pp = lane & 3, g = fq, f = q | ((g & 1) << 2);
#pragma unroll
      for (int n = 0; n < 2; ++n) boff[n] = (int)(unsigned)(uintptr_t)lds + 4 * HTB + (8 * g + q) * 256 + (((wc * 2 + n) ^ f) * 32) + 8 * pp; }

#define SA(b, h) (((b) * 2 + (h)) * HTB)
#define SB(b, h) ((4 + (b) * 2 + (h)) * HTB)
#define LDA(dst, b, h) do { _Pragma("unroll") for (int m = 0; m < 4; ++m) _Pragma("unroll") for (int k = 0; k < 2; ++k) dst[m][k] = *(const LAS bf16x8*)(lds + SA(b, h) + aoff + m * 2048 + k * 1024); } while (0)
#define TRRD(dst, addr, off) asm volatile("ds_read_b64_tr_b16 %0, %1 offset:%2" : "=v"(dst) : "v"(addr), "i"(off) : "memory")
#define LDB(dst, b, h) do { _Pragma("unroll") for (int n = 0; n < 2; ++n) _Pragma("unroll") for (int k = 0; k < 2; ++k) { \
        TRRD(dst[n][k][0], boff[n], (SB(b, h) - 4 * HTB) + k * 8192); TRRD(dst[n][k][1], boff[n], (SB(b, h) - 4 * HTB) + k * 8192 + 1024); } } while (0)
#define WAIT_LB(Bt) asm volatile("s_waitcnt lgkmcnt(0)" : "+v"(Bt[0][0][0]), "+v"(Bt[0][0][1]), "+v"(Bt[0][1][0]), "+v"(Bt[0][1][1]), "+v"(Bt[1][0][0]), "+v"(Bt[1][0][1]), "+v"(Bt[1][1][0]), "+v"(Bt[1][1][1]) :: "memory")
#define BFR(Bt, n, k) __builtin_shufflevector(Bt[n][k][0], Bt[n][k][1], 0, 1, 2, 3, 4, 5, 6, 7)
#define MMA(ai, bj, At, Bt) do { __builtin_amdgcn_s_setprio(1); _Pragma("unroll") for (int m = 0; m < 4; ++m) _Pragma("unroll") for (int n = 0; n < 2; ++n) _Pragma("unroll") for (int k = 0; k < 2; ++k) \
        acc[ai][bj][m][n] = __builtin_amdgcn_mfma_f32_16x16x32_bf16(BFR(Bt, n, k), At[m][k], acc[ai][bj][m][n], 0, 0, 0); __builtin_amdgcn_s_setprio(0); } while (0)

#define STAMP(i)
#define PST(ph)
    bool bad = false;

    f32x4 acc[2][2][4][2];
#pragma unroll
    for (int a = 0; a < 2; ++a)
#pragma unroll
        for (int b = 0; b < 2; ++b)
#pragma unroll
            for (int m = 0; m < 4; ++m)
#pragma unroll
                for (int n = 0; n < 2; ++n) acc[a][b][m][n] = (f32x4){0.f, 0.f, 0.f, 0.f};
    bf16x8 At[4][2]; s16x4 B0[2][2][2], B1[2][2][2];

    STAGE(SB(0, 0), rsB, 0, voffB, voffBx); STAGE(SB(0, 1), rsB, hstepB, voffB, voffBx);
    STAGE(SB(1, 0), rsB, kstepB, voffB, voffBx); STAGE(SB(1, 1), rsB, kstepB + hstepB, voffB, voffBx);
    const unsigned ctl = (unsigned)(uintptr_t)lds + LDS_CTL;
#define CTL_WR(v) asm volatile("ds_write_b32 %0, %1" :: "v"(ctl), "v"((unsigned)(v)) : "memory")
#define BAD_WR() asm volatile("ds_write_b32 %0, %1 offset:8" :: "v"(ctl), "v"(1u) : "memory")
#define CTL_RD(r) asm volatile("ds_read_b32 %0, %1" : "=v"(r) : "v"(ctl) : "memory")
#define CTL_GET(r) ({ asm volatile("s_waitcnt lgkmcnt(0)" : "+v"(r) :: "memory"); (int)__builtin_amdgcn_readfirstlane((int)(r)); })
    if (wid == 2) { CTL_WR(0); asm volatile("ds_write_b32 %0, %1 offset:4" :: "v"(ctl), "v"(0u) : "memory"); asm volatile("ds_write_b32 %0, %1 offset:8" :: "v"(ctl), "v"(0u) : "memory"); WAIT_L(0); }
    const unsigned seq = (unsigned)__builtin_amdgcn_readfirstlane((int)seqv) / NWG + 1u;
    const unsigned par = seq & 1u;
    const __amdgpu_buffer_rsrc_t rsAr = __builtin_amdgcn_make_buffer_rsrc(local ? (void*)(p.x + (size_t)(pm & 1) * 256 * (GK * 2)) : (void*)(wsl + WS_GATHER + par * GATHER_BYTES + (size_t)pm * (NT * 32768)), 0, NT * 32768, 0x00020000);
    const unsigned* flagp = (const unsigned*)(wsl + WS_FLAGS) + pm * (NT * 32) + lane;
    unsigned* tmo = (unsigned*)(wsl + WS_TMO);
    STAMP(33);
#define UI(g_) ((unsigned)((((rank * 2 + (((g_) >> 5) & 1)) * NT + ((g_) >> 6)) << 5) + ((g_) & 31)))
    unsigned uic = UI(g), uip = 0;
    const unsigned goff = (unsigned)(WS_GATHER + par * GATHER_BYTES);
    const int pj0 = unit % 7;
    int st = (wid == 3 && ev < nev) ? 1 : 0;
    bool flp = false;
    unsigned long long nowr = t_start;
    auto pump = [&]() __attribute__((always_inline)) {
        if (st != 0) {
            if (st < 3) {
                if (st == 1) {
                    unsigned r; asm volatile("s_getreg_b32 %0, hwreg(HW_REG_IB_STS)" : "=s"(r) :: "memory");
                    if ((r & 0xC0000Fu) == 0u) {
                        asm volatile("v_mov_b32 %0, %4\n\tv_mov_b32 %1, %5\n\tv_mov_b32 %2, %6\n\tv_mov_b32 %3, %7" : "=&v"(d0.x), "=&v"(d0.y), "=&v"(d0.z), "=&v"(d0.w) : "v"(dn.x), "v"(dn.y), "v"(dn.z), "v"(dn.w));
                        if (flp) {
                            const unsigned long long fa = wsv + WS_FLAGS + (unsigned long long)uip * 4u;
                            if (lane < 8 && lane != rank && !(p.dbg & 2)) asm volatile("global_store_dword %0, %1, off sc0 sc1" :: "v"(fa), "v"(seq) : "memory");
                        }
                        st = ev < nev ? 2 : 0;
                    }
                }
                if (st == 2 && (int)((unsigned)nowr - (unsigned)t_start) >= (int)(((unsigned)g * (unsigned)p.slot) >> 6) + p.pslot) st = 3;
                nowr = __builtin_amdgcn_s_memrealtime();
            }
            if (st >= 3) {
                int j = pj0 + st - 3; j = j >= 7 ? j - 7 : j;
                const int pr = (p.dbg & 1) ? rank : (rank + 1 + j) & 7;
                char* base = (char*)(uintptr_t)(((unsigned long long)(unsigned)__builtin_amdgcn_readlane((int)ws_hi, pr) << 32) | (unsigned)__builtin_amdgcn_readlane((int)ws_lo, pr));
                const __amdgpu_buffer_rsrc_t rs = __builtin_amdgcn_make_buffer_rsrc((void*)base, 0, (int)WS_NEED, 0x00020000);
                __builtin_amdgcn_raw_buffer_store_b128(d0, rs, (int)(lane * 16), (int)(goff + uic * 1024u), 17);
                if (++st == 10) {
                    flp = true; ++ev; st = 1; uip = uic;
                    g += gr == gsp ? 256 : gdl; gr = gr + 1 == gcyc ? 0 : gr + 1; uic = UI(g);
                    if (ev < nev) dn = __builtin_amdgcn_raw_buffer_load_b128(rsX, (int)vsrc, (int)SO(g), 0);
                }
            }
        }
    };
    auto slow_wait = [&](int kt) {
        const unsigned long long t0 = __builtin_amdgcn_s_memrealtime();
        for (unsigned it = 1;; ++it) {
            const unsigned v = __hip_atomic_load(flagp + kt * 32, __ATOMIC_RELAXED, __HIP_MEMORY_SCOPE_SYSTEM);
            if (__all(v >= seq)) break;
            if (((it & 63u) == 0u && (unsigned)__builtin_amdgcn_readfirstlane(__hip_atomic_load(tmo, __ATOMIC_RELAXED, __HIP_MEMORY_SCOPE_AGENT)) == seq) || __builtin_amdgcn_s_memrealtime() - t0 > DEADLINE_TICKS) {
                __hip_atomic_store(tmo, seq, __ATOMIC_RELAXED, __HIP_MEMORY_SCOPE_AGENT); BAD_WR(); break; }
        }
        CTL_WR(kt + 2);
    };
    unsigned rv = 0;
    auto pump_until = [&](int need) __attribute__((always_inline)) {
        if (CTL_GET(rv) < need) {
            const unsigned long long t0 = __builtin_amdgcn_s_memrealtime();
            do { do pump(); while (st >= 3); CTL_RD(rv);
                if (__builtin_amdgcn_s_memrealtime() - t0 > 2 * DEADLINE_TICKS) { __hip_atomic_store(tmo, seq, __ATOMIC_RELAXED, __HIP_MEMORY_SCOPE_AGENT); BAD_WR(); break; }
            } while (CTL_GET(rv) < need);
        }
    };
    const bool poller = (wid == 2) && !nocomm;
    if (wid == 3) { nowr = __builtin_amdgcn_s_memrealtime(); pump(); while (st != 0 && ev == 0 && unit < EARLY) pump(); }
    BAR;
    if (poller) slow_wait(0);
    if (wid == 3) {
        if (!nocomm) { CTL_RD(rv); pump_until(2); }
    }
    STAMP(34);
    BAR;
    STAGE(SA(0, 0), rsAr, 0, voffA, voffAx); STAGE(SA(0, 1), rsAr, hstepA, voffA, voffAx);
    STAGE(SA(1, 0), rsAr, kstepA, voffA, voffAx);
    if (wid != 3) { WAIT_V(0); if (lane == 0) asm volatile("ds_add_u32 %0, %1 offset:4" :: "v"(ctl), "v"(1u) : "memory"); WAIT_L(0); }
    else {
        const unsigned long long t0 = __builtin_amdgcn_s_memrealtime(); unsigned cn;
        do { do pump(); while (st >= 3); asm volatile("ds_read_b32 %0, %1 offset:4" : "=v"(cn) : "v"(ctl) : "memory"); }
        while (CTL_GET(cn) < 7 && __builtin_amdgcn_s_memrealtime() - t0 < 2 * DEADLINE_TICKS);
        CTL_RD(rv);
    }
    BAR;
    if (wr == 1) BAR;
    unsigned fv = 0; bool ok = false;
    auto main_loop = [&](auto role_c) __attribute__((always_inline)) {
    constexpr int ROLE = decltype(role_c)::value;
#define RSTAGE(bufoff, rs_, soff, voff, voffx) do { \
        if constexpr (ROLE != 2) { BLDS(rs_, lds + (bufoff) + ldsw, (voff)[0], soff); BLDS(rs_, lds + (bufoff) + ldsw + 8192, (voff)[1], soff); } \
        if constexpr (ROLE == 0) BLDS(rs_, lds + (bufoff) + 3072 + wid * 8192, voffx, soff); } while (0)
#define RWAIT_STAGE() do { if constexpr (ROLE == 0) WAIT_V(15); else if constexpr (ROLE == 1) WAIT_V(10); } while (0)
#define PUMP() do { if constexpr (ROLE == 2) pump(); } while (0)
    for (int t = 0; t < NT; t += 2) {
        const bool last = (t == NT - 2);
        const unsigned a1 = (unsigned)(t + 1) * kstepA;
        const unsigned a2 = last ? 0u : (unsigned)(t + 2) * kstepA, b2 = last ? 0u : (unsigned)(t + 2) * kstepB;
        const unsigned a3 = a2 + kstepA, b3 = b2 + kstepB;
        if constexpr (ROLE == 1) {
        if (poller && !last) { if (!ok) slow_wait(t + 2); }
        }
        if constexpr (ROLE == 2) { if (!nocomm && !last) pump_until(t + 4); }
        STAMP(t >> 1);
        PST(0); PUMP(); LDB(B0, 0, 0); SCHED; LDA(At, 0, 0); RSTAGE(SA(1, 1), rsAr, a1 + hstepA, voffA, voffAx);
        RWAIT_STAGE(); WAIT_L(8); BAR; WAIT_LB(B0); MMA(0, 0, At, B0); BAR; SCHED;
        PST(1); PUMP(); LDB(B1, 0, 1); RSTAGE(SB(0, 0), rsB, b2, voffB, voffBx);
        RWAIT_STAGE();
        if constexpr (ROLE == 1) { if (poller) { const unsigned* fa = flagp + ((t + 4 < NT) ? (t + 4) : 0) * 32;
            asm volatile("global_load_dword %0, %1, off sc0 sc1" : "=v"(fv) : "v"(fa) : "memory"); } }
        BAR; WAIT_LB(B1); MMA(0, 1, At, B1); BAR;
        PST(2); PUMP(); LDA(At, 0, 1); RSTAGE(SA(0, 0), rsAr, a2, voffA, voffAx);
        BAR; WAIT_L(0); MMA(1, 0, At, B0); BAR; SCHED;
        PST(3); PUMP(); RSTAGE(SB(0, 1), rsB, b2 + hstepB, voffB, voffBx);
        RWAIT_STAGE();
        BAR; MMA(1, 1, At, B1); BAR;
        PST(4); PUMP(); LDB(B0, 1, 0); SCHED; LDA(At, 1, 0); RSTAGE(SA(0, 1), rsAr, a2 + hstepA, voffA, voffAx);
        RWAIT_STAGE(); WAIT_L(8); BAR; WAIT_LB(B0); MMA(0, 0, At, B0); BAR; SCHED;
        PST(5); PUMP(); LDB(B1, 1, 1); RSTAGE(SB(1, 0), rsB, b3, voffB, voffBx);
        RWAIT_STAGE();
        BAR; WAIT_LB(B1); MMA(0, 1, At, B1); BAR;
        PST(6); PUMP(); LDA(At, 1, 1); RSTAGE(SA(1, 0), rsAr, a3, voffA, voffAx);
        BAR; WAIT_L(0); MMA(1, 0, At, B0); BAR; SCHED;
        PST(7); PUMP(); RSTAGE(SB(1, 1), rsB, b3 + hstepB, voffB, voffBx);
        RWAIT_STAGE();
        if constexpr (ROLE == 1) { asm volatile("" : "+v"(fv)); if (poller) { ok = __all(fv >= seq); if (ok) { CTL_WR(t + 6); WAIT_L(0); } } }
        BAR;
        if constexpr (ROLE == 2) CTL_RD(rv);
        MMA(1, 1, At, B1); BAR;
        PST(8);
    }
    };
    if (wid < 2) main_loop(std::integral_constant<int, 0>{});
    else if (wid != 3) main_loop(std::integral_constant<int, 1>{});
    else main_loop(std::integral_constant<int, 2>{});
    if (wid == 3) { const unsigned long long t0 = __builtin_amdgcn_s_memrealtime(); while (st != 0 && __builtin_amdgcn_s_memrealtime() - t0 < DEADLINE_TICKS) pump(); }
    WAIT_V(0);
    WAIT_L(0);
    if (wr == 0) BAR;
    STAMP(32);

    { unsigned bw; asm volatile("ds_read_b32 %0, %1 offset:8" : "=v"(bw) : "v"(ctl) : "memory"); asm volatile("s_waitcnt lgkmcnt(0)" : "+v"(bw) :: "memory"); bad = bw != 0u; }
    {
        const int colb = pn * 256 + wc * 32 + ((fq & 1) * 16 + (fq >> 1) * 8);
        char* orow = p.out + ((size_t)(pm * 256 + wr * 64 + fr) * GN + colb) * 2;
#pragma unroll
        for (int ai = 0; ai < 2; ++ai)
#pragma unroll
            for (int m = 0; m < 4; ++m)
#pragma unroll
                for (int bj = 0; bj < 2; ++bj) {
                    const f32x4 v0 = acc[ai][bj][m][0], v1 = acc[ai][bj][m][1];
                    unsigned a0 = silu2(v0[0], v0[1]), a1 = silu2(v0[2], v0[3]);
                    unsigned b0 = silu2(v1[0], v1[1]), b1 = silu2(v1[2], v1[3]);
                    const auto s0 = __builtin_amdgcn_permlane16_swap(a0, b0, false, false);
                    const auto s1 = __builtin_amdgcn_permlane16_swap(a1, b1, false, false);
                    u32x4 o; o.x = s0[0]; o.y = s1[0]; o.z = s0[1]; o.w = s1[1];
                    if (bad) o = (u32x4){0x7fc07fc0u, 0x7fc07fc0u, 0x7fc07fc0u, 0x7fc07fc0u};
                    *(u32x4*)(orow + ((size_t)(ai * 128 + m * 16) * GN + bj * 128) * 2) = o;
                }
    }
    if (tid == 0) __hip_atomic_fetch_add((unsigned*)(wsl + WS_CTR), 1u, __ATOMIC_RELAXED, __HIP_MEMORY_SCOPE_AGENT);
}

static int g_mode = 0, g_slot = 136, g_pslot = -80, g_dbg = 0, g_pit = 10, g_late = 0;

extern "C" void kernel_launch(int rank, int nranks, ncclComm_t comm,
                              hipStream_t stream,
                              void* const* d_in, const int* in_sizes, int n_in,
                              void* d_out, int out_size,
                              const int* in_dtypes, int out_dtype,
                              void* const* d_out_all,
                              void* const* d_ws_all, size_t ws_bytes) {
    if (nranks != NR || ws_bytes < WS_NEED || (unsigned)rank >= (unsigned)NR) { fprintf(stderr, "unsupported configuration\n"); return; }
    static bool init = false;
    if (!init) {
        HIP_CHECK(hipFuncSetAttribute((const void*)ag_gemm_silu, hipFuncAttributeMaxDynamicSharedMemorySize, LDS_BYTES));
        init = true;
    }
    Params p;
    p.x = (const char*)d_in[0]; p.w = (const char*)d_in[1]; p.out = (char*)d_out;
    for (int i = 0; i < NR; ++i) p.ws[i] = (char*)d_ws_all[i];
    p.rank = rank; p.mode = g_mode; p.slot = g_slot; p.pslot = g_pslot; p.pit = g_pit; p.dbg = g_dbg;
    hipLaunchKernelGGL(ag_gemm_silu, dim3(NWG), dim3(512), LDS_BYTES, stream, p);
    HIP_CHECK(hipGetLastError());
}
```

```cpp
#include <hip/hip_runtime.h>
#include <rccl/rccl.h>
#include <cstdio>
#include <cstdint>
#include <type_traits>

#define HIP_CHECK(expr) do { hipError_t _e = (expr); if (_e != hipSuccess) { \
    fprintf(stderr, "HIP error %s at %s:%d\n", hipGetErrorString(_e), __FILE__, __LINE__); \
    return; } } while (0)

#define LAS __attribute__((address_space(3)))
#define GAS __attribute__((address_space(1)))
typedef short bf16x8 __attribute__((ext_vector_type(8)));
typedef short s16x4 __attribute__((ext_vector_type(4)));
typedef float f32x4 __attribute__((ext_vector_type(4)));
typedef unsigned u32x4 __attribute__((ext_vector_type(4)));
typedef unsigned u32x2 __attribute__((ext_vector_type(2)));

constexpr int NR = 8;
constexpr int GM = 4096, GK = 4096, GN = 4096;
constexpr int BK = 64, NT = GK / BK;
constexpr int HTB = 128 * BK * 2;
constexpr int LDS_CTL = 8 * HTB;
constexpr int LDS_BYTES = 8 * HTB + 1024;
constexpr int NWG = 256;

constexpr size_t WS_CTR = 0;
constexpr size_t WS_TMO = 256;
constexpr size_t WS_FLAGS = 4096;
constexpr size_t WS_STAMPS = 512 * 1024;
constexpr int NEV = 16;
constexpr size_t WS_GATHER = (size_t)1 << 20;
constexpr size_t GATHER_BYTES = (size_t)GM * GK * 2;
constexpr size_t WS_NEED = WS_GATHER + 2 * GATHER_BYTES;
constexpr unsigned long long DEADLINE_TICKS = 20000000ull;
#define USH 0
#define FENCE 0
#define EARLY 128

struct Params {
    const char* x;
    const char* w;
    char* out;
    char* ws[NR];
    int rank;
    int mode;
    int dbg;
    int pit;
    int pslot;
    int slot;
};

__device__ __forceinline__ int lds_byte(int r, int c) { const int st = (r >> 4) * 2 + (c >> 5), rr = r & 15, cc = c & 31, ob = rr * 64 + cc * 2; return st * 1024 + (ob ^ (((ob >> 9) & 1) << 5)); }
__device__ __forceinline__ void stage_rc(int b, int& R, int& C) { const int st = b / 1024, sb = b % 1024, swz = sb ^ (((sb >> 9) & 1) << 5); R = (st >> 1) * 16 + swz / 64; C = (st & 1) * 32 + (swz % 64) / 2; }

__device__ __forceinline__ unsigned cvt_pk_bf16(float lo, float hi) { unsigned r; asm volatile("v_cvt_pk_bf16_f32 %0, %1, %2" : "=v"(r) : "v"(lo), "v"(hi)); return r; }
typedef float f32x2 __attribute__((ext_vector_type(2)));
__device__ __forceinline__ unsigned silu2(float y0, float y1) {
    const f32x2 y = {y0, y1};
    const f32x2 a = y * -1.4426950408889634f;
    const f32x2 d = (f32x2){__builtin_amdgcn_exp2f(__builtin_amdgcn_fmed3f(a.x, -86.5617f, 86.5617f)), __builtin_amdgcn_exp2f(__builtin_amdgcn_fmed3f(a.y, -86.5617f, 86.5617f))} + 1.0f;
    const f32x2 r = y * (f32x2){__builtin_amdgcn_rcpf(d.x), __builtin_amdgcn_rcpf(d.y)};
    return cvt_pk_bf16(r.x, r.y);
}

__global__ __launch_bounds__(512, 2) void ag_gemm_silu(const Params p) {
    extern __shared__ __attribute__((aligned(16))) unsigned char smem_raw[];
    LAS unsigned char* lds = (LAS unsigned char*)smem_raw;
    const int tid = threadIdx.x, wid = __builtin_amdgcn_readfirstlane(tid >> 6), lane = tid & 63, wr = wid >> 2, wc = wid & 3, fr = lane & 15, fq = lane >> 4;
    const unsigned long long t_start = __builtin_amdgcn_s_memrealtime();
    const int bid = blockIdx.x, xcd = bid & 7, idx = bid >> 3;
    const int pm = (xcd >> 1) * 4 + (idx & 3), pn = (xcd & 1) * 8 + (idx >> 2);
    const int rank = p.rank;
    const bool local = (pm >> 1) == rank;
    const bool nocomm = local || p.mode >= 1;
    const int unit = (((pm - 2 * rank - USH) & 15) << 4) + pn;
    char* const wsl = p.ws[rank];

#define WAIT_V(n) asm volatile("s_waitcnt vmcnt(" #n ")" ::: "memory")
#define WAIT_L(n) asm volatile("s_waitcnt lgkmcnt(" #n ")" ::: "memory")
#define BAR __builtin_amdgcn_s_barrier()
#define SCHED __builtin_amdgcn_sched_barrier(0)
#define BLDS(rs, l, voff, soff) __builtin_amdgcn_raw_ptr_buffer_load_lds(rs, (LAS void*)(l), 16, (int)(voff), (int)(soff), 0, 0)
#define STAGE(bufoff, rs, soff, voff, voffx) do { \
        if (wid != 3) { BLDS(rs, lds + (bufoff) + ldsw, (voff)[0], soff); BLDS(rs, lds + (bufoff) + ldsw + 8192, (voff)[1], soff); } \
        if (wid < 2) BLDS(rs, lds + (bufoff) + 3072 + wid * 8192, voffx, soff); } while (0)
#define WAIT_STAGE() do { if (wid < 2) WAIT_V(9); else if (wid != 3) WAIT_V(6); } while (0)
    const unsigned ldsw = (unsigned)wid * 1024u;
    const __amdgpu_buffer_rsrc_t rsX = __builtin_amdgcn_make_buffer_rsrc((void*)p.x, 0, 512 * GK * 2, 0x00020000);
    const unsigned vsrc = (unsigned)((lane >> 3) * (GK * 2) + (lane & 7) * 16);
    const bool nosend = (p.mode == 1 || ((p.dbg & 4) && rank != 0) || ((p.dbg & 8) && rank == 0));
    int ev = nosend ? NEV : 0;
    const unsigned so0 = (unsigned)((unit & 63) * 8 * (GK * 2) + (unit >> 6) * 128);
    u32x4 d0 = {0u, 0u, 0u, 0u}, dn = {0u, 0u, 0u, 0u};
    if (wid == 3 && ev < NEV) dn = __builtin_amdgcn_raw_buffer_load_b128(rsX, (int)vsrc, (int)so0, 0);
    const unsigned seqv = __hip_atomic_load((const unsigned*)(wsl + WS_CTR), __ATOMIC_RELAXED, __HIP_MEMORY_SCOPE_AGENT);
    unsigned long long wsv = (unsigned long long)(uintptr_t)p.ws[0];
#pragma unroll
    for (int i = 1; i < NR; ++i) { unsigned long long w = (unsigned long long)(uintptr_t)p.ws[i]; asm volatile("" : "+s"(w)); wsv = ((lane & 7) == i) ? w : wsv; }
    const unsigned ws_lo = (unsigned)wsv, ws_hi = (unsigned)(wsv >> 32);

    const unsigned rsA = local ? (unsigned)(GK * 2) : 128u;
    unsigned voffA[2], voffB[2], voffAx, voffBx;
#pragma unroll
    for (int i = 0; i < 3; ++i) {
        const int tt = i < 2 ? tid : 192 + lane, ii = i < 2 ? i : (wid & 1);
        int R, C; stage_rc(tt * 16 + ii * 8192, R, C);
        const unsigned va = (unsigned)R * rsA + (unsigned)C * 2u;
        const int k = (tt >> 4) + 32 * ii, c = tt & 15, f = (k & 3) | (((k >> 3) & 1) << 2);
        const unsigned vb = (unsigned)(k * (GN * 2) + ((c ^ (f << 1)) * 16));
        if (i < 2) { voffA[i] = va; voffB[i] = vb; } else { voffAx = va; voffBx = vb; }
    }
    const unsigned kstepA = local ? 128u : 32768u, hstepA = 128u * rsA;
    const unsigned kstepB = (unsigned)(BK * GN * 2), hstepB = 256u;
    const __amdgpu_buffer_rsrc_t rsB = __builtin_amdgcn_make_buffer_rsrc((void*)(p.w + pn * 512), 0, GK * GN * 2 - pn * 512, 0x00020000);
    const int aoff = lds_byte(wr * 64 + fr, fq * 8);
    int boff[2];
    { const int q = (lane >> 2) & 3, pp = lane & 3, g = fq, f = q | ((g & 1) << 2);
#pragma unroll
      for (int n = 0; n < 2; ++n) boff[n] = (int)(unsigned)(uintptr_t)lds + 4 * HTB + (8 * g + q) * 256 + (((wc * 2 + n) ^ f) * 32) + 8 * pp; }

#define SA(b, h) (((b) * 2 + (h)) * HTB)
#define SB(b, h) ((4 + (b) * 2 + (h)) * HTB)
#define LDA(dst, b, h) do { _Pragma("unroll") for (int m = 0; m < 4; ++m) _Pragma("unroll") for (int k = 0; k < 2; ++k) dst[m][k] = *(const LAS bf16x8*)(lds + SA(b, h) + aoff + m * 2048 + k * 1024); } while (0)
#define TRRD(dst, addr, off) asm volatile("ds_read_b64_tr_b16 %0, %1 offset:%2" : "=v"(dst) : "v"(addr), "i"(off) : "memory")
#define LDB(dst, b, h) do { _Pragma("unroll") for (int n = 0; n < 2; ++n) _Pragma("unroll") for (int k = 0; k < 2; ++k) { \
        TRRD(dst[n][k][0], boff[n], (SB(b, h) - 4 * HTB) + k * 8192); TRRD(dst[n][k][1], boff[n], (SB(b, h) - 4 * HTB) + k * 8192 + 1024); } } while (0)
#define WAIT_LB(Bt) asm volatile("s_waitcnt lgkmcnt(0)" : "+v"(Bt[0][0][0]), "+v"(Bt[0][0][1]), "+v"(Bt[0][1][0]), "+v"(Bt[0][1][1]), "+v"(Bt[1][0][0]), "+v"(Bt[1][0][1]), "+v"(Bt[1][1][0]), "+v"(Bt[1][1][1]) :: "memory")
#define BFR(Bt, n, k) __builtin_shufflevector(Bt[n][k][0], Bt[n][k][1], 0, 1, 2, 3, 4, 5, 6, 7)
#define MMA(ai, bj, At, Bt) do { __builtin_amdgcn_s_setprio(1); _Pragma("unroll") for (int m = 0; m < 4; ++m) _Pragma("unroll") for (int n = 0; n < 2; ++n) _Pragma("unroll") for (int k = 0; k < 2; ++k) \
        acc[ai][bj][m][n] = __builtin_amdgcn_mfma_f32_16x16x32_bf16(BFR(Bt, n, k), At[m][k], acc[ai][bj][m][n], 0, 0, 0); __builtin_amdgcn_s_setprio(0); } while (0)

#define STAMP(i)
#define PST(ph)
    bool bad = false;

    f32x4 acc[2][2][4][2];
#pragma unroll
    for (int a = 0; a < 2; ++a)
#pragma unroll
        for (int b = 0; b < 2; ++b)
#pragma unroll
            for (int m = 0; m < 4; ++m)
#pragma unroll
                for (int n = 0; n < 2; ++n) acc[a][b][m][n] = (f32x4){0.f, 0.f, 0.f, 0.f};
    bf16x8 At[4][2]; s16x4 B0[2][2][2], B1[2][2][2];

    STAGE(SB(0, 0), rsB, 0, voffB, voffBx); STAGE(SB(0, 1), rsB, hstepB, voffB, voffBx);
    STAGE(SB(1, 0), rsB, kstepB, voffB, voffBx); STAGE(SB(1, 1), rsB, kstepB + hstepB, voffB, voffBx);
    const unsigned ctl = (unsigned)(uintptr_t)lds + LDS_CTL;
#define CTL_WR(v) asm volatile("ds_write_b32 %0, %1" :: "v"(ctl), "v"((unsigned)(v)) : "memory")
#define BAD_WR() asm volatile("ds_write_b32 %0, %1 offset:8" :: "v"(ctl), "v"(1u) : "memory")
#define CTL_RD(r) asm volatile("ds_read_b32 %0, %1" : "=v"(r) : "v"(ctl) : "memory")
#define CTL_GET(r) ({ asm volatile("s_waitcnt lgkmcnt(0)" : "+v"(r) :: "memory"); (int)__builtin_amdgcn_readfirstlane((int)(r)); })
    if (wid == 2) { CTL_WR(0); asm volatile("ds_write_b32 %0, %1 offset:4" :: "v"(ctl), "v"(0u) : "memory"); asm volatile("ds_write_b32 %0, %1 offset:8" :: "v"(ctl), "v"(0u) : "memory"); WAIT_L(0); }
    const unsigned seq = (unsigned)__builtin_amdgcn_readfirstlane((int)seqv) / NWG + 1u;
    const unsigned par = seq & 1u;
    const __amdgpu_buffer_rsrc_t rsAr = __builtin_amdgcn_make_buffer_rsrc(local ? (void*)(p.x + (size_t)(pm & 1) * 256 * (GK * 2)) : (void*)(wsl + WS_GATHER + par * GATHER_BYTES + (size_t)pm * (NT * 32768)), 0, NT * 32768, 0x00020000);
    const unsigned* flagp = (const unsigned*)(wsl + WS_FLAGS) + pm * (NT * 32) + lane;
    unsigned* tmo = (unsigned*)(wsl + WS_TMO);
    STAMP(33);
    const unsigned ui0 = (unsigned)((((rank * 2 + ((unit >> 5) & 1)) * NT + (unit >> 6)) << 5) + (unit & 31));
    const unsigned goff = (unsigned)(WS_GATHER + par * GATHER_BYTES);
    const int pj0 = unit % 7;
    int st = (wid == 3 && ev < NEV) ? 1 : 0;
    bool flp = false;
    unsigned long long nowr = t_start;
    auto pump = [&]() __attribute__((always_inline)) {
        if (st != 0) {
            if (st < 3) {
                if (st == 1) {
                    unsigned r; asm volatile("s_getreg_b32 %0, hwreg(HW_REG_IB_STS)" : "=s"(r) :: "memory");
                    if ((r & 0xC0000Fu) == 0u) {
                        asm volatile("v_mov_b32 %0, %4\n\tv_mov_b32 %1, %5\n\tv_mov_b32 %2, %6\n\tv_mov_b32 %3, %7" : "=&v"(d0.x), "=&v"(d0.y), "=&v"(d0.z), "=&v"(d0.w) : "v"(dn.x), "v"(dn.y), "v"(dn.z), "v"(dn.w));
                        if (flp) {
                            const unsigned long long fa = wsv + WS_FLAGS + (unsigned long long)(ui0 + 128u * (unsigned)(ev - 1)) * 4u;
                            if (lane < 8 && lane != rank && !(p.dbg & 2)) asm volatile("global_store_dword %0, %1, off sc0 sc1" :: "v"(fa), "v"(seq) : "memory");
                        }
                        st = ev < NEV ? 2 : 0;
                    }
                }
                if (st == 2 && (int)((unsigned)nowr - (unsigned)t_start) >= (int)(((unsigned)(unit + 256 * ev) * (unsigned)p.slot) >> 6) + p.pslot) st = 3;
                nowr = __builtin_amdgcn_s_memrealtime();
            }
            if (st >= 3) {
                int j = pj0 + st - 3; j = j >= 7 ? j - 7 : j;
                const int pr = (p.dbg & 1) ? rank : (rank + 1 + j) & 7;
                char* base = (char*)(uintptr_t)(((unsigned long long)(unsigned)__builtin_amdgcn_readlane((int)ws_hi, pr) << 32) | (unsigned)__builtin_amdgcn_readlane((int)ws_lo, pr));
                const __amdgpu_buffer_rsrc_t rs = __builtin_amdgcn_make_buffer_rsrc((void*)base, 0, (int)WS_NEED, 0x00020000);
                __builtin_amdgcn_raw_buffer_store_b128(d0, rs, (int)(lane * 16), (int)(goff + (ui0 + 128u * (unsigned)ev) * 1024u), 17);
                if (++st == 10) {
                    flp = true; ++ev; st = 1;
                    if (ev < NEV) dn = __builtin_amdgcn_raw_buffer_load_b128(rsX, (int)vsrc, (int)(so0 + 512u * (unsigned)ev), 0);
                }
            }
        }
    };
    auto slow_wait = [&](int kt) {
        const unsigned long long t0 = __builtin_amdgcn_s_memrealtime();
        for (unsigned it = 1;; ++it) {
            const unsigned v = __hip_atomic_load(flagp + kt * 32, __ATOMIC_RELAXED, __HIP_MEMORY_SCOPE_SYSTEM);
            if (__all(v >= seq)) break;
            if (((it & 63u) == 0u && (unsigned)__builtin_amdgcn_readfirstlane(__hip_atomic_load(tmo, __ATOMIC_RELAXED, __HIP_MEMORY_SCOPE_AGENT)) == seq) || __builtin_amdgcn_s_memrealtime() - t0 > DEADLINE_TICKS) {
                __hip_atomic_store(tmo, seq, __ATOMIC_RELAXED, __HIP_MEMORY_SCOPE_AGENT); BAD_WR(); break; }
        }
        CTL_WR(kt + 2);
    };
    unsigned rv = 0;
    auto pump_until = [&](int need) __attribute__((always_inline)) {
        if (CTL_GET(rv) < need) {
            const unsigned long long t0 = __builtin_amdgcn_s_memrealtime();
            do { do pump(); while (st >= 3); CTL_RD(rv);
                if (__builtin_amdgcn_s_memrealtime() - t0 > 2 * DEADLINE_TICKS) { __hip_atomic_store(tmo, seq, __ATOMIC_RELAXED, __HIP_MEMORY_SCOPE_AGENT); BAD_WR(); break; }
            } while (CTL_GET(rv) < need);
        }
    };
    const bool poller = (wid == 2) && !nocomm;
    if (wid == 3) { nowr = __builtin_amdgcn_s_memrealtime(); pump(); while (st != 0 && ev == 0 && unit < EARLY && (unsigned)nowr - (unsigned)t_start < (unsigned)DEADLINE_TICKS) pump(); }
    BAR;
    if (poller) slow_wait(0);
    if (wid == 3) {
        if (!nocomm) { CTL_RD(rv); pump_until(2); }
    }
    STAMP(34);
    BAR;
    STAGE(SA(0, 0), rsAr, 0, voffA, voffAx); STAGE(SA(0, 1), rsAr, hstepA, voffA, voffAx);
    STAGE(SA(1, 0), rsAr, kstepA, voffA, voffAx);
    if (wid != 3) { WAIT_V(0); if (lane == 0) asm volatile("ds_add_u32 %0, %1 offset:4" :: "v"(ctl), "v"(1u) : "memory"); WAIT_L(0); }
    else {
        const unsigned long long t0 = __builtin_amdgcn_s_memrealtime(); unsigned cn;
        do { do pump(); while (st >= 3); asm volatile("ds_read_b32 %0, %1 offset:4" : "=v"(cn) : "v"(ctl) : "memory"); }
        while (CTL_GET(cn) < 7 && __builtin_amdgcn_s_memrealtime() - t0 < 2 * DEADLINE_TICKS);
        CTL_RD(rv);
    }
    BAR;
    if (wr == 1) BAR;
    unsigned fv = 0; bool ok = false;
    auto main_loop = [&](auto role_c) __attribute__((always_inline)) {
    constexpr int ROLE = decltype(role_c)::value;
#define RSTAGE(bufoff, rs_, soff, voff, voffx) do { \
        if constexpr (ROLE != 2) { BLDS(rs_, lds + (bufoff) + ldsw, (voff)[0], soff); BLDS(rs_, lds + (bufoff) + ldsw + 8192, (voff)[1], soff); } \
        if constexpr (ROLE == 0) BLDS(rs_, lds + (bufoff) + 3072 + wid * 8192, voffx, soff); } while (0)
#define RWAIT_STAGE() do { if constexpr (ROLE == 0) WAIT_V(15); else if constexpr (ROLE == 1) WAIT_V(10); } while (0)
#define PUMP() do { if constexpr (ROLE == 2) pump(); } while (0)
    for (int t = 0; t < NT; t += 2) {
        const bool last = (t == NT - 2);
        const unsigned a1 = (unsigned)(t + 1) * kstepA;
        const unsigned a2 = last ? 0u : (unsigned)(t + 2) * kstepA, b2 = last ? 0u : (unsigned)(t + 2) * kstepB;
        const unsigned a3 = a2 + kstepA, b3 = b2 + kstepB;
        if constexpr (ROLE == 1) {
        if (poller && !last) { if (!ok) slow_wait(t + 2); }
        }
        if constexpr (ROLE == 2) { if (!nocomm && !last) pump_until(t + 4); }
        STAMP(t >> 1);
        PST(0); PUMP(); LDB(B0, 0, 0); SCHED; LDA(At, 0, 0); RSTAGE(SA(1, 1), rsAr, a1 + hstepA, voffA, voffAx);
        RWAIT_STAGE(); WAIT_L(8); BAR; WAIT_LB(B0); MMA(0, 0, At, B0); BAR; SCHED;
        PST(1); PUMP(); LDB(B1, 0, 1); RSTAGE(SB(0, 0), rsB, b2, voffB, voffBx);
        RWAIT_STAGE();
        if constexpr (ROLE == 1) { if (poller) { const unsigned* fa = flagp + ((t + 4 < NT) ? (t + 4) : 0) * 32;
            asm volatile("global_load_dword %0, %1, off sc0 sc1" : "=v"(fv) : "v"(fa) : "memory"); } }
        BAR; WAIT_LB(B1); MMA(0, 1, At, B1); BAR;
        PST(2); PUMP(); LDA(At, 0, 1); RSTAGE(SA(0, 0), rsAr, a2, voffA, voffAx);
        BAR; WAIT_L(0); MMA(1, 0, At, B0); BAR; SCHED;
        PST(3); PUMP(); RSTAGE(SB(0, 1), rsB, b2 + hstepB, voffB, voffBx);
        RWAIT_STAGE();
        BAR; MMA(1, 1, At, B1); BAR;
        PST(4); PUMP(); LDB(B0, 1, 0); SCHED; LDA(At, 1, 0); RSTAGE(SA(0, 1), rsAr, a2 + hstepA, voffA, voffAx);
        RWAIT_STAGE(); WAIT_L(8); BAR; WAIT_LB(B0); MMA(0, 0, At, B0); BAR; SCHED;
        PST(5); PUMP(); LDB(B1, 1, 1); RSTAGE(SB(1, 0), rsB, b3, voffB, voffBx);
        RWAIT_STAGE();
        BAR; WAIT_LB(B1); MMA(0, 1, At, B1); BAR;
        PST(6); PUMP(); LDA(At, 1, 1); RSTAGE(SA(1, 0), rsAr, a3, voffA, voffAx);
        BAR; WAIT_L(0); MMA(1, 0, At, B0); BAR; SCHED;
        PST(7); PUMP(); RSTAGE(SB(1, 1), rsB, b3 + hstepB, voffB, voffBx);
        RWAIT_STAGE();
        if constexpr (ROLE == 1) { asm volatile("" : "+v"(fv)); if (poller) { ok = __all(fv >= seq); if (ok) { CTL_WR(t + 6); WAIT_L(0); } } }
        BAR;
        if constexpr (ROLE == 2) CTL_RD(rv);
        MMA(1, 1, At, B1); BAR;
        PST(8);
    }
    };
    if (wid < 2) main_loop(std::integral_constant<int, 0>{});
    else if (wid != 3) main_loop(std::integral_constant<int, 1>{});
    else main_loop(std::integral_constant<int, 2>{});
    if (wid == 3) { const unsigned long long t0 = __builtin_amdgcn_s_memrealtime(); while (st != 0 && __builtin_amdgcn_s_memrealtime() - t0 < DEADLINE_TICKS) pump(); }
    WAIT_V(0);
    WAIT_L(0);
    if (wr == 0) BAR;
    STAMP(32);

    { unsigned bw; asm volatile("ds_read_b32 %0, %1 offset:8" : "=v"(bw) : "v"(ctl) : "memory"); asm volatile("s_waitcnt lgkmcnt(0)" : "+v"(bw) :: "memory"); bad = bw != 0u; }
    {
        const int colb = pn * 256 + wc * 32 + ((fq & 1) * 16 + (fq >> 1) * 8);
        char* orow = p.out + ((size_t)(pm * 256 + wr * 64 + fr) * GN + colb) * 2;
#pragma unroll
        for (int ai = 0; ai < 2; ++ai)
#pragma unroll
            for (int m = 0; m < 4; ++m)
#pragma unroll
                for (int bj = 0; bj < 2; ++bj) {
                    const f32x4 v0 = acc[ai][bj][m][0], v1 = acc[ai][bj][m][1];
                    unsigned a0 = silu2(v0[0], v0[1]), a1 = silu2(v0[2], v0[3]);
                    unsigned b0 = silu2(v1[0], v1[1]), b1 = silu2(v1[2], v1[3]);
                    const auto s0 = __builtin_amdgcn_permlane16_swap(a0, b0, false, false);
                    const auto s1 = __builtin_amdgcn_permlane16_swap(a1, b1, false, false);
                    u32x4 o; o.x = s0[0]; o.y = s1[0]; o.z = s0[1]; o.w = s1[1];
                    if (bad) o = (u32x4){0x7fc07fc0u, 0x7fc07fc0u, 0x7fc07fc0u, 0x7fc07fc0u};
                    *(u32x4*)(orow + ((size_t)(ai * 128 + m * 16) * GN + bj * 128) * 2) = o;
                }
    }
    if (tid == 0) __hip_atomic_fetch_add((unsigned*)(wsl + WS_CTR), 1u, __ATOMIC_RELAXED, __HIP_MEMORY_SCOPE_AGENT);
}

static int g_mode = 0, g_slot = 136, g_pslot = -80, g_dbg = 0, g_pit = 10, g_late = 0;

extern "C" void kernel_launch(int rank, int nranks, ncclComm_t comm,
                              hipStream_t stream,
                              void* const* d_in, const int* in_sizes, int n_in,
                              void* d_out, int out_size,
                              const int* in_dtypes, int out_dtype,
                              void* const* d_out_all,
                              void* const* d_ws_all, size_t ws_bytes) {
    if (nranks != NR || ws_bytes < WS_NEED || (unsigned)rank >= (unsigned)NR) { fprintf(stderr, "unsupported configuration\n"); return; }
    static bool init = false;
    if (!init) {
        HIP_CHECK(hipFuncSetAttribute((const void*)ag_gemm_silu, hipFuncAttributeMaxDynamicSharedMemorySize, LDS_BYTES));
        init = true;
    }
    Params p;
    p.x = (const char*)d_in[0]; p.w = (const char*)d_in[1]; p.out = (char*)d_out;
    for (int i = 0; i < NR; ++i) p.ws[i] = (char*)d_ws_all[i];
    p.rank = rank; p.mode = g_mode; p.slot = g_slot; p.pslot = g_pslot; p.pit = g_pit; p.dbg = g_dbg;
    hipLaunchKernelGGL(ag_gemm_silu, dim3(NWG), dim3(512), LDS_BYTES, stream, p);
    HIP_CHECK(hipGetLastError());
}
```
